```python
import numpy as np

kernel_name = "gpu_gemm_a2a_m8192_k8192_n8192_f32_silu"


ABSMAX_RTOL = 0.001

COMPUTE_DTYPE = "float32"
EPILOGUE = "silu"
IN_QUANT = None
OUT_QUANT = None
IN_DTYPES = {"x": "float32", "w_mat": "float32"}
WORLD_SIZE = 8

def setup_inputs():

    x = np.random.randn(8192, 8192).astype(np.float32)
    w_mat = (np.random.randn(8192, 8192) / np.sqrt(8192)).astype(np.float32)
    return {"x": x, "w_mat": w_mat}

def _epilogue(y):
    return (y / (1.0 + np.exp(-np.clip(y, -60.0, 60.0)))).astype(np.float32)

def reference(x, w_mat):
    y = (x @ w_mat).astype(np.float32)
    return _epilogue(y)
```

```cpp
#include <hip/hip_runtime.h>
#include <rccl/rccl.h>
#include <cstdio>

#define HIP_CHECK(expr) do { hipError_t _e = (expr); if (_e != hipSuccess) { \
    fprintf(stderr, "HIP error %s at %s:%d\n", hipGetErrorString(_e), __FILE__, __LINE__); \
    return; } } while (0)

typedef _Float16 h2 __attribute__((ext_vector_type(2)));
typedef float f4 __attribute__((ext_vector_type(4)));
typedef float f2 __attribute__((ext_vector_type(2)));
typedef unsigned u4 __attribute__((ext_vector_type(4)));
#define LAS __attribute__((address_space(3)))
#define GAS __attribute__((address_space(1)))

#define WAUX 0
#define SLOT2 0
#define NOSEND 0
#define XSC "sc1"
#define NODUMP 0
#define NOREAD 0
#define DMASK 0
#define DSC "sc1"
#define OAUX 17
#define PAUX 16
#define CP 2
#define RWIN 8
constexpr int NR = 8;
constexpr int M = 1024, N = 8192, K = 8192;
constexpr int XS = 16384;
constexpr int LDS_BYTES = 8 * XS;
constexpr unsigned WS_TMO = 128, WS_TICKET = 256, WS_START = 1024, WS_DONE = 2048;
constexpr unsigned WS_XF = 4096;
constexpr unsigned WS_PF = 16384;
constexpr size_t WS_XS = 1u << 20, WS_PART = 17u << 20;
constexpr unsigned PART_BYTES = 200u << 20;
constexpr size_t WS_NEED = WS_PART + PART_BYTES;
constexpr unsigned long long DEADLINE = 40000000ull;

#define ACLOB "a0","a1","a2","a3","a4","a5","a6","a7","a8","a9","a10","a11","a12","a13","a14","a15", \
    "a16","a17","a18","a19","a20","a21","a22","a23","a24","a25","a26","a27","a28","a29","a30","a31", \
    "a32","a33","a34","a35","a36","a37","a38","a39","a40","a41","a42","a43","a44","a45","a46","a47", \
    "a48","a49","a50","a51","a52","a53","a54","a55","a56","a57","a58","a59","a60","a61","a62","a63", \
    "a64","a65","a66","a67","a68","a69","a70","a71","a72","a73","a74","a75","a76","a77","a78","a79", \
    "a80","a81","a82","a83","a84","a85","a86","a87","a88","a89","a90","a91","a92","a93","a94","a95", \
    "a96","a97","a98","a99","a100","a101","a102","a103","a104","a105","a106","a107","a108","a109","a110","a111", \
    "a112","a113","a114","a115","a116","a117","a118","a119","a120","a121","a122","a123","a124","a125","a126","a127", \
    "a128","a129","a130","a131","a132","a133","a134","a135","a136","a137","a138","a139","a140","a141","a142","a143", \
    "a144","a145","a146","a147","a148","a149","a150","a151","a152","a153","a154","a155","a156","a157","a158","a159", \
    "a160","a161","a162","a163","a164","a165","a166","a167","a168","a169","a170","a171","a172","a173","a174","a175", \
    "a176","a177","a178","a179","a180","a181","a182","a183","a184","a185","a186","a187","a188","a189","a190","a191", \
    "a192","a193","a194","a195","a196","a197","a198","a199","a200","a201","a202","a203","a204","a205","a206","a207", \
    "a208","a209","a210","a211","a212","a213","a214","a215","a216","a217","a218","a219","a220","a221","a222","a223", \
    "a224","a225","a226","a227","a228","a229","a230","a231","a232","a233","a234","a235","a236","a237","a238","a239", \
    "a240","a241","a242","a243","a244","a245","a246","a247","a248","a249","a250","a251","a252","a253","a254","a255"
#define BAR() do { asm volatile("" ::: "memory"); __builtin_amdgcn_s_barrier(); asm volatile("" ::: "memory"); } while (0)
#define SCHED() __builtin_amdgcn_sched_barrier(0)
#define WAIT_V(n) asm volatile("s_waitcnt vmcnt(%0)" ::"n"(n) : "memory")

__device__ __forceinline__ unsigned cvt2(float a, float b) { f2 v = {a, b}; h2 h = __builtin_convertvector(v, h2); return __builtin_bit_cast(unsigned, h); }
__device__ __forceinline__ float silu(float y) {
    const float yc = __builtin_amdgcn_fmed3f(y, -60.f, 60.f);
    const float e = __builtin_amdgcn_exp2f(yc * -1.44269504088896f);
    return y * __builtin_amdgcn_rcpf(1.0f + e);
}
__device__ __forceinline__ unsigned ld_sys(const void* p) { return __hip_atomic_load((GAS unsigned*)p, __ATOMIC_RELAXED, __HIP_MEMORY_SCOPE_SYSTEM); }
__device__ __forceinline__ unsigned ld_agt(const void* p) { return __hip_atomic_load((GAS unsigned*)p, __ATOMIC_RELAXED, __HIP_MEMORY_SCOPE_AGENT); }
__device__ __forceinline__ void st_sys(void* p, unsigned v) { __hip_atomic_store((GAS unsigned*)p, v, __ATOMIC_RELAXED, __HIP_MEMORY_SCOPE_SYSTEM); }
__device__ __forceinline__ void st_agt(void* p, unsigned v) { __hip_atomic_store((GAS unsigned*)p, v, __ATOMIC_RELAXED, __HIP_MEMORY_SCOPE_AGENT); }
__device__ __forceinline__ bool wave_wait(const void* p, unsigned call, void* tmo) {
    const unsigned long long t0 = __builtin_amdgcn_s_memrealtime();
    for (;;) {
        if (__all(ld_sys(p) >= call)) break;
        if (ld_agt(tmo) == call || __builtin_amdgcn_s_memrealtime() - t0 > DEADLINE) { st_agt(tmo, call); return false; }
        __builtin_amdgcn_s_sleep(4);
    }
    __builtin_amdgcn_fence(__ATOMIC_ACQUIRE, "wavefront");
    return true;
}

#define STAMP(i) do {} while (0)
#define STAMP_OUT() do {} while (0)
struct Args { const float* x; const float* w; void* ws; void* out[NR]; void* wsp[NR]; int rank; int pad; };

__device__ __forceinline__ int column_of(int jb, int ni, int g) { return jb == 0 ? 6 - ni : jb == 1 ? (g ? 20 - ni : 7 + ni) : jb == 2 ? 21 + ni : 28 + ni; }
__device__ __forceinline__ int strip_of(int o, int wv, int rank) {
    const int q = o * 4 + wv, pi = q % 7, e = q / 7;
    return q < 112 ? ((rank + 1 + pi) & 7) * 16 + ((e + pi) & 15) : rank * 16 + (q - 112);
}
__device__ __forceinline__ unsigned tile_slot(int o, int mt) {
    return (unsigned)(o < 7 ? o * 32 + mt * 8 : o < 21 ? 224 + (o - 7) * 16 + mt * 4 : 448 + (o - 21) * 32 + mt * 8) * 262144u;
}
__device__ __forceinline__ int win_k(int i, int e0, int kq, int g) {
    const int j = i & 15;
    return kq * 64 + (g ? 15 - j : j) * 4;
}
__device__ __forceinline__ int win_c(int i, int e0, int c0, int c1, int c2, int c3) {
    int c = c0; if (i >= e0) c = c1; if (i >= e0 + 16) c = c2; if (i >= 32) c = c3; return c;
}

constexpr unsigned PSTRIDE = 262144u;
#define PLOAD(dst, soff) _Pragma("unroll") for (int t_ = 0; t_ < 4; ++t_) \
        dst[t_] = __builtin_bit_cast(f4, __builtin_amdgcn_raw_buffer_load_b128(rp, lane * 16 + t_ * 1024, (soff), PAUX))
__device__ __forceinline__ void emit_block(const f4 (&s)[4], const __amdgpu_buffer_rsrc_t ro, unsigned obase, int lane, bool bad) {
    const unsigned voffo = (unsigned)(lane >> 4) * 16384u + (unsigned)(lane & 15) * 16u;
#pragma unroll
    for (int j = 0; j < 4; ++j) {
        f4 o = {silu(s[0][j]), silu(s[1][j]), silu(s[2][j]), silu(s[3][j])};
        if (bad) o = (f4){__builtin_nanf(""), __builtin_nanf(""), __builtin_nanf(""), __builtin_nanf("")};
        __builtin_amdgcn_raw_buffer_store_b128(__builtin_bit_cast(u4, o), ro, voffo, obase + j * 4096u, OAUX);
    }
}
__device__ __forceinline__ void reduce_block8(const __amdgpu_buffer_rsrc_t rp, const __amdgpu_buffer_rsrc_t ro, unsigned pbase, unsigned pstride,
                                              unsigned obase, int lane, bool bad) {
    f4 v[8][4];
#pragma unroll
    for (int p = 0; p < 8; ++p) PLOAD(v[p], pbase + p * pstride);
    SCHED();
#pragma unroll
    for (int p = 1; p < 8; ++p)
#pragma unroll
        for (int t = 0; t < 4; ++t) v[0][t] += v[p][t];
    emit_block(v[0], ro, obase, lane, bad);
    SCHED();
}

__global__ __launch_bounds__(256, 1) void gemm_a2a(Args A) {
    extern __shared__ __attribute__((aligned(16))) unsigned char smem_[];
    LAS unsigned char* smem = (LAS unsigned char*)smem_;
    const int tid = threadIdx.x, lane = tid & 63, wid = __builtin_amdgcn_readfirstlane(tid >> 6);
    const int rank = A.rank;
    unsigned char* ws = (unsigned char*)A.ws;
    STAMP(0);
    const int xcd = blockIdx.x & 7, idx = blockIdx.x >> 3;
    const int mt = idx & 3, ni = idx >> 2, kq = xcd & 3, g = xcd >> 2;
    const __amdgpu_buffer_rsrc_t rx = __builtin_amdgcn_make_buffer_rsrc((void*)(ws + WS_XS), 0, (unsigned)(M * K * 2), 0x00020000);
    const __amdgpu_buffer_rsrc_t rp = __builtin_amdgcn_make_buffer_rsrc((void*)(ws + WS_PART), 0, NOREAD ? 0u : PART_BYTES, 0x00020000);
    const __amdgpu_buffer_rsrc_t rpw = __builtin_amdgcn_make_buffer_rsrc((void*)(ws + WS_PART), 0, NODUMP ? 0u : PART_BYTES, 0x00020000);
    const unsigned lane16 = lane * 16;
    const unsigned tk0 = ld_agt(ws + WS_TICKET);
    {
        const int win = g * 8 + ni;
        const float* src = A.x + (size_t)(mt * 256 + wid * 64 + (lane & 15)) * K + kq * 2048 + win * 128 + (lane >> 4) * 8;
        f4 v[4][4][2];
#pragma unroll
        for (int f = 0; f < 4; ++f)
#pragma unroll
            for (int s = 0; s < 4; ++s) { v[f][s][0] = *(const f4*)(src + (size_t)f * 16 * K + s * 32); v[f][s][1] = *(const f4*)(src + (size_t)f * 16 * K + s * 32 + 4); }
        const unsigned xb = (unsigned)(mt * 256 + kq * 64 + win * 4) * XS + wid * 4096;
#pragma unroll
        for (int f = 0; f < 4; ++f)
#pragma unroll
            for (int s = 0; s < 4; ++s) {
                const u4 o = {cvt2(v[f][s][0][0], v[f][s][0][1]), cvt2(v[f][s][0][2], v[f][s][0][3]), cvt2(v[f][s][1][0], v[f][s][1][1]), cvt2(v[f][s][1][2], v[f][s][1][3])};
                __builtin_amdgcn_raw_buffer_store_b128(o, rx, lane16 + f * 1024, xb + s * XS, 16);
            }
    }
    const unsigned call = (__builtin_amdgcn_readfirstlane(tk0) >> 8) + 1u;
    WAIT_V(0);
    unsigned char* xf = ws + WS_XF + (mt * 4 + kq) * 256;
    if (lane == 0) st_agt(xf + ((g * 8 + ni) * 4 + wid) * 4, call);
    if (blockIdx.x == 0 && wid == 0 && lane < NR) st_sys((unsigned char*)A.wsp[lane & (NR - 1)] + WS_START + rank * 128, call);
    STAMP(1);
    bool bad = false;

    if (ni < 7) {
        const __amdgpu_buffer_rsrc_t rw = __builtin_amdgcn_make_buffer_rsrc((void*)A.w, 0, (unsigned)((size_t)K * N * 4), 0x00020000);
        const unsigned voffw = (unsigned)((lane >> 4) * 8) * (N * 4) + (unsigned)(lane & 15) * 16;
        const unsigned voffx = wid * 4096 + lane16;
        const unsigned ldsx = (unsigned)(size_t)smem + wid * 4096;
        const int n1 = g ? 15 - ni : ni + 1;
        const int e0 = n1, e1 = n1 + 16, e2 = 32, nwin = ni < 4 ? 40 : 32;
        const int c0 = strip_of(column_of(0, ni, g), wid, rank) * 256, c1 = strip_of(column_of(1, ni, g), wid, rank) * 256,
                  c2 = strip_of(column_of(2, ni, g), wid, rank) * 256, c3 = strip_of(column_of(3, ni & 3, g), wid, rank) * 256;
#define wk(i) win_k(i, e0, kq, g)
#define wc(i) win_c(i, e0, c0, c1, c2, c3)
#define WSO(k, c, s) ((unsigned)((k) + (((s) + mt) & 3)) * (32u * N * 4) + (unsigned)(c))
#define XSO(k, s) ((unsigned)(mt * 256 + (k) + (((s) + mt) & 3)) * XS)
#define SETM0(slot) asm volatile("s_mov_b32 m0, %0" :: "s"(ldsx + (slot) * XS) : "memory")
#define GLDS(so, i) asm volatile("buffer_load_dwordx4 %0, %1, %2 offen offset:%c3 " XSC " lds" :: "v"(voffx), "s"(rx), "s"(so), "i"((i) * 1024) : "memory")
#define WLD(rr, h, j, so) rr[h][j] = __builtin_amdgcn_raw_buffer_load_b128(rw, voffw, (so) + (j) * (N * 4), WAUX)
#define CVT(dst, rr, h, t, q) dst[h][t][q] = cvt2(__uint_as_float(rr[h][2 * (q)][t]), __uint_as_float(rr[h][2 * (q) + 1][t]))
#define MFMA(m, t, av, bv) asm volatile("v_mfma_f32_16x16x32_f16 a[%c2:%c3], %0, %1, a[%c2:%c3]" :: "v"(av), "v"(bv), "i"(((m) * 4 + (t)) * 4), "i"(((m) * 4 + (t)) * 4 + 3))
#define MFMA0(m, t, av, bv) asm volatile("v_mfma_f32_16x16x32_f16 a[%c2:%c3], %0, %1, 0" :: "v"(av), "v"(bv), "i"(((m) * 4 + (t)) * 4), "i"(((m) * 4 + (t)) * 4 + 3))
#define LDA(dst, aaddr, off) dst = *(const LAS u4*)(smem + (aaddr) + (off))
        u4 a[4][2], b0[2][4], b1[2][4], r0[2][8], r1[2][8];
        asm volatile("" ::: ACLOB);
        int k1 = wk(1), cc1 = wc(1);
        {
            const int k0 = wk(0);
#pragma unroll
            for (int h = 0; h < 2; ++h)
#pragma unroll
                for (int j = 0; j < 8; ++j) WLD(r0, h, j, WSO(k0, c0, h));
#pragma unroll
            for (int h = 0; h < 2; ++h)
#pragma unroll
                for (int j = 0; j < 8; ++j) WLD(r1, h, j, WSO(k0, c0, 2 + h));
            SCHED();
            if (!wave_wait(xf + lane * 4, call, ws + WS_TMO)) bad = true;
#pragma unroll
            for (int s = 0; s < 4; ++s) { const unsigned so = XSO(k0, s); SETM0(s); asm volatile("s_nop 0"); GLDS(so, 0); GLDS(so, 1); GLDS(so, 2); GLDS(so, 3); }
#pragma unroll
            for (int h = 0; h < 2; ++h)
#pragma unroll
                for (int q = 0; q < 4; ++q)
#pragma unroll
                    for (int t = 0; t < 4; ++t) CVT(b0, r0, h, t, q);
            SCHED();
#pragma unroll
            for (int h = 0; h < 2; ++h)
#pragma unroll
                for (int j = 0; j < 8; ++j) WLD(r0, h, j, WSO(k1, cc1, h));
            __builtin_amdgcn_s_waitcnt(0xC07F);
            WAIT_V(16); BAR();
#pragma unroll
            for (int m = 0; m < 2; ++m) { LDA(a[m][0], lane16, m * 1024); LDA(a[m][1], lane16, XS + m * 1024); }
        }
#define STEP(FIRST, bc, bn, rr, slot, sw0, sw1, sx0, sx1) do { \
        const unsigned ac = lane16 + (slot) * (2 * XS), an = lane16 + (((slot) + 1) & 3) * (2 * XS); \
        const int ls = (((slot) + 2) & 3) * 2; \
        _Pragma("unroll") for (int m = 0; m < 16; ++m) { \
            const int t0 = (m & 1) ? 3 : 0, dt = (m & 1) ? -1 : 1, hh = m >> 3, qq = (m >> 1) & 3; \
            if (m == 14) { WAIT_V(24); BAR(); } \
            if (FIRST) MFMA0(m, t0, a[m & 3][0], bc[0][t0]); else MFMA(m, t0, a[m & 3][0], bc[0][t0]); \
            if (m < 14) LDA(a[(m + 2) & 3][0], ac, (m + 2) * 1024); else LDA(a[(m + 2) & 3][0], an, (m - 14) * 1024); \
            SCHED(); \
            MFMA(m, t0, a[m & 3][1], bc[1][t0]); \
            if (m < 14) LDA(a[(m + 2) & 3][1], ac, XS + (m + 2) * 1024); else LDA(a[(m + 2) & 3][1], an, XS + (m - 14) * 1024); \
            SCHED(); \
            if (FIRST) MFMA0(m, t0 + dt, a[m & 3][1], bc[1][t0 + dt]); else MFMA(m, t0 + dt, a[m & 3][1], bc[1][t0 + dt]); \
            CVT(bn, rr, hh, (m & 1) * 2, qq); \
            SCHED(); \
            MFMA(m, t0 + dt, a[m & 3][0], bc[0][t0 + dt]); \
            CVT(bn, rr, hh, (m & 1) * 2 + 1, qq); \
            SCHED(); \
            if (FIRST) MFMA0(m, t0 + 2 * dt, a[m & 3][0], bc[0][t0 + 2 * dt]); else MFMA(m, t0 + 2 * dt, a[m & 3][0], bc[0][t0 + 2 * dt]); \
            if (m & 1) WLD(rr, hh, 2 * qq, hh ? (sw1) : (sw0)); else if (m == 0) SETM0(ls); else if (m == 6) SETM0(ls + 1); \
            SCHED(); \
            MFMA(m, t0 + 2 * dt, a[m & 3][1], bc[1][t0 + 2 * dt]); \
            SCHED(); \
            if (FIRST) MFMA0(m, t0 + 3 * dt, a[m & 3][1], bc[1][t0 + 3 * dt]); else MFMA(m, t0 + 3 * dt, a[m & 3][1], bc[1][t0 + 3 * dt]); \
            if (m & 1) WLD(rr, hh, 2 * qq + 1, hh ? (sw1) : (sw0)); \
            else if (m == 0) GLDS(sx0, 0); else if (m == 2) GLDS(sx0, 1); else if (m == 4) GLDS(sx0, 3); \
            else if (m == 6) GLDS(sx1, 0); else if (m == 8) GLDS(sx1, 1); else if (m == 10) GLDS(sx1, 2); else if (m == 12) GLDS(sx1, 3); \
            SCHED(); \
            MFMA(m, t0 + 3 * dt, a[m & 3][0], bc[0][t0 + 3 * dt]); \
            if (m == 2) GLDS(sx0, 2); \
            SCHED(); \
        } \
    } while (0)
        STAMP(2);
        int i = 0;
        const __amdgpu_buffer_rsrc_t rf = __builtin_amdgcn_make_buffer_rsrc((void*)(ws + WS_PF), 0, 65536u, 0x00020000);
        unsigned pend = 65536u;
        for (int jb = 0; i < nwin; ++jb) {
            const int iend = jb == 0 ? e0 : jb == 1 ? e1 : jb == 2 ? e2 : nwin;
#define WINDOW(FIRST) do { \
                const int k2 = wk(i + 2), cc2 = wc(i + 2), sl = (i & 1) * 2; \
                const unsigned wa0 = WSO(k1, cc1, 2), wa1 = WSO(k1, cc1, 3), xa0 = XSO(k1, 0), xa1 = XSO(k1, 1); \
                STEP(FIRST, b0, b1, r1, sl, wa0, wa1, xa0, xa1); \
                if (FIRST) { WAIT_V(24);     \
                    asm volatile("buffer_store_dword %0, %1, %2, %3 offen sc1" :: "v"(call), "v"(0u), "s"(rf), "s"(pend) : "memory"); SCHED(); } \
                const unsigned wb0 = WSO(k2, cc2, 0), wb1 = WSO(k2, cc2, 1), xb0 = XSO(k1, 2), xb1 = XSO(k1, 3); \
                STEP(false, b1, b0, r0, sl + 1, wb0, wb1, xb0, xb1); \
                k1 = k2; cc1 = cc2; ++i; \
            } while (0)
            WINDOW(true);
            while (i < iend) WINDOW(false);
            asm volatile("s_nop 15"); SCHED();
            STAMP(3 + jb * 2);
            const unsigned ob = (DMASK ? (blockIdx.x & (DMASK - 1)) * PSTRIDE : tile_slot(column_of(jb, ni, g), mt) + (jb == 1 ? kq : xcd) * PSTRIDE) + wid * 65536u;
#pragma unroll
            for (int q = 0; q < 64; ++q) {
                const unsigned so = ob + (q >> 2) * 4096;
                asm volatile("buffer_store_dwordx4 a[%c3:%c4], %0, %1, %2 offen offset:%c5 " DSC :: "v"(lane16), "s"(rpw), "s"(so), "i"(q * 4), "i"(q * 4 + 3), "i"((q & 3) * 1024) : "memory");
            }
            asm volatile("s_nop 1");
            pend = bad ? 65536u : (unsigned)(((column_of(jb, ni, g) * 4 + mt) * 4 + wid) * 128 + xcd * 4);
            STAMP(4 + jb * 2);
        }
        WAIT_V(0);
        asm volatile("buffer_store_dword %0, %1, %2, %3 offen sc1" :: "v"(call), "v"(0u), "s"(rf), "s"(pend) : "memory");
        STAMP(11);
        if (ni >= 4) {
            if (!wave_wait(ws + WS_START + (lane & 7) * 128, call, ws + WS_TMO)) bad = true;
            for (int it = ((xcd * 3 + ni - 4) * 4 + mt) * 4 + wid; it < 1792; it += 384) {
                const int q = 84 + it % 28, blk = it / 28, cmt = blk >> 4, m = blk & 15, o = q >> 2, wv = q & 3, st = strip_of(o, wv, rank);
                const __amdgpu_buffer_rsrc_t ro = __builtin_amdgcn_make_buffer_rsrc(A.out[(st >> 4) & (NR - 1)], 0, NOSEND ? 0u : 8192u * 1024 * 4, 0x00020000);
                if (!wave_wait(ws + WS_PF + ((o * 4 + cmt) * 4 + wv) * 128 + (lane & 7) * 4, call, ws + WS_TMO)) bad = true;
                reduce_block8(rp, ro, tile_slot(o, cmt) + wv * 65536u + m * 4096u, PSTRIDE,
                              (unsigned)(rank * 1024 + cmt * 256 + m * 16) * 4096u + (st & 15) * 256u, lane, bad);
            }
        } else {
            const __amdgpu_buffer_rsrc_t ro = __builtin_amdgcn_make_buffer_rsrc(A.out[rank & (NR - 1)], 0, 8192u * 1024 * 4, 0x00020000);
            if (!wave_wait(ws + WS_PF + (((28 + ni) * 4 + mt) * 4 + wid) * 128 + (lane & 7) * 4, call, ws + WS_TMO)) bad = true;
            for (int m = xcd * 2; m < xcd * 2 + 2; ++m)
                reduce_block8(rp, ro, tile_slot(28 + ni, mt) + wid * 65536u + m * 4096u, PSTRIDE,
                              (unsigned)(rank * 1024 + mt * 256 + m * 16) * 4096u + (ni * 4 + wid) * 256u, lane, bad);
        }
    } else {
        if (!wave_wait(ws + WS_START + (lane & 7) * 128, call, ws + WS_TMO)) bad = true;
        constexpr int NB = 2 * CP;
        const unsigned fbase = WS_PF + (mt * 4 + wid) * 128, poff = wid * 65536u + (xcd / CP) * (NB * 4096u);
        const unsigned orow = (unsigned)(rank * 1024 + mt * 256 + (xcd / CP) * (NB * 16)) * 4096u;
#define FLAGP(o) (ws + fbase + (o) * 2048 + (((o) >= 14 ? 4 : 0) + (lane & ((o) >= 7 ? 3 : 7))) * 4)
#define PBASE(o) (tile_slot(o, mt) + poff)
        f4 v[8][4], sum[4];
        bool have = false;
        unsigned f1 = 0;
#define COLUMN(NP) do { \
            const unsigned pb = PBASE(o); \
            if (!have) { \
                if (!wave_wait(FLAGP(o), call, ws + WS_TMO)) bad = true; \
                _Pragma("unroll") for (int i = 0; i < 8; ++i) PLOAD(v[i], pb + (i % (NP)) * PSTRIDE + (i / (NP)) * 4096u); \
            } \
            SCHED(); \
            const int st = strip_of(o, wid, rank); \
            const __amdgpu_buffer_rsrc_t ro = __builtin_amdgcn_make_buffer_rsrc(A.out[(st >> 4) & (NR - 1)], 0, NOSEND ? 0u : 8192u * 1024 * 4, 0x00020000); \
            unsigned nb = 0, nhi = 0; \
            _Pragma("unroll") for (int c = 0; c < NB * (NP); ++c) { \
                if (c == NB * (NP) - 16) f1 = ld_agt(FLAGP(o + CP)); \
                if (c == NB * (NP) - 8) { \
                    have = o + CP < 21 && __all(f1 >= call); \
                    __builtin_amdgcn_fence(__ATOMIC_ACQUIRE, "wavefront"); \
                    nb = have ? PBASE(o + CP) : 0x80000000u;           \
                    nhi = o + CP < 7 ? 4 * PSTRIDE : 4096u; \
                } \
                if (c % (NP) != 0) { \
                    _Pragma("unroll") for (int t = 0; t < 4; ++t) { if (c % (NP) == 1) sum[t] = v[(c - 1) % 8][t] + v[c % 8][t]; else sum[t] += v[c % 8][t]; } \
                    _Pragma("unroll") for (int t = 0; t < 4; ++t) asm volatile("" : "+v"(sum[t])); \
                    SCHED(); \
                    _Pragma("unroll") for (int d = (c % (NP) == 1 ? c - 1 : c); d <= c; ++d) { \
                        if (d + 8 < NB * (NP)) { PLOAD(v[d % 8], pb + ((d + 8) % (NP)) * PSTRIDE + ((d + 8) / (NP)) * 4096u); } \
                        else { PLOAD(v[d % 8], nb + (d & 3) * PSTRIDE + ((d >> 2) & 1) * nhi); } \
                    } \
                    SCHED(); \
                } \
                if (c % (NP) == (NP) - 1) emit_block(sum, ro, orow + (c / (NP)) * 65536u + (st & 15) * 256u, lane, bad); \
                SCHED(); \
            } \
            STAMP(40 + o); \
        } while (0)
        int o = xcd % CP;
        for (; o < 7; o += CP) COLUMN(8);
        for (; o < 21; o += CP) COLUMN(4);
    }
    WAIT_V(0);
    STAMP(30);
    STAMP_OUT();
    __syncthreads();
    if (wid == 0) {
        unsigned tk = 0;
        if (lane == 0) tk = __hip_atomic_fetch_add((GAS unsigned*)(ws + WS_TICKET), 1u, __ATOMIC_RELEASE, __HIP_MEMORY_SCOPE_AGENT);
        tk = __builtin_amdgcn_readfirstlane(tk);
        if ((tk & 255u) == 255u) {
            __builtin_amdgcn_fence(__ATOMIC_ACQUIRE, "agent");
            if (lane < NR) st_sys((unsigned char*)A.wsp[lane & (NR - 1)] + WS_DONE + rank * 128, call);
            const bool ok = wave_wait(ws + WS_DONE + (lane & 7) * 128, call, ws + WS_TMO);
            STAMP(31);
            STAMP_OUT();
            if ((!ok || ld_agt(ws + WS_TMO) == call) && lane == 0) ((float*)A.out[rank & (NR - 1)])[(size_t)rank * 1024 * 1024] = __builtin_nanf("");
        }
    }
}

extern "C" void kernel_launch(int rank, int nranks, ncclComm_t comm, hipStream_t stream,
                              void* const* d_in, const int* in_sizes, int n_in,
                              void* d_out, int out_size, const int* in_dtypes, int out_dtype,
                              void* const* d_out_all, void* const* d_ws_all, size_t ws_bytes) {
    if (nranks != NR || ws_bytes < WS_NEED || n_in < 2 || in_sizes[0] != M * K || in_sizes[1] != K * N || out_size != 8192 * 1024) { fprintf(stderr, "unexpected problem shape\n"); return; }
    static bool attr = false;
    if (!attr) { HIP_CHECK(hipFuncSetAttribute((const void*)gemm_a2a, hipFuncAttributeMaxDynamicSharedMemorySize, LDS_BYTES)); attr = true; }
    Args a; a.x = (const float*)d_in[0]; a.w = (const float*)d_in[1]; a.ws = d_ws_all[rank]; a.rank = rank; a.pad = 0;
    for (int p = 0; p < NR; ++p) { a.out[p] = d_out_all[p]; a.wsp[p] = d_ws_all[p]; }
    hipLaunchKernelGGL(gemm_a2a, dim3(256), dim3(256), LDS_BYTES, stream, a);
    HIP_CHECK(hipGetLastError());
}
```

```cpp
#include <hip/hip_runtime.h>
#include <rccl/rccl.h>
#include <cstdio>

#define HIP_CHECK(expr) do { hipError_t _e = (expr); if (_e != hipSuccess) { \
    fprintf(stderr, "HIP error %s at %s:%d\n", hipGetErrorString(_e), __FILE__, __LINE__); \
    return; } } while (0)

typedef _Float16 h2 __attribute__((ext_vector_type(2)));
typedef float f4 __attribute__((ext_vector_type(4)));
typedef float f2 __attribute__((ext_vector_type(2)));
typedef unsigned u4 __attribute__((ext_vector_type(4)));
typedef unsigned u2 __attribute__((ext_vector_type(2)));
#define LAS __attribute__((address_space(3)))
#define GAS __attribute__((address_space(1)))

#define NOSEND 0
#define NODUMP 0
#define NOREAD 0
#define NOST 0
#define DSC "sc1 nt"
#define OAUX 17
#define PAUX 18
#define WMASK 255
#define XMASK 255
#define HW 18
#define PS 24
#define PSYS false
#define MIR 1
#define EPS 0
#define NTAB 0x765321
#define SPN 16
#define LEND 223
#define DA 141
#define DSN 13
constexpr int NR = 8;
constexpr int M = 1024, N = 8192, K = 8192;
constexpr int XS = 16384;
constexpr int LDS_BYTES = 8 * XS;
constexpr unsigned WS_TMO = 128, WS_TICKET = 256, WS_START = 1024, WS_DONE = 2048;
constexpr unsigned WS_XF = 4096;
constexpr unsigned WS_PF = 16384;
constexpr size_t WS_XS = 1u << 20, WS_PART = 17u << 20;
constexpr unsigned PART_BYTES = 184u << 20;
constexpr size_t WS_NEED = WS_PART + PART_BYTES;
constexpr unsigned long long DEADLINE = 40000000ull;

#define ACLOB "a0","a1","a2","a3","a4","a5","a6","a7","a8","a9","a10","a11","a12","a13","a14","a15", \
    "a16","a17","a18","a19","a20","a21","a22","a23","a24","a25","a26","a27","a28","a29","a30","a31", \
    "a32","a33","a34","a35","a36","a37","a38","a39","a40","a41","a42","a43","a44","a45","a46","a47", \
    "a48","a49","a50","a51","a52","a53","a54","a55","a56","a57","a58","a59","a60","a61","a62","a63", \
    "a64","a65","a66","a67","a68","a69","a70","a71","a72","a73","a74","a75","a76","a77","a78","a79", \
    "a80","a81","a82","a83","a84","a85","a86","a87","a88","a89","a90","a91","a92","a93","a94","a95", \
    "a96","a97","a98","a99","a100","a101","a102","a103","a104","a105","a106","a107","a108","a109","a110","a111", \
    "a112","a113","a114","a115","a116","a117","a118","a119","a120","a121","a122","a123","a124","a125","a126","a127", \
    "a128","a129","a130","a131","a132","a133","a134","a135","a136","a137","a138","a139","a140","a141","a142","a143", \
    "a144","a145","a146","a147","a148","a149","a150","a151","a152","a153","a154","a155","a156","a157","a158","a159", \
    "a160","a161","a162","a163","a164","a165","a166","a167","a168","a169","a170","a171","a172","a173","a174","a175", \
    "a176","a177","a178","a179","a180","a181","a182","a183","a184","a185","a186","a187","a188","a189","a190","a191", \
    "a192","a193","a194","a195","a196","a197","a198","a199","a200","a201","a202","a203","a204","a205","a206","a207", \
    "a208","a209","a210","a211","a212","a213","a214","a215","a216","a217","a218","a219","a220","a221","a222","a223", \
    "a224","a225","a226","a227","a228","a229","a230","a231","a232","a233","a234","a235","a236","a237","a238","a239", \
    "a240","a241","a242","a243","a244","a245","a246","a247","a248","a249","a250","a251","a252","a253","a254","a255"
#define BAR() do { asm volatile("" ::: "memory"); __builtin_amdgcn_s_barrier(); asm volatile("" ::: "memory"); } while (0)
#define SCHED() __builtin_amdgcn_sched_barrier(0)
#define WAIT_V(n) asm volatile("s_waitcnt vmcnt(%0)" ::"n"(n) : "memory")

__device__ __forceinline__ unsigned cvt2(float a, float b) { f2 v = {a, b}; h2 h = __builtin_convertvector(v, h2); return __builtin_bit_cast(unsigned, h); }
__device__ __forceinline__ float silu(float y) {
    const float yc = __builtin_amdgcn_fmed3f(y, -60.f, 60.f);
    const float e = __builtin_amdgcn_exp2f(yc * -1.44269504088896f);
    return y * __builtin_amdgcn_rcpf(1.0f + e);
}
__device__ __forceinline__ unsigned ld_sys(const void* p) { return __hip_atomic_load((GAS unsigned*)p, __ATOMIC_RELAXED, __HIP_MEMORY_SCOPE_SYSTEM); }
__device__ __forceinline__ unsigned ld_agt(const void* p) { return __hip_atomic_load((GAS unsigned*)p, __ATOMIC_RELAXED, __HIP_MEMORY_SCOPE_AGENT); }
__device__ __forceinline__ void st_sys(void* p, unsigned v) { __hip_atomic_store((GAS unsigned*)p, v, __ATOMIC_RELAXED, __HIP_MEMORY_SCOPE_SYSTEM); }
__device__ __forceinline__ void st_agt(void* p, unsigned v) { __hip_atomic_store((GAS unsigned*)p, v, __ATOMIC_RELAXED, __HIP_MEMORY_SCOPE_AGENT); }
template <bool SYS, int NAP>
__device__ __forceinline__ bool wave_wait(const void* p, unsigned call, void* tmo) {
    const unsigned long long t0 = __builtin_amdgcn_s_memrealtime();
    for (unsigned it = 1;; ++it) {
        if (__all((SYS ? ld_sys(p) : ld_agt(p)) >= call)) break;
        if ((it & 31u) == 0u && (ld_agt(tmo) == call || __builtin_amdgcn_s_memrealtime() - t0 > DEADLINE)) { st_agt(tmo, call); return false; }
        __builtin_amdgcn_s_sleep(NAP);
    }
    __builtin_amdgcn_fence(__ATOMIC_ACQUIRE, "wavefront");
    return true;
}

#define STAMP(i) do {} while (0)
#define STAMPC(i) do {} while (0)
#define STAMPW(i) do {} while (0)
#define STAMP_OUT() do {} while (0)
struct Args { const float* x; const float* w; void* ws; void* out[NR]; void* wsp[NR]; int rank; int pad; };

__device__ __forceinline__ int column_of(int jb, int ni, int role) {
    return ni >= 6 ? ni - 6 : jb == 0 ? 7 - ni : jb == 1 ? (role ? 19 - ni : 8 + ni) : (jb == 2 && !role) ? 20 + ni : 26 + ni;
}
__device__ __forceinline__ int strip_of(int o, int wv, int rank) {
    const int q = o * 4 + wv, pi = q % 7, e = q / 7;
    return q < 112 ? ((rank + 1 + pi) & 7) * 16 + ((e + pi) & 15) : rank * 16 + (q - 112);
}
__device__ __forceinline__ unsigned part_slot(int o, int mt) {
    return (unsigned)(o < 8 ? o * 32 + mt * 8 : o < 26 ? 128 + o * 16 + mt * 4 : o * 32 - 288 + mt * 8) * 262144u;
}
__device__ __forceinline__ int stage_of(int t, int kq, int g, int rot) {
    const int p = (g ? 63 - t : t) & 63;
    return kq * 64 + ((p & ~3) | ((p + rot) & 3));
}
__device__ __forceinline__ unsigned wrow_of(int stage) { return (unsigned)((stage >> 1) * 64 + (stage & 1) * 16); }
__device__ __forceinline__ int win_c(int i, int e0, int e2, int c0, int c1, int c2, int c3) {
    int c = c0; if (i >= e0) c = c1; if (i >= e0 + 16) c = c2; if (i >= e2) c = c3; return c;
}

constexpr int lpos(int j) { return LEND - (47 - j) * SPN / 4; }
constexpr int lat(int p) { for (int j = 0; j < 48; ++j) if (lpos(j) == p) return j; return -1; }
constexpr int dpos(int q) { return DA + q * DSN / 4; }
constexpr int dat(int p) { for (int q = 0; q < 64; ++q) if (dpos(q) == p) return q; return -1; }
constexpr int dfirst() { int q = 0; while (dpos(q) < 256) ++q; return q; }
constexpr int vwait(int j, int p, int mode) {
    const int a = lpos(j) < p ? lpos(j) : lpos(j) - 256;
    int n = 0;
    for (int k = 0; k < 48; ++k) { if (lpos(k) > a && lpos(k) < p) ++n; if (lpos(k) - 256 > a) ++n; }
    if (!NOST) for (int q = 0; q < 64; ++q) { const int d = dpos(q) - (mode == 1 ? 256 : 0); if (mode && d >= a && d < p) ++n; }
    return n < 63 ? n : 63;
}
constexpr int lafter(int p, int mode) { int n = (mode == 2 && !NOST) ? dfirst() : 0; for (int k = 0; k < 48; ++k) if (lpos(k) > p) ++n; return n < 63 ? n : 63; }
constexpr bool sched_ok() {
    for (int q = 0; q < 64; ++q) if (dpos(q) < 128 + ((q >> 2) + 1) * 8 + 4 || dpos(q) >= 256 + (q >> 2) * 8 - 2) return false;
    for (int j = 0; j < 8; ++j) if (lpos(16 + j) < (j | 1) * 8 + 4 || lpos(24 + j) < 64 + (j | 1) * 8 + 4 || lpos(40 + j) < 128 + (j | 1) * 8 + 4) return false;
    return lpos(0) >= 1 && lpos(32) > 113 && lpos(47) < 256 && dpos(63) - 256 < lpos(15);
}
static_assert(sched_ok(), "schedule");

constexpr unsigned PSTRIDE = 262144u;
#define PLOAD(dst, soff) _Pragma("unroll") for (int t_ = 0; t_ < 4; ++t_) \
        dst[t_] = __builtin_bit_cast(f4, __builtin_amdgcn_raw_buffer_load_b128(rp, lane * 16 + t_ * 1024, (soff), PAUX))
__device__ __forceinline__ void reduce_run8(const __amdgpu_buffer_rsrc_t rp, const __amdgpu_buffer_rsrc_t ro, unsigned pb, unsigned ob, int nb, int lane, bool bad) {
    const unsigned voffo = (unsigned)(lane >> 4) * 16384u + (unsigned)(lane & 15) * 16u;
    f4 v[8][4], sum[4], outv[4];
#pragma unroll
    for (int c = 0; c < 8; ++c) PLOAD(v[c], pb + c * PSTRIDE);
#pragma unroll
    for (int j = 0; j < 4; ++j) outv[j] = (f4){0.f, 0.f, 0.f, 0.f};
    unsigned oprev = 0x80000000u;
#pragma nounroll
    for (int b = 0; b < nb; ++b) {
        const unsigned nx = b + 1 < nb ? pb + (b + 1) * 4096u : 0x80000000u;
#pragma unroll
        for (int c = 1; c < 8; ++c) {
#pragma unroll
            for (int u = 0; u < 4; ++u) { if (c == 1) sum[u] = v[0][u] + v[1][u]; else sum[u] += v[c][u]; }
#pragma unroll
            for (int u = 0; u < 4; ++u) asm volatile("" : "+v"(sum[u]));
            SCHED();
            if (c == 1) PLOAD(v[0], nx);
            PLOAD(v[c], nx + c * PSTRIDE);
            if (c & 1) __builtin_amdgcn_raw_buffer_store_b128(__builtin_bit_cast(u4, outv[c >> 1]), ro, voffo, oprev + (c >> 1) * 4096u, OAUX);
            SCHED();
        }
#pragma unroll
        for (int j = 0; j < 4; ++j) {
            outv[j] = (f4){silu(sum[0][j]), silu(sum[1][j]), silu(sum[2][j]), silu(sum[3][j])};
            if (bad) outv[j] = (f4){__builtin_nanf(""), __builtin_nanf(""), __builtin_nanf(""), __builtin_nanf("")};
        }
        oprev = ob + b * 65536u;
        SCHED();
    }
#pragma unroll
    for (int j = 0; j < 4; ++j) __builtin_amdgcn_raw_buffer_store_b128(__builtin_bit_cast(u4, outv[j]), ro, voffo, oprev + j * 4096u, OAUX);
}

__global__ __launch_bounds__(256, 1) void gemm_a2a(Args A) {
    extern __shared__ __attribute__((aligned(16))) unsigned char smem_[];
    LAS unsigned char* smem = (LAS unsigned char*)smem_;
    const int tid = threadIdx.x, lane = tid & 63, wid = __builtin_amdgcn_readfirstlane(tid >> 6);
    const int rank = A.rank;
    unsigned char* ws = (unsigned char*)A.ws;
    STAMP(0);
    const int xcd = blockIdx.x & 7, idx = blockIdx.x >> 3;
    const int mt = idx & 3, ni = idx >> 2, kq = xcd & 3, g = xcd >> 2;
    const __amdgpu_buffer_rsrc_t rx = __builtin_amdgcn_make_buffer_rsrc((void*)(ws + WS_XS), 0, (unsigned)(M * K * 2), 0x00020000);
    const __amdgpu_buffer_rsrc_t rp = __builtin_amdgcn_make_buffer_rsrc((void*)(ws + WS_PART), 0, NOREAD ? 0u : PART_BYTES, 0x00020000);
    const __amdgpu_buffer_rsrc_t rpw = __builtin_amdgcn_make_buffer_rsrc((void*)(ws + WS_PART), 0, NODUMP ? 0u : PART_BYTES, 0x00020000);
    const unsigned lane16 = lane * 16;
    const unsigned tk0 = ld_agt(ws + WS_TICKET);
    const __amdgpu_buffer_rsrc_t rxi = __builtin_amdgcn_make_buffer_rsrc((void*)A.x, 0, (unsigned)(M * K * 4), 0x00020000);
    const unsigned xvi = (unsigned)(lane >> 3) * (K * 4) + (lane & 7) * 16, xvo = ((lane >> 2) & 1) * XS + (lane & 3) * 256 + (lane >> 3) * 16;
    const unsigned xbi = (unsigned)(mt * 256 + wid * 64) * (K * 4) + kq * 8192, xbo = (unsigned)(mt * 256 + kq * 64) * XS + wid * 4096;
#define XLD(dst, pr) _Pragma("unroll") for (int f_ = 0; f_ < 8; ++f_) _Pragma("unroll") for (int e_ = 0; e_ < 2; ++e_) \
        dst[f_][e_] = __builtin_bit_cast(f4, __builtin_amdgcn_raw_buffer_load_b128(rxi, xvi + e_ * 128, xbi + f_ * (8 * K * 4) + (pr) * 256, 0))
#define XST(src, pr) _Pragma("unroll") for (int f_ = 0; f_ < 8; ++f_) { \
        const u4 o_ = {cvt2(src[f_][0][0], src[f_][0][1]), cvt2(src[f_][0][2], src[f_][0][3]), cvt2(src[f_][1][0], src[f_][1][1]), cvt2(src[f_][1][2], src[f_][1][3])}; \
        __builtin_amdgcn_raw_buffer_store_b128(o_, rx, xvo + (f_ >> 1) * 1024 + (f_ & 1) * 128, xbo + (pr) * (2 * XS), 16); }
    const int pr1 = g ? 31 - ni : ni, pr2 = g ? 23 - ni : 8 + ni;
    {
        f4 v[8][2];
        XLD(v, pr1);
        XST(v, pr1);
    }
    const unsigned call = (__builtin_amdgcn_readfirstlane(tk0) >> 8) + 1u;
    const unsigned xfo = WS_XF + (mt * 4 + kq) * 512;
    unsigned char* xf = ws + xfo;
#define XFW(pr) ((((pr) >> 1) * 4 + wid) * 8 + ((pr) & 1) * 4)
    bool bad = false;

    {
        const __amdgpu_buffer_rsrc_t rw = __builtin_amdgcn_make_buffer_rsrc((void*)A.w, 0, (unsigned)((size_t)K * N * 4), 0x00020000);
        const unsigned voffw = (unsigned)((lane >> 4) * 4) * (N * 4) + (unsigned)(lane & 15) * 16;
        const unsigned voffx = wid * 4096 + lane16;
        const unsigned ldsx = (unsigned)(size_t)smem + wid * 4096;
        const int role = MIR ? g ^ (ni & 1) : g, n = (NTAB >> (4 * ni)) & 15;
        const int nwin = ni >= 6 ? 8 : role ? 40 + EPS : 40 - EPS;
        const int e0 = ni >= 6 ? 8 : role ? 16 - n : n, e1 = e0 + 16, e2 = role ? nwin : e0 + 32;
        const int c0 = strip_of(column_of(0, ni, role), wid, rank) * 256, c1 = strip_of(column_of(1, ni, role), wid, rank) * 256,
                  c2 = strip_of(column_of(2, ni, role), wid, rank) * 256, c3 = strip_of(column_of(3, ni, role), wid, rank) * 256;
#define wk(i) (i)
#define wc(i) win_c(i, e0, e2, c0, c1, c2, c3)
#define KST(k, s) stage_of((k) * 4 + (s), kq, g, mt)
#define WSO(k, c, s) (wrow_of(KST(k, s) & WMASK) * (N * 4) + (unsigned)(c))
#define XSO(k, s) ((unsigned)(mt * 256 + (KST(k, s) & XMASK)) * XS)
#define SETM0(slot) asm volatile("s_mov_b32 m0, %0" :: "s"(ldsx + (slot) * XS) : "memory")
#define GLDS(so, i) asm volatile("buffer_load_dwordx4 %0, %1, %2 offen offset:%c3 sc1 lds" :: "v"(voffx), "s"(rx), "s"(so), "i"((i) * 1024) : "memory")
#define WLD(rr, h, j, so) asm volatile("buffer_load_dwordx4 %0, %1, %2, %3 offen" : "=v"(rr[h][j]) : "v"(voffw), "s"(rw), "s"((so) + (((j) & 3) + 32 * ((j) >> 2)) * (N * 4)) : "memory")
#define CVT(dst, rr, h, t, q) do { unsigned c_; asm volatile("v_cvt_pk_f16_f32 %0, %1, %2" : "=v"(c_) : "v"(rr[h][2 * (q)][t]), "v"(rr[h][2 * (q) + 1][t])); dst[h][t][q] = c_; } while (0)
#define MFMA(m, t, av, bv) asm volatile("v_mfma_f32_16x16x32_f16 a[%c2:%c3], %0, %1, a[%c2:%c3]" :: "v"(av), "v"(bv), "i"(((m) * 4 + (t)) * 4), "i"(((m) * 4 + (t)) * 4 + 3))
#define MFMA0(m, t, av, bv) asm volatile("v_mfma_f32_16x16x32_f16 a[%c2:%c3], %0, %1, 0" :: "v"(av), "v"(bv), "i"(((m) * 4 + (t)) * 4), "i"(((m) * 4 + (t)) * 4 + 3))
#define LDA(dst, aaddr, off) dst = *(const LAS u4*)(smem + (aaddr) + (off))
        u4 a[4][2], b0[2][4], b1[2][4], r0[2][8], r1[2][8];
        u2 xv; const unsigned lane8 = lane * 8;
        asm volatile("" ::: ACLOB);
        int k1 = wk(1), cc1 = wc(1);
        {
            const int k0 = wk(0);
#pragma unroll
            for (int j = 0; j < 8; ++j) WLD(r0, 0, j, WSO(k0, c0, 0));
            WAIT_V(8);
            if (lane == 0) st_agt(xf + XFW(pr1), call);
            if (blockIdx.x == 0 && wid == 0 && lane < NR) st_sys((unsigned char*)A.wsp[lane & (NR - 1)] + WS_START + rank * 128, call);
            STAMP(1);
            f4 x2[8][2];
            XLD(x2, pr2);
#pragma unroll
            for (int j = 0; j < 8; ++j) WLD(r0, 1, j, WSO(k0, c0, 1));
#pragma unroll
            for (int h = 0; h < 2; ++h)
#pragma unroll
                for (int j = 0; j < 8; ++j) WLD(r1, h, j, WSO(k0, c0, 2 + h));
            if (!wave_wait<false, 4>(xf + (g ? 384 : 0) + (lane & 31) * 4, call, ws + WS_TMO)) bad = true;
            __builtin_amdgcn_s_waitcnt(0x0F70);
            WAIT_V(0);
#pragma unroll
            for (int s = 0; s < 4; ++s) { const unsigned so = XSO(k0, s); SETM0(s); asm volatile("s_nop 0"); GLDS(so, 0); GLDS(so, 1); GLDS(so, 2); GLDS(so, 3); }
            XST(x2, pr2);
            SCHED();
#pragma unroll
            for (int h = 0; h < 2; ++h)
#pragma unroll
                for (int q = 0; q < 4; ++q)
#pragma unroll
                    for (int t = 0; t < 4; ++t) CVT(b0, r0, h, t, q);
            SCHED();
#pragma unroll
            for (int j = 0; j < 8; ++j) WLD(r0, 0, j, WSO(k1, cc1, 0));
            __builtin_amdgcn_s_waitcnt(0xC07F);
            WAIT_V(16); BAR();
#pragma unroll
            for (int m = 0; m < 2; ++m) { LDA(a[m][0], lane16, m * 1024); LDA(a[m][1], lane16, XS + m * 1024); }
        }
#define DUMP(q, ob) do { if ((q) < 64 && !NOST) asm volatile("buffer_store_dwordx4 a[%c3:%c4], %0, %1, %2 offen offset:%c5 " DSC :: "v"(lane16), "s"(rpw), "s"((ob) + (((q) & 63) >> 2) * 4096u), \
            "i"(((q) & 63) * 4), "i"(((q) & 63) * 4 + 3), "i"(((q) & 3) * 1024) : "memory"); } while (0)
#define DUMPC(q, ob) do { if ((q) < 64 && !NOST) asm volatile("s_bitcmp1_b32 %2, 31\n\ts_cbranch_scc1 1f\n\tbuffer_store_dwordx4 a[%c3:%c4], %0, %1, %2 offen offset:%c5 " DSC "\n1:" \
            :: "v"(lane16), "s"(rpw), "s"((ob) + (((q) & 63) >> 2) * 4096u), "i"(((q) & 63) * 4), "i"(((q) & 63) * 4 + 3), "i"(((q) & 3) * 1024) : "memory", "scc"); } while (0)
#define PUBLISH() asm volatile("buffer_store_dword %0, %1, %2, %3 offen sc1" :: "v"(call), "v"(0u), "s"(rf), "s"(pend) : "memory")
#define GAP(MODE, B, m, e) do { constexpr int p_ = (B) * 128 + (m) * 8 + (e), j_ = lat(p_), jn_ = lat(p_ + 1), q_ = dat(p_ + ((MODE) == 1 ? 256 : 0)); \
            if (j_ >= 40) WLD(r0, 0, j_ & 7, WSO(k2, cc2, 0)); else if (j_ >= 32) GLDS(XSO(k1, 2 + ((j_ >> 2) & 1)), j_ & 3); \
            else if (j_ >= 24) WLD(r1, 1, j_ & 7, WSO(k1, cc1, 3)); else if (j_ >= 16) WLD(r1, 0, j_ & 7, WSO(k1, cc1, 2)); \
            else if (j_ >= 8) GLDS(XSO(k1, (j_ >> 2) & 1), j_ & 3); else if (j_ >= 0) WLD(r0, 1, j_ & 7, WSO(k1, cc1, 1)); \
            if (jn_ == 8 || jn_ == 12 || jn_ == 32 || jn_ == 36) SETM0(((i + 1) & 1) * 4 + (jn_ >= 32 ? 2 : 0) + ((jn_ >> 2) & 1)); \
            if ((MODE) == 1 && q_ >= 0) DUMPC(q_, ob); else if ((MODE) == 2 && q_ >= 0) DUMP(q_, ob); \
            SCHED(); } while (0)
#define ITER(MODE, B, bc, bn, rc, m) do { \
            constexpr bool FIRST = (MODE) == 1; \
            constexpr int t0 = (m & 1) ? 3 : 0, dt = (m & 1) ? -1 : 1, hh = m >> 3, qq = (m >> 1) & 3; \
            if (m == 14) { WAIT_V(vwait((B) ? 15 : 39, (B) * 128 + 112, MODE)); BAR(); if (B) PUBLISH(); } \
            if (FIRST) MFMA0(m, t0, a[m & 3][0], bc[0][t0]); else MFMA(m, t0, a[m & 3][0], bc[0][t0]); \
            if (m < 14) LDA(a[(m + 2) & 3][0], ac, (m + 2) * 1024); else LDA(a[(m + 2) & 3][0], an, (m - 14) * 1024); \
            GAP(MODE, B, m, 0); \
            MFMA(m, t0, a[m & 3][1], bc[1][t0]); \
            if (m < 14) LDA(a[(m + 2) & 3][1], ac, XS + (m + 2) * 1024); else LDA(a[(m + 2) & 3][1], an, XS + (m - 14) * 1024); \
            GAP(MODE, B, m, 1); \
            if (FIRST) MFMA0(m, t0 + dt, a[m & 3][1], bc[1][t0 + dt]); else MFMA(m, t0 + dt, a[m & 3][1], bc[1][t0 + dt]); \
            if (m == 0) WAIT_V(vwait((B) ? 47 : 23, (B) * 128 + 2, MODE)); \
            if (m == 8) WAIT_V(vwait((B) ? 7 : 31, (B) * 128 + 66, MODE)); \
            CVT(bn, rc, hh, (m & 1) * 2, qq); \
            GAP(MODE, B, m, 2); \
            MFMA(m, t0 + dt, a[m & 3][0], bc[0][t0 + dt]); \
            CVT(bn, rc, hh, (m & 1) * 2 + 1, qq); \
            GAP(MODE, B, m, 3); \
            if (FIRST) MFMA0(m, t0 + 2 * dt, a[m & 3][0], bc[0][t0 + 2 * dt]); else MFMA(m, t0 + 2 * dt, a[m & 3][0], bc[0][t0 + 2 * dt]); \
            GAP(MODE, B, m, 4); \
            if ((B) && m == 0) asm volatile("global_load_dwordx2 %0, %1, %2 sc1" : "=v"(xv) : "v"(lane8), "s"(xf) : "memory"); \
            MFMA(m, t0 + 2 * dt, a[m & 3][1], bc[1][t0 + 2 * dt]); \
            GAP(MODE, B, m, 5); \
            if (FIRST) MFMA0(m, t0 + 3 * dt, a[m & 3][1], bc[1][t0 + 3 * dt]); else MFMA(m, t0 + 3 * dt, a[m & 3][1], bc[1][t0 + 3 * dt]); \
            GAP(MODE, B, m, 6); \
            MFMA(m, t0 + 3 * dt, a[m & 3][0], bc[0][t0 + 3 * dt]); \
            GAP(MODE, B, m, 7); \
        } while (0)
#define STEP(MODE, B, bc, bn, rc) do { \
        const int slot = (i & 1) * 2 + (B); \
        const unsigned ac = lane16 + slot * (2 * XS), an = lane16 + ((slot + 1) & 3) * (2 * XS); \
        ITER(MODE, B, bc, bn, rc, 0); ITER(MODE, B, bc, bn, rc, 1); ITER(MODE, B, bc, bn, rc, 2); ITER(MODE, B, bc, bn, rc, 3); \
        ITER(MODE, B, bc, bn, rc, 4); ITER(MODE, B, bc, bn, rc, 5); ITER(MODE, B, bc, bn, rc, 6); ITER(MODE, B, bc, bn, rc, 7); \
        ITER(MODE, B, bc, bn, rc, 8); ITER(MODE, B, bc, bn, rc, 9); ITER(MODE, B, bc, bn, rc, 10); ITER(MODE, B, bc, bn, rc, 11); \
        ITER(MODE, B, bc, bn, rc, 12); ITER(MODE, B, bc, bn, rc, 13); ITER(MODE, B, bc, bn, rc, 14); ITER(MODE, B, bc, bn, rc, 15); \
    } while (0)
        STAMP(2); STAMPC(62); STAMPW(0);
        int i = 0;
        const __amdgpu_buffer_rsrc_t rf = __builtin_amdgcn_make_buffer_rsrc((void*)ws, 0, WS_PF + 65536u, 0x00020000);
        unsigned pend = xfo + XFW(pr2);
        unsigned ob = 0x80000000u;
        int k2 = wk(2), cc2 = wc(2);
#define STEP_A(MODE) STEP(MODE, 0, b0, b1, r1)
#define STEP_B(MODE) do { STEP(MODE, 1, b1, b0, r0); \
            pend = 0x80000000u; k1 = k2; cc1 = cc2; ++i; k2 = wk(i + 2); cc2 = wc(i + 2); STAMPW(i); \
            asm volatile("s_waitcnt vmcnt(%1)" : "+v"(xv) : "n"(lafter(132, MODE)) : "memory"); \
            if (i >= 3 && i < 15) {     \
                const int pw = g ? 14 - i : i + 1; \
                if (((__ballot(xv[0] >= call && xv[1] >= call) >> (pw * 4)) & 15) != 15) { if (!wave_wait<false, 4>(xf + pw * 32 + (lane & 7) * 4, call, ws + WS_TMO)) bad = true; __builtin_amdgcn_s_waitcnt(0xC07F); } \
                SCHED(); \
            } } while (0)
        for (int jb = 0; i < nwin; ++jb) {
            const int iend = jb == 0 ? e0 : jb == 1 ? e1 : jb == 2 ? e2 : nwin;
            STEP_A(1);
            while (i + 1 < iend) { STEP_B(0); STEP_A(0); }
            const int o = column_of(jb, ni, role), pslot = (o >= 8 && o < 26) ? kq : xcd;
            ob = part_slot(o, mt) + pslot * PSTRIDE + wid * 65536u;
            STEP_B(2);
            pend = bad ? 0x80000000u : WS_PF + (unsigned)(((o * 4 + mt) * 4 + wid) * 128 + pslot * 4);
            STAMP(3 + jb);
        }
        STAMP(7); STAMPC(63);
        asm volatile("s_nop 15"); SCHED();
#pragma unroll
        for (int q = dfirst(); q < 64; ++q) DUMP(q, ob);
        WAIT_V(0);
        PUBLISH();
        STAMP(8);
    }
    if (ni < 6) {
        const int o = 26 + ni, st = strip_of(o, wid, rank), dst = (st >> 4) & (NR - 1);
        const __amdgpu_buffer_rsrc_t ro = __builtin_amdgcn_make_buffer_rsrc(A.out[dst], 0, (NOSEND && ni < 2) ? 0u : 8192u * 1024 * 4, 0x00020000);
        if (!wave_wait<true, 8>(lane < 8 ? ws + WS_PF + ((o * 4 + mt) * 4 + wid) * 128 + lane * 4 : ws + WS_START + dst * 128, call, ws + WS_TMO)) bad = true;
        STAMP(10);
        reduce_run8(rp, ro, part_slot(o, mt) + wid * 65536u + xcd * 8192u, (unsigned)(rank * 1024 + mt * 256 + xcd * 32) * 4096u + (st & 15) * 256u, 2, lane, bad);
    } else {
        const int h = wid * 64 + ((ni - 6) * 8 + xcd) * 4 + mt, pi = h % 7, hj = h / 7;
        if (h < 7 * HW) {
            const int dst = (rank + 1 + pi) & (NR - 1);
            const __amdgpu_buffer_rsrc_t ro = __builtin_amdgcn_make_buffer_rsrc(A.out[dst], 0, NOSEND ? 0u : 8192u * 1024 * 4, 0x00020000);
            if (!wave_wait<true, 8>(ws + WS_START + dst * 128, call, ws + WS_TMO)) bad = true;
            const int t8 = ((pi < 4 ? 320 : 256) - hj + HW - 1) / HW, nu = t8 + ((pi < 6 ? 960 : 896) - hj + HW - 1) / HW;
#define U_B(u) (hj + HW * ((u) < 2 * t8 ? (u) >> 1 : (u) - t8))
#define U_H(u) ((u) < 2 * t8 ? (u) & 1 : 0)
#define U_Q(u) (pi + 7 * (U_B(u) >> 6))
#define U_FLAG(u) (ws + WS_PF + (((U_Q(u) >> 2) * 4 + ((U_B(u) >> 4) & 3)) * 4 + (U_Q(u) & 3)) * 128 + (U_H(u) * 4 + (lane & 3)) * 4)
#define U_PB(u) (part_slot(U_Q(u) >> 2, (U_B(u) >> 4) & 3) + U_H(u) * (4 * PSTRIDE) + (U_Q(u) & 3) * 65536u + (U_B(u) & 15) * 4096u)
            const unsigned voffo = (unsigned)(lane >> 4) * 16384u + (unsigned)(lane & 15) * 16u;
            f4 v[8][4], sum[4], outv[4];
            bool h0 = false, h1 = false;
            unsigned f0 = 0, f1 = 0;
            unsigned oprev = 0x80000000u;
#define UNIT(B, hv, fv) do { \
                if (!hv) { \
                    _Pragma("unroll") for (int j = 0; j < 4; ++j) __builtin_amdgcn_raw_buffer_store_b128(__builtin_bit_cast(u4, outv[j]), ro, voffo, oprev + j * 4096u, OAUX); \
                    oprev = 0x80000000u; \
                    if (!wave_wait<PSYS, PS>(U_FLAG(u), call, ws + WS_TMO)) bad = true; \
                    fv = ld_agt(U_FLAG(u + 2)); \
                    const unsigned pb = U_PB(u); \
                    _Pragma("unroll") for (int i = 0; i < 4; ++i) PLOAD(v[(B) * 4 + i], pb + i * PSTRIDE); \
                    __builtin_amdgcn_s_waitcnt(0x0F70); \
                } \
                SCHED(); \
                if (!(B) || u >= 2 * t8) { _Pragma("unroll") for (int j = 0; j < 4; ++j) sum[j] = (f4){0.f, 0.f, 0.f, 0.f}; } \
                hv = u + 2 >= nu || __all(fv >= call); \
                __builtin_amdgcn_fence(__ATOMIC_ACQUIRE, "wavefront"); \
                const unsigned nb = (hv && u + 2 < nu) ? U_PB(u + 2) : 0x80000000u;     \
                fv = ld_agt(U_FLAG(u + 4)); \
                _Pragma("unroll") for (int c = 0; c < 4; ++c) { \
                    _Pragma("unroll") for (int j = 0; j < 4; ++j) sum[j] += v[(B) * 4 + c][j]; \
                    _Pragma("unroll") for (int j = 0; j < 4; ++j) asm volatile("" : "+v"(sum[j])); \
                    SCHED(); \
                    PLOAD(v[(B) * 4 + c], nb + c * PSTRIDE); \
                    __builtin_amdgcn_raw_buffer_store_b128(__builtin_bit_cast(u4, outv[c]), ro, voffo, oprev + c * 4096u, OAUX); \
                    SCHED(); \
                } \
                oprev = 0x80000000u; \
                if (((B) || u >= 2 * t8) && u < nu) { \
                    _Pragma("unroll") for (int j = 0; j < 4; ++j) { \
                        outv[j] = (f4){silu(sum[0][j]), silu(sum[1][j]), silu(sum[2][j]), silu(sum[3][j])}; \
                        if (bad) outv[j] = (f4){__builtin_nanf(""), __builtin_nanf(""), __builtin_nanf(""), __builtin_nanf("")}; \
                    } \
                    oprev = (unsigned)(rank * 1024 + (U_B(u) & 63) * 16) * 4096u + (((U_B(u) >> 6) + pi) & 15) * 256u; \
                } \
                SCHED(); \
                ++u; \
            } while (0)
#pragma unroll
            for (int j = 0; j < 4; ++j) outv[j] = (f4){0.f, 0.f, 0.f, 0.f};
            for (int u = 0; u < nu;) {
                UNIT(0, h0, f0);
                UNIT(1, h1, f1);
                STAMP(40 + (U_B(u - 1) >> 6));
            }
#pragma unroll
            for (int j = 0; j < 4; ++j) __builtin_amdgcn_raw_buffer_store_b128(__builtin_bit_cast(u4, outv[j]), ro, voffo, oprev + j * 4096u, OAUX);
        }
    }
    WAIT_V(0);
    STAMP(30);
    STAMP_OUT();
    __syncthreads();
    if (wid == 0) {
        unsigned tk = 0;
        if (lane == 0) tk = __hip_atomic_fetch_add((GAS unsigned*)(ws + WS_TICKET), 1u, __ATOMIC_RELAXED, __HIP_MEMORY_SCOPE_AGENT);
        tk = __builtin_amdgcn_readfirstlane(tk);
        if ((tk & 255u) == 255u) {
            if (lane < NR) st_sys((unsigned char*)A.wsp[lane & (NR - 1)] + WS_DONE + rank * 128, call);
            const bool ok = wave_wait<true, 2>(ws + WS_DONE + (lane & 7) * 128, call, ws + WS_TMO);
            STAMP(31);
            STAMP_OUT();
            if ((!ok || ld_agt(ws + WS_TMO) == call) && lane == 0) ((float*)A.out[rank & (NR - 1)])[(size_t)rank * 1024 * 1024] = __builtin_nanf("");
        }
    }
}

extern "C" void kernel_launch(int rank, int nranks, ncclComm_t comm, hipStream_t stream,
                              void* const* d_in, const int* in_sizes, int n_in,
                              void* d_out, int out_size, const int* in_dtypes, int out_dtype,
                              void* const* d_out_all, void* const* d_ws_all, size_t ws_bytes) {
    if (nranks != NR || ws_bytes < WS_NEED || n_in < 2 || in_sizes[0] != M * K || in_sizes[1] != K * N || out_size != 8192 * 1024) { fprintf(stderr, "unexpected problem shape\n"); return; }
    static bool attr = false;
    if (!attr) { HIP_CHECK(hipFuncSetAttribute((const void*)gemm_a2a, hipFuncAttributeMaxDynamicSharedMemorySize, LDS_BYTES)); attr = true; }
    Args a; a.x = (const float*)d_in[0]; a.w = (const float*)d_in[1]; a.ws = d_ws_all[rank]; a.rank = rank; a.pad = 0;
    for (int p = 0; p < NR; ++p) { a.out[p] = d_out_all[p]; a.wsp[p] = d_ws_all[p]; }
    hipLaunchKernelGGL(gemm_a2a, dim3(256), dim3(256), LDS_BYTES, stream, a);
    HIP_CHECK(hipGetLastError());
}
```

```cpp
#include <hip/hip_runtime.h>
#include <rccl/rccl.h>
#include <cstdio>

#define HIP_CHECK(expr) do { hipError_t _e = (expr); if (_e != hipSuccess) { \
    fprintf(stderr, "HIP error %s at %s:%d\n", hipGetErrorString(_e), __FILE__, __LINE__); \
    return; } } while (0)

typedef _Float16 h2 __attribute__((ext_vector_type(2)));
typedef float f4 __attribute__((ext_vector_type(4)));
typedef float f2 __attribute__((ext_vector_type(2)));
typedef unsigned u4 __attribute__((ext_vector_type(4)));
typedef unsigned u2 __attribute__((ext_vector_type(2)));
#define LAS __attribute__((address_space(3)))
#define GAS __attribute__((address_space(1)))

#define NOSEND 0
#define NODUMP 0
#define NOREAD 0
#define NOST 0
#define DSC "sc1"
#define OAUX 17
#define PAUX 18
#define WMASK 255
#define XMASK 255
#define PS 24
#define PSYS false
#define WSC ""
#define XSC "sc1"
#define PACE1 290
#define PACE2 0
#define PN 28
#define MIR 1
#define NOFF 1
#define TWV 20
#define SPN 16
#define LEND 222
#define DA 141
#define DSN 13
constexpr int NR = 8;
constexpr int M = 1024, N = 8192, K = 8192;
constexpr int XS = 16384;
constexpr int LDS_BYTES = 8 * XS;
constexpr unsigned WS_TMO = 128, WS_TICKET = 256, WS_START = 1024, WS_DONE = 2048;
constexpr unsigned WS_XF = 4096;
constexpr unsigned WS_PF = 16384;
constexpr size_t WS_XS = 1u << 20, WS_PART = 17u << 20;
constexpr unsigned PART_BYTES = 156u << 20;
constexpr size_t WS_NEED = WS_PART + PART_BYTES;
constexpr unsigned long long DEADLINE = 40000000ull;

#define ACLOB "a0","a1","a2","a3","a4","a5","a6","a7","a8","a9","a10","a11","a12","a13","a14","a15", \
    "a16","a17","a18","a19","a20","a21","a22","a23","a24","a25","a26","a27","a28","a29","a30","a31", \
    "a32","a33","a34","a35","a36","a37","a38","a39","a40","a41","a42","a43","a44","a45","a46","a47", \
    "a48","a49","a50","a51","a52","a53","a54","a55","a56","a57","a58","a59","a60","a61","a62","a63", \
    "a64","a65","a66","a67","a68","a69","a70","a71","a72","a73","a74","a75","a76","a77","a78","a79", \
    "a80","a81","a82","a83","a84","a85","a86","a87","a88","a89","a90","a91","a92","a93","a94","a95", \
    "a96","a97","a98","a99","a100","a101","a102","a103","a104","a105","a106","a107","a108","a109","a110","a111", \
    "a112","a113","a114","a115","a116","a117","a118","a119","a120","a121","a122","a123","a124","a125","a126","a127", \
    "a128","a129","a130","a131","a132","a133","a134","a135","a136","a137","a138","a139","a140","a141","a142","a143", \
    "a144","a145","a146","a147","a148","a149","a150","a151","a152","a153","a154","a155","a156","a157","a158","a159", \
    "a160","a161","a162","a163","a164","a165","a166","a167","a168","a169","a170","a171","a172","a173","a174","a175", \
    "a176","a177","a178","a179","a180","a181","a182","a183","a184","a185","a186","a187","a188","a189","a190","a191", \
    "a192","a193","a194","a195","a196","a197","a198","a199","a200","a201","a202","a203","a204","a205","a206","a207", \
    "a208","a209","a210","a211","a212","a213","a214","a215","a216","a217","a218","a219","a220","a221","a222","a223", \
    "a224","a225","a226","a227","a228","a229","a230","a231","a232","a233","a234","a235","a236","a237","a238","a239", \
    "a240","a241","a242","a243","a244","a245","a246","a247","a248","a249","a250","a251","a252","a253","a254","a255"
#define BAR() do { asm volatile("" ::: "memory"); __builtin_amdgcn_s_barrier(); asm volatile("" ::: "memory"); } while (0)
#define SCHED() __builtin_amdgcn_sched_barrier(0)
#define WAIT_V(n) asm volatile("s_waitcnt vmcnt(%0)" ::"n"(n) : "memory")

__device__ __forceinline__ unsigned cvt2(float a, float b) { f2 v = {a, b}; h2 h = __builtin_convertvector(v, h2); return __builtin_bit_cast(unsigned, h); }
__device__ __forceinline__ float silu(float y) {
    const float yc = __builtin_amdgcn_fmed3f(y, -60.f, 60.f);
    const float e = __builtin_amdgcn_exp2f(yc * -1.44269504088896f);
    return y * __builtin_amdgcn_rcpf(1.0f + e);
}
__device__ __forceinline__ unsigned ld_sys(const void* p) { return __hip_atomic_load((GAS unsigned*)p, __ATOMIC_RELAXED, __HIP_MEMORY_SCOPE_SYSTEM); }
__device__ __forceinline__ unsigned ld_agt(const void* p) { return __hip_atomic_load((GAS unsigned*)p, __ATOMIC_RELAXED, __HIP_MEMORY_SCOPE_AGENT); }
__device__ __forceinline__ void st_sys(void* p, unsigned v) { __hip_atomic_store((GAS unsigned*)p, v, __ATOMIC_RELAXED, __HIP_MEMORY_SCOPE_SYSTEM); }
__device__ __forceinline__ void st_agt(void* p, unsigned v) { __hip_atomic_store((GAS unsigned*)p, v, __ATOMIC_RELAXED, __HIP_MEMORY_SCOPE_AGENT); }
template <bool SYS, int NAP>
__device__ __forceinline__ bool wave_wait(const void* p, unsigned call, void* tmo) {
    const unsigned long long t0 = __builtin_amdgcn_s_memrealtime();
    for (unsigned it = 1;; ++it) {
        if (__all((SYS ? ld_sys(p) : ld_agt(p)) >= call)) break;
        if ((it & 31u) == 0u && (ld_agt(tmo) == call || __builtin_amdgcn_s_memrealtime() - t0 > DEADLINE)) { st_agt(tmo, call); return false; }
        __builtin_amdgcn_s_sleep(NAP);
    }
    __builtin_amdgcn_fence(__ATOMIC_ACQUIRE, "wavefront");
    return true;
}

#define STAMP(i) do {} while (0)
#define STAMPC(i) do {} while (0)
#define STAMPW(i) do {} while (0)
#define STAMP_OUT() do {} while (0)
struct Args { const float* x; const float* w; void* ws; void* out[NR]; void* wsp[NR]; int rank; int pad; };

__device__ __forceinline__ int column_of(int jb, int ni, int role) {
    return jb == 0 ? 6 - ni : jb == 1 ? (role ? 20 - ni : 7 + ni) : role ? 28 + ((ni + 1) & 3) : 21 + ni;
}
__device__ __forceinline__ int strip_of(int o, int wv, int rank) {
    const int q = o * 4 + wv, pi = q % 7, e = q / 7;
    return q < 112 ? ((rank + 1 + pi) & 7) * 16 + ((e + pi) & 15) : rank * 16 + (q - 112);
}
__device__ __forceinline__ unsigned part_slot(int o, int mt) { return (unsigned)(o < 7 ? o * 32 + mt * 8 : 112 + o * 16 + mt * 4) * 262144u; }
__device__ __forceinline__ int stage_of(int t, int kq, int g, int rot) {
    const int p = (g ? 63 - t : t) & 63;
    return kq * 64 + ((p & ~3) | ((p + rot) & 3));
}
__device__ __forceinline__ unsigned wrow_of(int stage) { return (unsigned)((stage >> 1) * 64 + (stage & 1) * 16); }
__device__ __forceinline__ int win_c(int i, int e0, int e2, int c0, int c1, int c2, int c3) {
    int c = c0; if (i >= e0) c = c1; if (i >= e0 + 16) c = c2; if (i >= e2) c = c3; return c;
}

constexpr int lpos(int j) { return LEND - (47 - j) * SPN / 4; }
constexpr int lat(int p) { for (int j = 0; j < 48; ++j) if (lpos(j) == p) return j; return -1; }
constexpr int dpos(int q) { return DA + q * DSN / 4; }
constexpr int dat(int p) { for (int q = 0; q < 64; ++q) if (dpos(q) == p) return q; return -1; }
constexpr int dfirst() { int q = 0; while (dpos(q) < 256) ++q; return q; }
constexpr int vwait(int j, int p, int mode) {
    const int a = lpos(j) < p ? lpos(j) : lpos(j) - 256;
    int n = 0;
    for (int k = 0; k < 48; ++k) { if (lpos(k) > a && lpos(k) < p) ++n; if (lpos(k) - 256 > a) ++n; }
    if (!NOST) for (int q = 0; q < 64; ++q) { const int d = dpos(q) - (mode == 1 ? 256 : 0); if (mode && d >= a && d < p) ++n; }
    return n < 63 ? n : 63;
}
constexpr int lafter(int p, int mode) { int n = (mode == 2 && !NOST) ? dfirst() : 0; for (int k = 0; k < 48; ++k) if (lpos(k) > p) ++n; return n < 63 ? n : 63; }
constexpr bool sched_ok() {
    for (int q = 0; q < 64; ++q) if (dpos(q) < 128 + ((q >> 2) + 1) * 8 + 4 || dpos(q) >= 256 + (q >> 2) * 8 - 2) return false;
    for (int j = 0; j < 8; ++j) if (lpos(16 + j) < (j | 1) * 8 + 5 || lpos(24 + j) < 64 + (j | 1) * 8 + 5 || lpos(40 + j) < 128 + (j | 1) * 8 + 5) return false;
    return lpos(0) >= 32 && lpos(32) > 113 && lpos(47) < 256 && dpos(63) - 256 < lpos(15);
}
static_assert(sched_ok(), "schedule");

constexpr unsigned PSTRIDE = 262144u;
#define PLOAD(dst, soff) _Pragma("unroll") for (int t_ = 0; t_ < 4; ++t_) \
        dst[t_] = __builtin_bit_cast(f4, __builtin_amdgcn_raw_buffer_load_b128(rp, lane * 16 + t_ * 1024, (soff), PAUX))
template <int NP>
__device__ __forceinline__ void reduce_run(const __amdgpu_buffer_rsrc_t rp, const __amdgpu_buffer_rsrc_t ro, unsigned pb, unsigned ob, int nb, int lane, bool bad) {
    const unsigned voffo = (unsigned)(lane >> 4) * 16384u + (unsigned)(lane & 15) * 16u;
    f4 v[NP][4], sum[4], outv[4];
#pragma unroll
    for (int c = 0; c < NP; ++c) PLOAD(v[c], pb + c * PSTRIDE);
#pragma unroll
    for (int j = 0; j < 4; ++j) outv[j] = (f4){0.f, 0.f, 0.f, 0.f};
    unsigned oprev = 0x80000000u;
#pragma nounroll
    for (int b = 0; b < nb; ++b) {
        const unsigned nx = b + 1 < nb ? pb + (b + 1) * 4096u : 0x80000000u;
#pragma unroll
        for (int c = 1; c < NP; ++c) {
#pragma unroll
            for (int u = 0; u < 4; ++u) { if (c == 1) sum[u] = v[0][u] + v[1][u]; else sum[u] += v[c][u]; }
#pragma unroll
            for (int u = 0; u < 4; ++u) asm volatile("" : "+v"(sum[u]));
            SCHED();
            if (c == 1) PLOAD(v[0], nx);
            PLOAD(v[c], nx + c * PSTRIDE);
            if (NP >= 8 && (c & 1) && c < 8) __builtin_amdgcn_raw_buffer_store_b128(__builtin_bit_cast(u4, outv[c >> 1]), ro, voffo, oprev + (c >> 1) * 4096u, OAUX);
            if (NP == 4) { if (c == 1) __builtin_amdgcn_raw_buffer_store_b128(__builtin_bit_cast(u4, outv[0]), ro, voffo, oprev, OAUX);
                __builtin_amdgcn_raw_buffer_store_b128(__builtin_bit_cast(u4, outv[c & 3]), ro, voffo, oprev + c * 4096u, OAUX); }
            SCHED();
        }
#pragma unroll
        for (int j = 0; j < 4; ++j) {
            outv[j] = (f4){silu(sum[0][j]), silu(sum[1][j]), silu(sum[2][j]), silu(sum[3][j])};
            if (bad) outv[j] = (f4){__builtin_nanf(""), __builtin_nanf(""), __builtin_nanf(""), __builtin_nanf("")};
        }
        oprev = ob + b * 65536u;
        SCHED();
    }
#pragma unroll
    for (int j = 0; j < 4; ++j) __builtin_amdgcn_raw_buffer_store_b128(__builtin_bit_cast(u4, outv[j]), ro, voffo, oprev + j * 4096u, OAUX);
}

__global__ __launch_bounds__(256, 1) void gemm_a2a(Args A) {
    extern __shared__ __attribute__((aligned(16))) unsigned char smem_[];
    LAS unsigned char* smem = (LAS unsigned char*)smem_;
    const int tid = threadIdx.x, lane = tid & 63, wid = __builtin_amdgcn_readfirstlane(tid >> 6);
    const int rank = A.rank;
    unsigned char* ws = (unsigned char*)A.ws;
    STAMP(0);
    const int xcd = blockIdx.x & 7, idx = blockIdx.x >> 3;
    const int mt = idx & 3, ni = idx >> 2, kq = xcd & 3, g = xcd >> 2;
    const __amdgpu_buffer_rsrc_t rx = __builtin_amdgcn_make_buffer_rsrc((void*)(ws + WS_XS), 0, (unsigned)(M * K * 2), 0x00020000);
    const __amdgpu_buffer_rsrc_t rp = __builtin_amdgcn_make_buffer_rsrc((void*)(ws + WS_PART), 0, NOREAD ? 0u : PART_BYTES, 0x00020000);
    const __amdgpu_buffer_rsrc_t rpw = __builtin_amdgcn_make_buffer_rsrc((void*)(ws + WS_PART), 0, NODUMP ? 0u : PART_BYTES, 0x00020000);
    const unsigned lane16 = lane * 16;
    const unsigned tk0 = ld_agt(ws + WS_TICKET);
    const __amdgpu_buffer_rsrc_t rxi = __builtin_amdgcn_make_buffer_rsrc((void*)A.x, 0, (unsigned)(M * K * 4), 0x00020000);
    const unsigned xvi = (unsigned)(lane >> 3) * (K * 4) + (lane & 7) * 16, xvo = ((lane >> 2) & 1) * XS + (lane & 3) * 256 + (lane >> 3) * 16;
    const unsigned xbi = (unsigned)(mt * 256 + wid * 64) * (K * 4) + kq * 8192, xbo = (unsigned)(mt * 256 + kq * 64) * XS + wid * 4096;
#define XLD(dst, pr) _Pragma("unroll") for (int f_ = 0; f_ < 8; ++f_) _Pragma("unroll") for (int e_ = 0; e_ < 2; ++e_) \
        dst[f_][e_] = __builtin_bit_cast(f4, __builtin_amdgcn_raw_buffer_load_b128(rxi, xvi + e_ * 128, xbi + f_ * (8 * K * 4) + (pr) * 256, 0))
#define XST(src, pr) _Pragma("unroll") for (int f_ = 0; f_ < 8; ++f_) { \
        const u4 o_ = {cvt2(src[f_][0][0], src[f_][0][1]), cvt2(src[f_][0][2], src[f_][0][3]), cvt2(src[f_][1][0], src[f_][1][1]), cvt2(src[f_][1][2], src[f_][1][3])}; \
        __builtin_amdgcn_raw_buffer_store_b128(o_, rx, xvo + (f_ >> 1) * 1024 + (f_ & 1) * 128, xbo + (pr) * (2 * XS), 16); }
    const int pr1 = g ? 31 - ni : ni, pr2 = g ? 23 - ni : 8 + ni;
    {
        f4 v[8][2];
        XLD(v, pr1);
        XST(v, pr1);
    }
    const unsigned call = (__builtin_amdgcn_readfirstlane(tk0) >> 8) + 1u;
    const unsigned xfo = WS_XF + (mt * 4 + kq) * 512;
    unsigned char* xf = ws + xfo;
#define XFW(pr) ((((pr) >> 1) * 4 + wid) * 8 + ((pr) & 1) * 4)
    bool bad = false;
    const int role = MIR ? g ^ (ni & 1) : g;

    if (ni < 7) {
        const __amdgpu_buffer_rsrc_t rw = __builtin_amdgcn_make_buffer_rsrc((void*)A.w, 0, (unsigned)((size_t)K * N * 4), 0x00020000);
        unsigned voffw[8];
#pragma unroll
        for (int j = 0; j < 8; ++j) { voffw[j] = (unsigned)((lane >> 4) * 4 + (j & 3) + 32 * (j >> 2)) * (N * 4) + (unsigned)(lane & 15) * 16; asm volatile("" : "+v"(voffw[j])); }
        const unsigned voffx = wid * 4096 + lane16;
        const unsigned ldsx = (unsigned)(size_t)smem + wid * 4096;
        const int e0 = role ? 16 - NOFF - ni : ni + NOFF, e1 = e0 + 16, nwin = (role && ni < 3) ? e1 : e0 + 32, e2 = 64, ksh = 0;
        const int c0 = strip_of(column_of(0, ni, role), wid, rank) * 256, c1 = strip_of(column_of(1, ni, role), wid, rank) * 256,
                  c2 = strip_of(column_of(2, ni, role), wid, rank) * 256, c3 = c2;
#define wk(i) ((i) >= 32 ? (i) + ksh : (i))
#define wc(i) win_c(i, e0, e2, c0, c1, c2, c3)
        unsigned wrel[4], xrel[4];
#pragma unroll
        for (int s = 0; s < 4; ++s) { const int st = stage_of(s, 0, g, mt) & 3; wrel[s] = wrow_of(st) * (N * 4); xrel[s] = (unsigned)(mt * 256 + st) * XS; }
#define WIN(i) (kq * 16 + (((g ? 15 - wk(i) : wk(i)) & 15) & (WMASK >> 2)))
#define WB(i) ((unsigned)WIN(i) * (128u * N * 4) + (unsigned)wc(i))
#define XB(i) ((unsigned)WIN(i) * (4u * XS))
#define PIN(x) asm volatile("" : "+s"(x))
#define SETM0(slot) asm volatile("s_mov_b32 m0, %0" :: "s"(ldsx + (slot) * XS) : "memory")
#define GLDS(so, i) asm volatile("buffer_load_dwordx4 %0, %1, %2 offen offset:%c3 " XSC " lds" :: "v"(voffx), "s"(rx), "s"(so), "i"((i) * 1024) : "memory")
#define WLD(rr, h, j, so) asm volatile("buffer_load_dwordx4 %0, %1, %2, %3 offen " WSC : "=v"(rr[h][j]) : "v"(voffw[j]), "s"(rw), "s"(so) : "memory")
#define CVT(dst, rr, h, t, q) do { unsigned c_; asm volatile("v_cvt_pk_f16_f32 %0, %1, %2" : "=v"(c_) : "v"(rr[h][2 * (q)][t]), "v"(rr[h][2 * (q) + 1][t])); dst[h][t][q] = c_; } while (0)
#define MFMA(m, t, av, bv) asm volatile("v_mfma_f32_16x16x32_f16 a[%c2:%c3], %0, %1, a[%c2:%c3]" :: "v"(av), "v"(bv), "i"(((m) * 4 + (t)) * 4), "i"(((m) * 4 + (t)) * 4 + 3))
#define MFMA0(m, t, av, bv) asm volatile("v_mfma_f32_16x16x32_f16 a[%c2:%c3], %0, %1, 0" :: "v"(av), "v"(bv), "i"(((m) * 4 + (t)) * 4), "i"(((m) * 4 + (t)) * 4 + 3))
#define LDA(dst, aaddr, off) dst = *(const LAS u4*)(smem + (aaddr) + (off))
        u4 a[4][2], b0[2][4], b1[2][4], r0[2][8], r1[2][8];
        u2 xv; const unsigned lane8 = lane * 8;
        asm volatile("" ::: ACLOB);
        unsigned wb1 = WB(1), xb1 = XB(1);
        {
            const unsigned wb0 = WB(0), xb0 = XB(0);
#pragma unroll
            for (int j = 0; j < 8; ++j) WLD(r0, 0, j, wb0 + wrel[0]);
            WAIT_V(8);
            if (lane == 0) st_agt(xf + XFW(pr1), call);
            STAMP(1);
            f4 x2[8][2];
            XLD(x2, pr2);
#pragma unroll
            for (int j = 0; j < 8; ++j) WLD(r0, 1, j, wb0 + wrel[1]);
#pragma unroll
            for (int h = 0; h < 2; ++h)
#pragma unroll
                for (int j = 0; j < 8; ++j) WLD(r1, h, j, wb0 + wrel[2 + h]);
            if (!wave_wait<false, 4>(xf + (g ? 384 : 0) + (lane & 31) * 4, call, ws + WS_TMO)) bad = true;
            __builtin_amdgcn_s_waitcnt(0x0F70);
            WAIT_V(0);
#pragma unroll
            for (int s = 0; s < 4; ++s) { const unsigned so = xb0 + xrel[s]; SETM0(s); asm volatile("s_nop 0"); GLDS(so, 0); GLDS(so, 1); GLDS(so, 2); GLDS(so, 3); }
            XST(x2, pr2);
            SCHED();
#pragma unroll
            for (int h = 0; h < 2; ++h)
#pragma unroll
                for (int q = 0; q < 4; ++q)
#pragma unroll
                    for (int t = 0; t < 4; ++t) CVT(b0, r0, h, t, q);
            SCHED();
#pragma unroll
            for (int j = 0; j < 8; ++j) WLD(r0, 0, j, wb1 + wrel[0]);
            __builtin_amdgcn_s_waitcnt(0xC07F);
            WAIT_V(16); BAR();
#pragma unroll
            for (int m = 0; m < 2; ++m) { LDA(a[m][0], lane16, m * 1024); LDA(a[m][1], lane16, XS + m * 1024); }
        }
#define DUMP(q, ob) do { if ((q) < 64 && !NOST) asm volatile("buffer_store_dwordx4 a[%c3:%c4], %0, %1, %2 offen offset:%c5 " DSC :: "v"(lane16), "s"(rpw), "s"((ob) + (((q) & 63) >> 2) * 4096u), \
            "i"(((q) & 63) * 4), "i"(((q) & 63) * 4 + 3), "i"(((q) & 3) * 1024) : "memory"); } while (0)
#define DUMPC(q, ob) do { if ((q) < 64 && !NOST) asm volatile("s_bitcmp1_b32 %2, 31\n\ts_cbranch_scc1 1f\n\tbuffer_store_dwordx4 a[%c3:%c4], %0, %1, %2 offen offset:%c5 " DSC "\n1:" \
            :: "v"(lane16), "s"(rpw), "s"((ob) + (((q) & 63) >> 2) * 4096u), "i"(((q) & 63) * 4), "i"(((q) & 63) * 4 + 3), "i"(((q) & 3) * 1024) : "memory", "scc"); } while (0)
#define PUBLISH() asm volatile("buffer_store_dword %0, %1, %2, %3 offen sc1" :: "v"(callv), "v"(zerov), "s"(rf), "s"(pend) : "memory")
#define GAP(MODE, B, m, e) do { constexpr int p_ = (B) * 128 + (m) * 8 + (e), j_ = lat(p_), jn_ = lat(p_ + 1), q_ = dat(p_ + ((MODE) == 1 ? 256 : 0)); \
            if (j_ >= 40) WLD(r0, 0, j_ & 7, so); else if (j_ >= 32) GLDS(so, j_ & 3); \
            else if (j_ >= 24) WLD(r1, 1, j_ & 7, so); else if (j_ >= 16) WLD(r1, 0, j_ & 7, so); \
            else if (j_ >= 8) GLDS(so, j_ & 3); else if (j_ >= 0) WLD(r0, 1, j_ & 7, so); \
            if (jn_ == 0 || jn_ == 16 || jn_ == 24) { so = wb1 + wrel[jn_ == 0 ? 1 : jn_ >> 3]; PIN(so); } \
            if (jn_ == 40) { so = wb2 + wrel[0]; PIN(so); } \
            if (jn_ == 8 || jn_ == 12 || jn_ == 32 || jn_ == 36) { so = xb1 + xrel[(jn_ >= 32 ? 2 : 0) + ((jn_ >> 2) & 1)]; PIN(so); \
                asm volatile("s_add_u32 m0, %0, %1" :: "s"(lds2), "i"(((jn_ >= 32 ? 2 : 0) + ((jn_ >> 2) & 1)) * XS) : "memory", "scc"); } \
              \
            if (p_ == 2) { t2 = i + 2; PIN(t2); } \
            if (p_ == 5) { cc = t2 >= e0 ? c1 : c0; PIN(cc); } \
            if (p_ == 6) { cc = t2 >= e1 ? c2 : cc; PIN(cc); } \
            if (p_ == 10) { cc = t2 >= e2 ? c3 : cc; PIN(cc); } \
            if (p_ == 13) { kw = t2 == 32 ? kstep32 : kstep; PIN(kw); } \
            if (p_ == 14) { kp = (kp + kw) & (15 & (WMASK >> 2)); PIN(kp); } \
            if (p_ == 18) { kw = kp | (kq * 16); PIN(kw); } \
            if (p_ == 21) { wb2 = ((unsigned)kw << 22) + (unsigned)cc; PIN(wb2); } \
            if (p_ == 22) { xb2 = (unsigned)kw << 16; PIN(xb2); } \
            if (p_ == 26) { lds2 ^= 4 * XS; PIN(lds2); } \
            if (p_ == 109 || p_ == 237) { an ^= 4 * XS; asm volatile("" : "+v"(an)); } \
            if ((MODE) == 1 && q_ >= 0) DUMPC(q_, ob); else if ((MODE) == 2 && q_ >= 0) DUMP(q_, ob); \
            SCHED(); } while (0)
#define ITER(MODE, B, bc, bn, rc, m, ac, an) do { \
            constexpr bool FIRST = (MODE) == 1; \
            constexpr int t0 = (m & 1) ? 3 : 0, dt = (m & 1) ? -1 : 1, hh = m >> 3, qq = (m >> 1) & 3; \
            if (m == 14) { WAIT_V(vwait((B) ? 15 : 39, (B) * 128 + 112, MODE)); BAR(); } \
            if (FIRST) MFMA0(m, t0, a[m & 3][0], bc[0][t0]); else MFMA(m, t0, a[m & 3][0], bc[0][t0]); \
            if (m < 14) LDA(a[(m + 2) & 3][0], ac, (m + 2) * 1024); else LDA(a[(m + 2) & 3][0], an, (m - 14) * 1024); \
            GAP(MODE, B, m, 0); \
            MFMA(m, t0, a[m & 3][1], bc[1][t0]); \
            if (m == 0) WAIT_V(vwait((B) ? 47 : 23, (B) * 128 + 1, MODE)); \
            if (m == 8) WAIT_V(vwait((B) ? 7 : 31, (B) * 128 + 65, MODE)); \
            CVT(bn, rc, hh, (m & 1) * 2, qq); \
            GAP(MODE, B, m, 1); \
            if (FIRST) MFMA0(m, t0 + dt, a[m & 3][1], bc[1][t0 + dt]); else MFMA(m, t0 + dt, a[m & 3][1], bc[1][t0 + dt]); \
            if ((B) && m == 14) PUBLISH(); \
            GAP(MODE, B, m, 2); \
            MFMA(m, t0 + dt, a[m & 3][0], bc[0][t0 + dt]); \
            if (m < 14) LDA(a[(m + 2) & 3][1], ac, XS + (m + 2) * 1024); else LDA(a[(m + 2) & 3][1], an, XS + (m - 14) * 1024); \
            GAP(MODE, B, m, 3); \
            if (FIRST) MFMA0(m, t0 + 2 * dt, a[m & 3][0], bc[0][t0 + 2 * dt]); else MFMA(m, t0 + 2 * dt, a[m & 3][0], bc[0][t0 + 2 * dt]); \
            CVT(bn, rc, hh, (m & 1) * 2 + 1, qq); \
            GAP(MODE, B, m, 4); \
            if ((B) && m == 0) asm volatile("global_load_dwordx2 %0, %1, %2 sc1" : "=v"(xv) : "v"(lane8), "s"(xf) : "memory"); \
            MFMA(m, t0 + 2 * dt, a[m & 3][1], bc[1][t0 + 2 * dt]); \
            GAP(MODE, B, m, 5); \
            if (FIRST) MFMA0(m, t0 + 3 * dt, a[m & 3][1], bc[1][t0 + 3 * dt]); else MFMA(m, t0 + 3 * dt, a[m & 3][1], bc[1][t0 + 3 * dt]); \
            GAP(MODE, B, m, 6); \
            MFMA(m, t0 + 3 * dt, a[m & 3][0], bc[0][t0 + 3 * dt]); \
            GAP(MODE, B, m, 7); \
        } while (0)
#define STEP(MODE, B, bc, bn, rc, ac, an_) do { unsigned& an = an_; \
        ITER(MODE, B, bc, bn, rc, 0, ac, an); ITER(MODE, B, bc, bn, rc, 1, ac, an); ITER(MODE, B, bc, bn, rc, 2, ac, an); ITER(MODE, B, bc, bn, rc, 3, ac, an); \
        ITER(MODE, B, bc, bn, rc, 4, ac, an); ITER(MODE, B, bc, bn, rc, 5, ac, an); ITER(MODE, B, bc, bn, rc, 6, ac, an); ITER(MODE, B, bc, bn, rc, 7, ac, an); \
        ITER(MODE, B, bc, bn, rc, 8, ac, an); ITER(MODE, B, bc, bn, rc, 9, ac, an); ITER(MODE, B, bc, bn, rc, 10, ac, an); ITER(MODE, B, bc, bn, rc, 11, ac, an); \
        ITER(MODE, B, bc, bn, rc, 12, ac, an); ITER(MODE, B, bc, bn, rc, 13, ac, an); ITER(MODE, B, bc, bn, rc, 14, ac, an); ITER(MODE, B, bc, bn, rc, 15, ac, an); \
    } while (0)
        STAMP(2); STAMPC(62); STAMPW(0);
        int i = 0;
        const __amdgpu_buffer_rsrc_t rf = __builtin_amdgcn_make_buffer_rsrc((void*)ws, 0, WS_PF + 65536u, 0x00020000);
        unsigned pend = xfo + XFW(pr2);
        unsigned ob = 0x80000000u;
        unsigned so = 0, wb2 = 0, xb2 = 0, lds2 = ldsx, aA = lane16, aB = lane16 + 6 * XS, callv = call, zerov = 0;
        unsigned tprev = (unsigned)__builtin_amdgcn_s_memrealtime(); __builtin_amdgcn_s_waitcnt(0xC07F);
        int t2 = 0, cc = 0, kw = 0, kp = WIN(1) & 15;
        const int kstep = g ? -1 : 1, kstep32 = kstep * (1 + ksh);
        asm volatile("" : "+v"(callv), "+v"(zerov));
#define STEP_A(MODE) STEP(MODE, 0, b0, b1, r1, aA, aB)
#define STEP_B(MODE) do { STEP(MODE, 1, b1, b0, r0, aB, aA); \
            pend = 0x80000000u; wb1 = wb2; xb1 = xb2; ++i; \
            if (PACE1) { const unsigned due = tprev + (i <= PN ? PACE1 : PACE2); \
                while ((int)((tprev = (unsigned)__builtin_amdgcn_s_memrealtime()) - due) < 0) __builtin_amdgcn_s_sleep(1); \
                __builtin_amdgcn_s_waitcnt(0xC07F); } \
            STAMPW(i); \
            asm volatile("s_waitcnt vmcnt(%1)" : "+v"(xv) : "n"(lafter(132, MODE)) : "memory"); \
            if (i >= 3 && i < 15) {     \
                const int pw = g ? 14 - i : i + 1; \
                if (((__ballot(xv[0] >= call && xv[1] >= call) >> (pw * 4)) & 15) != 15) { if (!wave_wait<false, 4>(xf + pw * 32 + (lane & 7) * 4, call, ws + WS_TMO)) bad = true; __builtin_amdgcn_s_waitcnt(0xC07F); } \
                SCHED(); \
            } } while (0)
        for (int jb = 0; i < nwin; ++jb) {
            const int iend = jb == 0 ? e0 : jb == 1 ? e1 : nwin;
            STEP_A(1);
            while (i + 1 < iend) { STEP_B(0); STEP_A(0); }
            const int o = column_of(jb, ni, role), pslot = jb == 0 ? role * 4 + kq : kq;
            ob = part_slot(o, mt) + pslot * PSTRIDE + wid * 65536u;
            STEP_B(2);
            pend = bad ? 0x80000000u : WS_PF + (unsigned)(((o * 4 + mt) * 4 + wid) * 128 + pslot * 4);
            STAMP(3 + jb);
        }
        STAMP(7); STAMPC(63);
        asm volatile("s_nop 15"); SCHED();
#pragma unroll
        for (int q = dfirst(); q < 64; ++q) DUMP(q, ob);
        WAIT_V(0);
        PUBLISH();
        STAMP(8);
        if (role && ni >= 3) {
            const int o = column_of(2, ni, 1);
            const __amdgpu_buffer_rsrc_t ro = __builtin_amdgcn_make_buffer_rsrc(A.out[rank & (NR - 1)], 0, 8192u * 1024 * 4, 0x00020000);
            if (!wave_wait<false, 4>(ws + WS_PF + ((o * 4 + mt) * 4 + wid) * 128 + (lane & 3) * 4, call, ws + WS_TMO)) bad = true;
            STAMP(10);
            reduce_run<4>(rp, ro, part_slot(o, mt) + wid * 65536u + kq * 16384u, (unsigned)(rank * 1024 + mt * 256 + kq * 64) * 4096u + ((o - 28) * 4 + wid) * 256u, 4, lane, bad);
        }
    } else {
        WAIT_V(0);
        if (lane == 0) st_agt(xf + XFW(pr1), call);
        if (blockIdx.x == 224 && wid == 0 && lane < NR) st_sys((unsigned char*)A.wsp[lane & (NR - 1)] + WS_START + rank * 128, call);
        f4 v2[8][2];
        XLD(v2, pr2);
        XST(v2, pr2);
        WAIT_V(0);
        if (lane == 0) st_agt(xf + XFW(pr2), call);
        __builtin_amdgcn_s_waitcnt(0x0F70);
    }
    if (ni == 7 || (role && ni < 3)) {
        const bool hlp = ni == 7;
        const int h = hlp ? wid * 32 + xcd * 4 + mt : wid * 48 + (ni * 4 + kq) * 4 + mt, pi = h % 7, hs = hlp ? 18 : TWV;
        const int hj = h / 7 + (hlp ? 0 : 768);
        if (h < 7 * hs) {
        if (!wave_wait<true, 8>(ws + WS_START + ((rank + 1 + pi) & (NR - 1)) * 128, call, ws + WS_TMO)) bad = true;
        const __amdgpu_buffer_rsrc_t ro = __builtin_amdgcn_make_buffer_rsrc(A.out[(rank + 1 + pi) & (NR - 1)], 0, NOSEND ? 0u : 8192u * 1024 * 4, 0x00020000);
#define IT_Q(t) (pi + 7 * ((hj + hs * (t)) >> 6))
#define IT_MT(t) (((hj + hs * (t)) >> 4) & 3)
#define IT_FLAG(t, hf) (ws + WS_PF + (((IT_Q(t) >> 2) * 4 + IT_MT(t)) * 4 + (IT_Q(t) & 3)) * 128 + ((hf) * 4 + (lane & 3)) * 4)
#define IT_HF(t) (IT_Q(t) < 28 ? 1 : 0)
#define IT_PB(t, hf) (part_slot(IT_Q(t) >> 2, IT_MT(t)) + (hf) * (4 * PSTRIDE) + (IT_Q(t) & 3) * 65536u + ((hj + hs * (t)) & 15) * 4096u)
        f4 v[8][4], sum[4], outv[4];
        asm volatile("" ::: ACLOB);
        if (hlp) {
#pragma unroll
        for (int t = 14; t >= 0; t -= 2) {
            const bool va = hj + 18 * t < 256, vb = t > 0;
            if (va && !wave_wait<PSYS, PS>(IT_FLAG(t, 0), call, ws + WS_TMO)) bad = true;
            const unsigned pa = va ? IT_PB(t, 0) : 0x80000000u;
#pragma unroll
            for (int i = 0; i < 4; ++i) PLOAD(v[i], pa + i * PSTRIDE);
            if (vb) {
                if (!wave_wait<PSYS, PS>(IT_FLAG(t - 1, 0), call, ws + WS_TMO)) bad = true;
#pragma unroll
                for (int i = 0; i < 4; ++i) PLOAD(v[4 + i], IT_PB(t - 1, 0) + i * PSTRIDE);
            }
            SCHED();
#pragma unroll
            for (int b = 0; b < (vb ? 2 : 1); ++b)
#pragma unroll
                for (int u = 0; u < 4; ++u) {
                    const f4 x = (v[b * 4][u] + v[b * 4 + 1][u]) + (v[b * 4 + 2][u] + v[b * 4 + 3][u]);
#pragma unroll
                    for (int j = 0; j < 4; ++j) asm volatile("v_accvgpr_write_b32 a[%c1], %0" :: "v"(x[j]), "i"((t - b) * 16 + u * 4 + j));
                }
            SCHED();
        }
        }
        STAMP(39);
        const unsigned voffo = (unsigned)(lane >> 4) * 16384u + (unsigned)(lane & 15) * 16u;
        const int nit = ((hlp ? 768 : 1024) - hj + hs - 1) / hs, tpre = hlp ? 0 : 64;
        bool h0 = false, h1 = false;
        unsigned f0 = 0, f1 = 0;
        unsigned oprev = 0x80000000u;
#define PRE_CASE(T) case T: _Pragma("unroll") for (int u = 0; u < 16; ++u) { float f; asm volatile("v_accvgpr_read_b32 %0, a[%c1]" : "=v"(f) : "i"((T) * 16 + u)); sum[u >> 2][u & 3] = f; } break;
#define ITEM(B, hv, fv) do { \
            if (!hv) { \
                _Pragma("unroll") for (int j = 0; j < 4; ++j) __builtin_amdgcn_raw_buffer_store_b128(__builtin_bit_cast(u4, outv[j]), ro, voffo, oprev + j * 4096u, OAUX); \
                oprev = 0x80000000u; \
                if (!wave_wait<PSYS, PS>(IT_FLAG(t, IT_HF(t)), call, ws + WS_TMO)) bad = true; \
                fv = ld_agt(IT_FLAG(t + 2, IT_HF(t + 2))); \
                const unsigned pb = IT_PB(t, IT_HF(t)); \
                _Pragma("unroll") for (int i = 0; i < 4; ++i) PLOAD(v[(B) * 4 + i], pb + i * PSTRIDE); \
                __builtin_amdgcn_s_waitcnt(0x0F70); \
            } \
            SCHED(); \
            switch (t + tpre) { \
                PRE_CASE(B) PRE_CASE(2 + B) PRE_CASE(4 + B) PRE_CASE(6 + B) PRE_CASE(8 + B) PRE_CASE(10 + B) PRE_CASE(12 + B) PRE_CASE(14 + B) \
                default: _Pragma("unroll") for (int u = 0; u < 4; ++u) sum[u] = (f4){0.f, 0.f, 0.f, 0.f}; \
            } \
            hv = t + 2 >= nit || __all(fv >= call); \
            __builtin_amdgcn_fence(__ATOMIC_ACQUIRE, "wavefront"); \
            const unsigned nb = (hv && t + 2 < nit) ? IT_PB(t + 2, IT_HF(t + 2)) : 0x80000000u;     \
            fv = ld_agt(IT_FLAG(t + 4, IT_HF(t + 4))); \
            _Pragma("unroll") for (int c = 0; c < 4; ++c) { \
                _Pragma("unroll") for (int u = 0; u < 4; ++u) sum[u] += v[(B) * 4 + c][u]; \
                _Pragma("unroll") for (int u = 0; u < 4; ++u) asm volatile("" : "+v"(sum[u])); \
                SCHED(); \
                PLOAD(v[(B) * 4 + c], nb + c * PSTRIDE); \
                __builtin_amdgcn_raw_buffer_store_b128(__builtin_bit_cast(u4, outv[c]), ro, voffo, oprev + c * 4096u, OAUX); \
                SCHED(); \
            } \
            _Pragma("unroll") for (int j = 0; j < 4; ++j) { \
                outv[j] = (f4){silu(sum[0][j]), silu(sum[1][j]), silu(sum[2][j]), silu(sum[3][j])}; \
                if (bad) outv[j] = (f4){__builtin_nanf(""), __builtin_nanf(""), __builtin_nanf(""), __builtin_nanf("")}; \
            } \
            oprev = t < nit ? (unsigned)(rank * 1024 + ((hj + hs * t) & 63) * 16) * 4096u + ((((hj + hs * t) >> 6) + pi) & 15) * 256u : 0x80000000u; \
            SCHED(); \
            ++t; \
        } while (0)
        asm volatile("v_accvgpr_write_b32 a[240], 0\n\tv_accvgpr_write_b32 a[241], 0\n\tv_accvgpr_write_b32 a[242], 0\n\tv_accvgpr_write_b32 a[243], 0\n\t"
                     "v_accvgpr_write_b32 a[244], 0\n\tv_accvgpr_write_b32 a[245], 0\n\tv_accvgpr_write_b32 a[246], 0\n\tv_accvgpr_write_b32 a[247], 0\n\t"
                     "v_accvgpr_write_b32 a[248], 0\n\tv_accvgpr_write_b32 a[249], 0\n\tv_accvgpr_write_b32 a[250], 0\n\tv_accvgpr_write_b32 a[251], 0\n\t"
                     "v_accvgpr_write_b32 a[252], 0\n\tv_accvgpr_write_b32 a[253], 0\n\tv_accvgpr_write_b32 a[254], 0\n\tv_accvgpr_write_b32 a[255], 0");
#pragma unroll
        for (int j = 0; j < 4; ++j) outv[j] = (f4){0.f, 0.f, 0.f, 0.f};
        for (int t = 0; t < nit;) {
            ITEM(0, h0, f0);
            ITEM(1, h1, f1);
            STAMP(40 + ((hj + hs * (t - 1)) >> 6));
        }
#pragma unroll
        for (int j = 0; j < 4; ++j) __builtin_amdgcn_raw_buffer_store_b128(__builtin_bit_cast(u4, outv[j]), ro, voffo, oprev + j * 4096u, OAUX);
        }
        __builtin_amdgcn_s_waitcnt(0x0F70);
        STAMP(ni == 7 ? 60 : 12);
    }
    WAIT_V(0);
    STAMP(30);
    STAMP_OUT();
    __syncthreads();
    if (wid == 0) {
        unsigned tk = 0;
        if (lane == 0) tk = __hip_atomic_fetch_add((GAS unsigned*)(ws + WS_TICKET), 1u, __ATOMIC_RELAXED, __HIP_MEMORY_SCOPE_AGENT);
        tk = __builtin_amdgcn_readfirstlane(tk);
        if ((tk & 255u) == 255u) {
            if (lane < NR) st_sys((unsigned char*)A.wsp[lane & (NR - 1)] + WS_DONE + rank * 128, call);
            const bool ok = wave_wait<true, 2>(ws + WS_DONE + (lane & 7) * 128, call, ws + WS_TMO);
            STAMP(31);
            STAMP_OUT();
            if ((!ok || ld_agt(ws + WS_TMO) == call) && lane == 0) ((float*)A.out[rank & (NR - 1)])[(size_t)rank * 1024 * 1024] = __builtin_nanf("");
        }
    }
}

extern "C" void kernel_launch(int rank, int nranks, ncclComm_t comm, hipStream_t stream,
                              void* const* d_in, const int* in_sizes, int n_in,
                              void* d_out, int out_size, const int* in_dtypes, int out_dtype,
                              void* const* d_out_all, void* const* d_ws_all, size_t ws_bytes) {
    if (nranks != NR || ws_bytes < WS_NEED || n_in < 2 || in_sizes[0] != M * K || in_sizes[1] != K * N || out_size != 8192 * 1024) { fprintf(stderr, "unexpected problem shape\n"); return; }
    static bool attr = false;
    if (!attr) { HIP_CHECK(hipFuncSetAttribute((const void*)gemm_a2a, hipFuncAttributeMaxDynamicSharedMemorySize, LDS_BYTES)); attr = true; }
    Args a; a.x = (const float*)d_in[0]; a.w = (const float*)d_in[1]; a.ws = d_ws_all[rank]; a.rank = rank; a.pad = 0;
    for (int p = 0; p < NR; ++p) { a.out[p] = d_out_all[p]; a.wsp[p] = d_ws_all[p]; }
    hipLaunchKernelGGL(gemm_a2a, dim3(256), dim3(256), LDS_BYTES, stream, a);
    HIP_CHECK(hipGetLastError());
}
```

```cpp
#include <hip/hip_runtime.h>
#include <rccl/rccl.h>
#include <cstdio>

#define HIP_CHECK(expr) do { hipError_t _e = (expr); if (_e != hipSuccess) { \
    fprintf(stderr, "HIP error %s at %s:%d\n", hipGetErrorString(_e), __FILE__, __LINE__); \
    return; } } while (0)

typedef _Float16 h2 __attribute__((ext_vector_type(2)));
typedef float f4 __attribute__((ext_vector_type(4)));
typedef float f2 __attribute__((ext_vector_type(2)));
typedef unsigned u4 __attribute__((ext_vector_type(4)));
typedef unsigned u2 __attribute__((ext_vector_type(2)));
#define LAS __attribute__((address_space(3)))
#define GAS __attribute__((address_space(1)))

#define NOSEND 0
#define NODUMP 0
#define NOREAD 0
#define NOST 0
#define DSC "sc1"
#define OAUX 17
#define PAUX 18
#define WMASK 255
#define XMASK 255
#define PS 24
#define PSYS false
#define WSC ""
#define XSC "sc1"
#define PD 9
#define PF 275
#define PFN 30
#define JIT 1
#define HB 704
#define CUM 0
#define ALT 1
#define PACE1 400
#define PACE2 0
#define PN 16
#define MIR 1
#define NTAB 0x7654321
#define TWV 20
#define SPN 16
#define LEND 222
#define DA 141
#define DSN 13
constexpr int NR = 8;
constexpr int M = 1024, N = 8192, K = 8192;
constexpr int XS = 16384;
constexpr int LDS_BYTES = 8 * XS;
constexpr unsigned WS_TMO = 128, WS_TICKET = 256, WS_START = 1024, WS_DONE = 2048;
constexpr unsigned WS_XF = 4096;
constexpr unsigned WS_PF = 16384;
constexpr size_t WS_XS = 1u << 20, WS_PART = 17u << 20;
constexpr unsigned PART_BYTES = 156u << 20;
constexpr size_t WS_NEED = WS_PART + PART_BYTES;
constexpr unsigned long long DEADLINE = 40000000ull;

#define ACLOB "a0","a1","a2","a3","a4","a5","a6","a7","a8","a9","a10","a11","a12","a13","a14","a15", \
    "a16","a17","a18","a19","a20","a21","a22","a23","a24","a25","a26","a27","a28","a29","a30","a31", \
    "a32","a33","a34","a35","a36","a37","a38","a39","a40","a41","a42","a43","a44","a45","a46","a47", \
    "a48","a49","a50","a51","a52","a53","a54","a55","a56","a57","a58","a59","a60","a61","a62","a63", \
    "a64","a65","a66","a67","a68","a69","a70","a71","a72","a73","a74","a75","a76","a77","a78","a79", \
    "a80","a81","a82","a83","a84","a85","a86","a87","a88","a89","a90","a91","a92","a93","a94","a95", \
    "a96","a97","a98","a99","a100","a101","a102","a103","a104","a105","a106","a107","a108","a109","a110","a111", \
    "a112","a113","a114","a115","a116","a117","a118","a119","a120","a121","a122","a123","a124","a125","a126","a127", \
    "a128","a129","a130","a131","a132","a133","a134","a135","a136","a137","a138","a139","a140","a141","a142","a143", \
    "a144","a145","a146","a147","a148","a149","a150","a151","a152","a153","a154","a155","a156","a157","a158","a159", \
    "a160","a161","a162","a163","a164","a165","a166","a167","a168","a169","a170","a171","a172","a173","a174","a175", \
    "a176","a177","a178","a179","a180","a181","a182","a183","a184","a185","a186","a187","a188","a189","a190","a191", \
    "a192","a193","a194","a195","a196","a197","a198","a199","a200","a201","a202","a203","a204","a205","a206","a207", \
    "a208","a209","a210","a211","a212","a213","a214","a215","a216","a217","a218","a219","a220","a221","a222","a223", \
    "a224","a225","a226","a227","a228","a229","a230","a231","a232","a233","a234","a235","a236","a237","a238","a239", \
    "a240","a241","a242","a243","a244","a245","a246","a247","a248","a249","a250","a251","a252","a253","a254","a255"
#define BAR() do { asm volatile("" ::: "memory"); __builtin_amdgcn_s_barrier(); asm volatile("" ::: "memory"); } while (0)
#define SCHED() __builtin_amdgcn_sched_barrier(0)
#define WAIT_V(n) asm volatile("s_waitcnt vmcnt(%0)" ::"n"(n) : "memory")

__device__ __forceinline__ unsigned cvt2(float a, float b) { f2 v = {a, b}; h2 h = __builtin_convertvector(v, h2); return __builtin_bit_cast(unsigned, h); }
__device__ __forceinline__ float silu(float y) {
    const float yc = __builtin_amdgcn_fmed3f(y, -60.f, 60.f);
    const float e = __builtin_amdgcn_exp2f(yc * -1.44269504088896f);
    return y * __builtin_amdgcn_rcpf(1.0f + e);
}
__device__ __forceinline__ unsigned ld_sys(const void* p) { return __hip_atomic_load((GAS unsigned*)p, __ATOMIC_RELAXED, __HIP_MEMORY_SCOPE_SYSTEM); }
__device__ __forceinline__ unsigned ld_agt(const void* p) { return __hip_atomic_load((GAS unsigned*)p, __ATOMIC_RELAXED, __HIP_MEMORY_SCOPE_AGENT); }
__device__ __forceinline__ void st_sys(void* p, unsigned v) { __hip_atomic_store((GAS unsigned*)p, v, __ATOMIC_RELAXED, __HIP_MEMORY_SCOPE_SYSTEM); }
__device__ __forceinline__ void st_agt(void* p, unsigned v) { __hip_atomic_store((GAS unsigned*)p, v, __ATOMIC_RELAXED, __HIP_MEMORY_SCOPE_AGENT); }
template <bool SYS, int NAP>
__device__ __forceinline__ bool wave_wait(const void* p, unsigned want, void* tmo, unsigned call = 0) {
    if (call == 0) call = want;
    const unsigned long long t0 = __builtin_amdgcn_s_memrealtime();
    for (unsigned it = 1;; ++it) {
        if (__all((SYS ? ld_sys(p) : ld_agt(p)) >= want)) break;
        if ((it & 31u) == 0u && (ld_agt(tmo) == call || __builtin_amdgcn_s_memrealtime() - t0 > DEADLINE)) { st_agt(tmo, call); return false; }
        __builtin_amdgcn_s_sleep(NAP);
    }
    __builtin_amdgcn_fence(__ATOMIC_ACQUIRE, "wavefront");
    return true;
}

#define STAMP(i) do {} while (0)
#define STAMPC(i) do {} while (0)
#define STAMPW(i) do {} while (0)
#define STAMP_OUT() do {} while (0)
struct Args { const float* x; const float* w; void* ws; void* out[NR]; void* wsp[NR]; int rank; int pad;
};

__device__ __forceinline__ int column_of(int jb, int ni, int role) {
    return jb == 0 ? 6 - ni : jb == 1 ? (role ? 20 - ni : 7 + ni) : role ? 28 + ((ni + 1) & 3) : 21 + ni;
}
__device__ __forceinline__ int strip_of(int o, int wv, int rank, int em) {
    const int q = o * 4 + wv, pi = q % 7, e = q / 7;
    return q < 112 ? ((rank + 1 + pi) & 7) * 16 + (((e ^ em) + pi) & 15) : rank * 16 + (q - 112);
}
__device__ __forceinline__ unsigned part_slot(int o, int mt) { return (unsigned)(o < 7 ? o * 32 + mt * 8 : 112 + o * 16 + mt * 4) * 262144u; }
__device__ __forceinline__ int stage_of(int t, int kq, int g, int rot) {
    const int p = (g ? 63 - t : t) & 63;
    return kq * 64 + ((p & ~3) | ((p + rot) & 3));
}
__device__ __forceinline__ unsigned wrow_of(int stage) { return (unsigned)((stage >> 1) * 64 + (stage & 1) * 16); }
__device__ __forceinline__ int win_c(int i, int e0, int e2, int c0, int c1, int c2, int c3) {
    int c = c0; if (i >= e0) c = c1; if (i >= e0 + 16) c = c2; if (i >= e2) c = c3; return c;
}

constexpr int lpos(int j) { return LEND - (47 - j) * SPN / 4; }
constexpr int lat(int p) { for (int j = 0; j < 48; ++j) if (lpos(j) == p) return j; return -1; }
constexpr int dpos(int q) { return DA + q * DSN / 4; }
constexpr int dat(int p) { for (int q = 0; q < 64; ++q) if (dpos(q) == p) return q; return -1; }
constexpr int dfirst() { int q = 0; while (dpos(q) < 256) ++q; return q; }
constexpr int vwait(int j, int p, int mode) {
    const int a = lpos(j) < p ? lpos(j) : lpos(j) - 256;
    int n = 0;
    for (int k = 0; k < 48; ++k) { if (lpos(k) > a && lpos(k) < p) ++n; if (lpos(k) - 256 > a) ++n; }
    if (!NOST) for (int q = 0; q < 64; ++q) { const int d = dpos(q) - (mode == 1 ? 256 : 0); if (mode && d >= a && d < p) ++n; }
    return n < 63 ? n : 63;
}
constexpr int lafter(int p, int mode) { int n = (mode == 2 && !NOST) ? dfirst() : 0; for (int k = 0; k < 48; ++k) if (lpos(k) > p) ++n; return n < 63 ? n : 63; }
constexpr bool sched_ok() {
    for (int q = 0; q < 64; ++q) if (dpos(q) < 128 + ((q >> 2) + 1) * 8 + 4 || dpos(q) >= 256 + (q >> 2) * 8 - 2) return false;
    for (int j = 0; j < 8; ++j) if (lpos(16 + j) < (j | 1) * 8 + 5 || lpos(24 + j) < 64 + (j | 1) * 8 + 5 || lpos(40 + j) < 128 + (j | 1) * 8 + 5) return false;
    return lpos(0) >= 32 && lpos(32) > 113 && lpos(47) < 256 && dpos(63) - 256 < lpos(15);
}
static_assert(sched_ok(), "schedule");

constexpr unsigned PSTRIDE = 262144u;
#define PLOAD(dst, soff) _Pragma("unroll") for (int t_ = 0; t_ < 4; ++t_) \
        dst[t_] = __builtin_bit_cast(f4, __builtin_amdgcn_raw_buffer_load_b128(rp, lane * 16 + t_ * 1024, (soff), PAUX))
template <int NP>
__device__ __forceinline__ void reduce_run(const __amdgpu_buffer_rsrc_t rp, const __amdgpu_buffer_rsrc_t ro, unsigned pb, unsigned ob, int nb, int lane, bool bad) {
    const unsigned voffo = (unsigned)(lane >> 4) * 16384u + (unsigned)(lane & 15) * 16u;
    f4 v[NP][4], sum[4], outv[4];
#pragma unroll
    for (int c = 0; c < NP; ++c) PLOAD(v[c], pb + c * PSTRIDE);
#pragma unroll
    for (int j = 0; j < 4; ++j) outv[j] = (f4){0.f, 0.f, 0.f, 0.f};
    unsigned oprev = 0x80000000u;
#pragma nounroll
    for (int b = 0; b < nb; ++b) {
        const unsigned nx = b + 1 < nb ? pb + (b + 1) * 4096u : 0x80000000u;
#pragma unroll
        for (int c = 1; c < NP; ++c) {
#pragma unroll
            for (int u = 0; u < 4; ++u) { if (c == 1) sum[u] = v[0][u] + v[1][u]; else sum[u] += v[c][u]; }
#pragma unroll
            for (int u = 0; u < 4; ++u) asm volatile("" : "+v"(sum[u]));
            SCHED();
            if (c == 1) PLOAD(v[0], nx);
            PLOAD(v[c], nx + c * PSTRIDE);
            if (NP >= 8 && (c & 1) && c < 8) __builtin_amdgcn_raw_buffer_store_b128(__builtin_bit_cast(u4, outv[c >> 1]), ro, voffo, oprev + (c >> 1) * 4096u, OAUX);
            if (NP == 4) { if (c == 1) __builtin_amdgcn_raw_buffer_store_b128(__builtin_bit_cast(u4, outv[0]), ro, voffo, oprev, OAUX);
                __builtin_amdgcn_raw_buffer_store_b128(__builtin_bit_cast(u4, outv[c & 3]), ro, voffo, oprev + c * 4096u, OAUX); }
            SCHED();
        }
#pragma unroll
        for (int j = 0; j < 4; ++j) {
            outv[j] = (f4){silu(sum[0][j]), silu(sum[1][j]), silu(sum[2][j]), silu(sum[3][j])};
            if (bad) outv[j] = (f4){__builtin_nanf(""), __builtin_nanf(""), __builtin_nanf(""), __builtin_nanf("")};
        }
        oprev = ob + b * 65536u;
        SCHED();
    }
#pragma unroll
    for (int j = 0; j < 4; ++j) __builtin_amdgcn_raw_buffer_store_b128(__builtin_bit_cast(u4, outv[j]), ro, voffo, oprev + j * 4096u, OAUX);
}

__global__ __launch_bounds__(256, 1) void gemm_a2a(Args A) {
    extern __shared__ __attribute__((aligned(16))) unsigned char smem_[];
    LAS unsigned char* smem = (LAS unsigned char*)smem_;
    const int tid = threadIdx.x, lane = tid & 63, wid = __builtin_amdgcn_readfirstlane(tid >> 6);
    const int rank = A.rank;
    unsigned char* ws = (unsigned char*)A.ws;
    STAMP(0);
    const int xcd = blockIdx.x & 7, idx = blockIdx.x >> 3;
    const int mt = idx & 3, ni = idx >> 2, kq = xcd & 3, g = xcd >> 2;
    const __amdgpu_buffer_rsrc_t rx = __builtin_amdgcn_make_buffer_rsrc((void*)(ws + WS_XS), 0, (unsigned)(M * K * 2), 0x00020000);
    const __amdgpu_buffer_rsrc_t rp = __builtin_amdgcn_make_buffer_rsrc((void*)(ws + WS_PART), 0, NOREAD ? 0u : PART_BYTES, 0x00020000);
    const __amdgpu_buffer_rsrc_t rpw = __builtin_amdgcn_make_buffer_rsrc((void*)(ws + WS_PART), 0, NODUMP ? 0u : PART_BYTES, 0x00020000);
    const unsigned lane16 = lane * 16;
    const unsigned tk0 = ld_agt(ws + WS_TICKET);
    const __amdgpu_buffer_rsrc_t rxi = __builtin_amdgcn_make_buffer_rsrc((void*)A.x, 0, (unsigned)(M * K * 4), 0x00020000);
    const unsigned xvi = (unsigned)(lane >> 3) * (K * 4) + (lane & 7) * 16, xvo = ((lane >> 2) & 1) * XS + (lane & 3) * 256 + (lane >> 3) * 16;
    const unsigned xbi = (unsigned)(mt * 256 + wid * 64) * (K * 4) + kq * 8192, xbo = (unsigned)(mt * 256 + kq * 64) * XS + wid * 4096;
#define XLD(dst, pr) _Pragma("unroll") for (int f_ = 0; f_ < 8; ++f_) _Pragma("unroll") for (int e_ = 0; e_ < 2; ++e_) \
        dst[f_][e_] = __builtin_bit_cast(f4, __builtin_amdgcn_raw_buffer_load_b128(rxi, xvi + e_ * 128, xbi + f_ * (8 * K * 4) + (pr) * 256, 0))
#define XST(src, pr) _Pragma("unroll") for (int f_ = 0; f_ < 8; ++f_) { \
        const u4 o_ = {cvt2(src[f_][0][0], src[f_][0][1]), cvt2(src[f_][0][2], src[f_][0][3]), cvt2(src[f_][1][0], src[f_][1][1]), cvt2(src[f_][1][2], src[f_][1][3])}; \
        __builtin_amdgcn_raw_buffer_store_b128(o_, rx, xvo + (f_ >> 1) * 1024 + (f_ & 1) * 128, xbo + (pr) * (2 * XS), 16); }
    const int pr1 = g ? 31 - ni : ni, pr2 = g ? 23 - ni : 8 + ni;
    {
        f4 v[8][2];
        XLD(v, pr1);
        XST(v, pr1);
    }
    const unsigned call = (__builtin_amdgcn_readfirstlane(tk0) >> 8) + 1u;
    const unsigned xfo = WS_XF + (mt * 4 + kq) * 512;
    unsigned char* xf = ws + xfo;
#define XFW(pr) ((((pr) >> 1) * 4 + wid) * 8 + ((pr) & 1) * 4)
    bool bad = false;
    const int role = MIR ? g ^ (ni & 1) : g;
    const int em = (ALT && (call & 1)) ? 15 : 0;

    if (ni < 7) {
        const __amdgpu_buffer_rsrc_t rw = __builtin_amdgcn_make_buffer_rsrc((void*)A.w, 0, (unsigned)((size_t)K * N * 4), 0x00020000);
        unsigned voffw[8];
#pragma unroll
        for (int j = 0; j < 8; ++j) { voffw[j] = (unsigned)((lane >> 4) * 4 + (j & 3) + 32 * (j >> 2)) * (N * 4) + (unsigned)(lane & 15) * 16; asm volatile("" : "+v"(voffw[j])); }
        const unsigned voffx = wid * 4096 + lane16;
        const unsigned ldsx = (unsigned)(size_t)smem + wid * 4096;
        const int n = (NTAB >> (4 * ni)) & 15, e0 = role ? 16 - n : n, e1 = e0 + 16, nwin = (role && ni < 3) ? e1 : e0 + 32, e2 = 64, ksh = 0;
        const int c0 = strip_of(column_of(0, ni, role), wid, rank, em) * 256, c1 = strip_of(column_of(1, ni, role), wid, rank, em) * 256,
                  c2 = strip_of(column_of(2, ni, role), wid, rank, em) * 256, c3 = c2;
#define wk(i) ((i) >= 32 ? (i) + ksh : (i))
#define wc(i) win_c(i, e0, e2, c0, c1, c2, c3)
        unsigned wrel[4], xrel[4];
#pragma unroll
        for (int s = 0; s < 4; ++s) { const int st = stage_of(s, 0, g, mt) & 3; wrel[s] = wrow_of(st) * (N * 4); xrel[s] = (unsigned)(mt * 256 + st) * XS; }
#define WIN(i) (kq * 16 + (((g ? 15 - wk(i) : wk(i)) & 15) & (WMASK >> 2)))
#define WB(i) ((unsigned)WIN(i) * (128u * N * 4) + (unsigned)wc(i))
#define XB(i) ((unsigned)WIN(i) * (4u * XS))
#define PIN(x) asm volatile("" : "+s"(x))
#define SETM0(slot) asm volatile("s_mov_b32 m0, %0" :: "s"(ldsx + (slot) * XS) : "memory")
#define GLDS(so, i) asm volatile("buffer_load_dwordx4 %0, %1, %2 offen offset:%c3 " XSC " lds" :: "v"(voffx), "s"(rx), "s"(so), "i"((i) * 1024) : "memory")
#define WLD(rr, h, j, so) asm volatile("buffer_load_dwordx4 %0, %1, %2, %3 offen " WSC : "=v"(rr[h][j]) : "v"(voffw[j]), "s"(rw), "s"(so) : "memory")
#define CVT(dst, rr, h, t, q) do { unsigned c_; asm volatile("v_cvt_pk_f16_f32 %0, %1, %2" : "=v"(c_) : "v"(rr[h][2 * (q)][t]), "v"(rr[h][2 * (q) + 1][t])); dst[h][t][q] = c_; } while (0)
#define MFMA(m, t, av, bv) asm volatile("v_mfma_f32_16x16x32_f16 a[%c2:%c3], %0, %1, a[%c2:%c3]" :: "v"(av), "v"(bv), "i"(((m) * 4 + (t)) * 4), "i"(((m) * 4 + (t)) * 4 + 3))
#define MFMA0(m, t, av, bv) asm volatile("v_mfma_f32_16x16x32_f16 a[%c2:%c3], %0, %1, 0" :: "v"(av), "v"(bv), "i"(((m) * 4 + (t)) * 4), "i"(((m) * 4 + (t)) * 4 + 3))
#define LDA(dst, aaddr, off) dst = *(const LAS u4*)(smem + (aaddr) + (off))
        u4 a[4][2], b0[2][4], b1[2][4], r0[2][8], r1[2][8];
        u2 xv; const unsigned lane8 = lane * 8;
        asm volatile("" ::: ACLOB);
        unsigned wb1 = WB(1), xb1 = XB(1);
        {
            const unsigned wb0 = WB(0), xb0 = XB(0);
#pragma unroll
            for (int j = 0; j < 8; ++j) WLD(r0, 0, j, wb0 + wrel[0]);
            WAIT_V(8);
            if (lane == 0) st_agt(xf + XFW(pr1), call);
            STAMP(1);
            f4 x2[8][2];
            if (!JIT) XLD(x2, pr2);
#pragma unroll
            for (int j = 0; j < 8; ++j) WLD(r0, 1, j, wb0 + wrel[1]);
#pragma unroll
            for (int h = 0; h < 2; ++h)
#pragma unroll
                for (int j = 0; j < 8; ++j) WLD(r1, h, j, wb0 + wrel[2 + h]);
            if (!wave_wait<false, 4>(xf + (g ? 384 : 0) + (lane & 31) * 4, call, ws + WS_TMO)) bad = true;
            __builtin_amdgcn_s_waitcnt(0x0F70);
            WAIT_V(0);
#pragma unroll
            for (int s = 0; s < 4; ++s) { const unsigned so = xb0 + xrel[s]; SETM0(s); asm volatile("s_nop 0"); GLDS(so, 0); GLDS(so, 1); GLDS(so, 2); GLDS(so, 3); }
            if (!JIT) XST(x2, pr2);
            SCHED();
#pragma unroll
            for (int h = 0; h < 2; ++h)
#pragma unroll
                for (int q = 0; q < 4; ++q)
#pragma unroll
                    for (int t = 0; t < 4; ++t) CVT(b0, r0, h, t, q);
            SCHED();
#pragma unroll
            for (int j = 0; j < 8; ++j) WLD(r0, 0, j, wb1 + wrel[0]);
            __builtin_amdgcn_s_waitcnt(0xC07F);
            WAIT_V(JIT ? 8 : 16); BAR();
#pragma unroll
            for (int m = 0; m < 2; ++m) { LDA(a[m][0], lane16, m * 1024); LDA(a[m][1], lane16, XS + m * 1024); }
        }
#define DUMP(q, ob) do { if ((q) < 64 && !NOST) asm volatile("buffer_store_dwordx4 a[%c3:%c4], %0, %1, %2 offen offset:%c5 " DSC :: "v"(lane16), "s"(rpw), "s"((ob) + (((q) & 63) >> 2) * 4096u), \
            "i"(((q) & 63) * 4), "i"(((q) & 63) * 4 + 3), "i"(((q) & 3) * 1024) : "memory"); } while (0)
#define DUMPC(q, ob) do { if ((q) < 64 && !NOST) asm volatile("s_bitcmp1_b32 %2, 31\n\ts_cbranch_scc1 1f\n\tbuffer_store_dwordx4 a[%c3:%c4], %0, %1, %2 offen offset:%c5 " DSC "\n1:" \
            :: "v"(lane16), "s"(rpw), "s"((ob) + (((q) & 63) >> 2) * 4096u), "i"(((q) & 63) * 4), "i"(((q) & 63) * 4 + 3), "i"(((q) & 3) * 1024) : "memory", "scc"); } while (0)
#define PUBLISH() asm volatile("buffer_store_dword %0, %1, %2, %3 offen sc1" :: "v"(callv), "v"(zerov), "s"(rf), "s"(pend) : "memory")
#define GAP(MODE, B, m, e) do { constexpr int p_ = (B) * 128 + (m) * 8 + (e), j_ = lat(p_), jn_ = lat(p_ + 1), q_ = dat(p_ + ((MODE) == 1 ? 256 : 0)); \
            if (j_ >= 40) WLD(r0, 0, j_ & 7, so); else if (j_ >= 32) GLDS(so, j_ & 3); \
            else if (j_ >= 24) WLD(r1, 1, j_ & 7, so); else if (j_ >= 16) WLD(r1, 0, j_ & 7, so); \
            else if (j_ >= 8) GLDS(so, j_ & 3); else if (j_ >= 0) WLD(r0, 1, j_ & 7, so); \
            if (jn_ == 0 || jn_ == 16 || jn_ == 24) { so = wb1 + wrel[jn_ == 0 ? 1 : jn_ >> 3]; PIN(so); } \
            if (jn_ == 40) { so = wb2 + wrel[0]; PIN(so); } \
            if (jn_ == 8 || jn_ == 12 || jn_ == 32 || jn_ == 36) { so = xb1 + xrel[(jn_ >= 32 ? 2 : 0) + ((jn_ >> 2) & 1)]; PIN(so); \
                asm volatile("s_add_u32 m0, %0, %1" :: "s"(lds2), "i"(((jn_ >= 32 ? 2 : 0) + ((jn_ >> 2) & 1)) * XS) : "memory", "scc"); } \
              \
            if (p_ == 2) { t2 = i + 2; PIN(t2); } \
            if (p_ == 5) { cc = t2 >= e0 ? c1 : c0; PIN(cc); } \
            if (p_ == 6) { cc = t2 >= e1 ? c2 : cc; PIN(cc); } \
            if (p_ == 10) { cc = t2 >= e2 ? c3 : cc; PIN(cc); } \
            if (p_ == 13) { kw = t2 == 32 ? kstep32 : kstep; PIN(kw); } \
            if (p_ == 14) { kp = (kp + kw) & (15 & (WMASK >> 2)); PIN(kp); } \
            if (p_ == 18) { kw = kp | (kq * 16); PIN(kw); } \
            if (p_ == 21) { wb2 = ((unsigned)kw << 22) + (unsigned)cc; PIN(wb2); } \
            if (p_ == 22) { xb2 = (unsigned)kw << 16; PIN(xb2); } \
            if (p_ == 26) { lds2 ^= 4 * XS; PIN(lds2); } \
            if (p_ == 109 || p_ == 237) { an ^= 4 * XS; asm volatile("" : "+v"(an)); } \
            if ((MODE) == 1 && q_ >= 0) DUMPC(q_, ob); else if ((MODE) == 2 && q_ >= 0) DUMP(q_, ob); \
            SCHED(); } while (0)
#define ITER(MODE, B, bc, bn, rc, m, ac, an) do { \
            constexpr bool FIRST = (MODE) == 1; \
            constexpr int t0 = (m & 1) ? 3 : 0, dt = (m & 1) ? -1 : 1, hh = m >> 3, qq = (m >> 1) & 3; \
            if (m == 14) { WAIT_V(vwait((B) ? 15 : 39, (B) * 128 + 112, MODE)); BAR(); } \
            if (FIRST) MFMA0(m, t0, a[m & 3][0], bc[0][t0]); else MFMA(m, t0, a[m & 3][0], bc[0][t0]); \
            if (m < 14) LDA(a[(m + 2) & 3][0], ac, (m + 2) * 1024); else LDA(a[(m + 2) & 3][0], an, (m - 14) * 1024); \
            GAP(MODE, B, m, 0); \
            MFMA(m, t0, a[m & 3][1], bc[1][t0]); \
            if (m == 0) WAIT_V(vwait((B) ? 47 : 23, (B) * 128 + 1, MODE)); \
            if (m == 8) WAIT_V(vwait((B) ? 7 : 31, (B) * 128 + 65, MODE)); \
            CVT(bn, rc, hh, (m & 1) * 2, qq); \
            GAP(MODE, B, m, 1); \
            if (FIRST) MFMA0(m, t0 + dt, a[m & 3][1], bc[1][t0 + dt]); else MFMA(m, t0 + dt, a[m & 3][1], bc[1][t0 + dt]); \
            if ((B) && m == 14) PUBLISH(); \
            GAP(MODE, B, m, 2); \
            MFMA(m, t0 + dt, a[m & 3][0], bc[0][t0 + dt]); \
            if (m < 14) LDA(a[(m + 2) & 3][1], ac, XS + (m + 2) * 1024); else LDA(a[(m + 2) & 3][1], an, XS + (m - 14) * 1024); \
            GAP(MODE, B, m, 3); \
            if (FIRST) MFMA0(m, t0 + 2 * dt, a[m & 3][0], bc[0][t0 + 2 * dt]); else MFMA(m, t0 + 2 * dt, a[m & 3][0], bc[0][t0 + 2 * dt]); \
            CVT(bn, rc, hh, (m & 1) * 2 + 1, qq); \
            GAP(MODE, B, m, 4); \
            if ((B) && m == 0) asm volatile("global_load_dwordx2 %0, %1, %2 sc1" : "=v"(xv) : "v"(lane8), "s"(xf) : "memory"); \
            MFMA(m, t0 + 2 * dt, a[m & 3][1], bc[1][t0 + 2 * dt]); \
            GAP(MODE, B, m, 5); \
            if (FIRST) MFMA0(m, t0 + 3 * dt, a[m & 3][1], bc[1][t0 + 3 * dt]); else MFMA(m, t0 + 3 * dt, a[m & 3][1], bc[1][t0 + 3 * dt]); \
            GAP(MODE, B, m, 6); \
            MFMA(m, t0 + 3 * dt, a[m & 3][0], bc[0][t0 + 3 * dt]); \
            GAP(MODE, B, m, 7); \
        } while (0)
#define STEP(MODE, B, bc, bn, rc, ac, an_) do { unsigned& an = an_; \
        ITER(MODE, B, bc, bn, rc, 0, ac, an); ITER(MODE, B, bc, bn, rc, 1, ac, an); ITER(MODE, B, bc, bn, rc, 2, ac, an); ITER(MODE, B, bc, bn, rc, 3, ac, an); \
        ITER(MODE, B, bc, bn, rc, 4, ac, an); ITER(MODE, B, bc, bn, rc, 5, ac, an); ITER(MODE, B, bc, bn, rc, 6, ac, an); ITER(MODE, B, bc, bn, rc, 7, ac, an); \
        ITER(MODE, B, bc, bn, rc, 8, ac, an); ITER(MODE, B, bc, bn, rc, 9, ac, an); ITER(MODE, B, bc, bn, rc, 10, ac, an); ITER(MODE, B, bc, bn, rc, 11, ac, an); \
        ITER(MODE, B, bc, bn, rc, 12, ac, an); ITER(MODE, B, bc, bn, rc, 13, ac, an); ITER(MODE, B, bc, bn, rc, 14, ac, an); ITER(MODE, B, bc, bn, rc, 15, ac, an); \
    } while (0)
        STAMP(2); STAMPC(62); STAMPW(0);
        int i = 0;
        const __amdgpu_buffer_rsrc_t rf = __builtin_amdgcn_make_buffer_rsrc((void*)ws, 0, WS_PF + 65536u, 0x00020000);
        unsigned pend = JIT ? 0x80000000u : xfo + XFW(pr2);
        unsigned ob = 0x80000000u;
        unsigned so = 0, wb2 = 0, xb2 = 0, lds2 = ldsx, aA = lane16, aB = lane16 + 6 * XS, callv = call, zerov = 0;
        unsigned tprev = (unsigned)__builtin_amdgcn_s_memrealtime(); __builtin_amdgcn_s_waitcnt(0xC07F);
        int t2 = 0, cc = 0, kw = 0, kp = WIN(1) & 15;
        const int kstep = g ? -1 : 1, kstep32 = kstep * (1 + ksh);
        asm volatile("" : "+v"(callv), "+v"(zerov));
#define STEP_A(MODE) STEP(MODE, 0, b0, b1, r1, aA, aB)
#define STEP_B(MODE) do { STEP(MODE, 1, b1, b0, r0, aB, aA); \
            pend = 0x80000000u; wb1 = wb2; xb1 = xb2; ++i; \
            if (PACE1 && i <= PFN) { const int pc_ = PACE1 - i * PD; const unsigned due = tprev + (pc_ > PF ? pc_ : PF); \
                while ((int)((tprev = (unsigned)__builtin_amdgcn_s_memrealtime()) - due) < 0) __builtin_amdgcn_s_sleep(1); \
                if (CUM) tprev = due; \
                __builtin_amdgcn_s_waitcnt(0xC07F); } \
            STAMPW(i); \
            asm volatile("s_waitcnt vmcnt(%1)" : "+v"(xv) : "n"(lafter(132, MODE)) : "memory"); \
            if (i >= 3 && i < 15) {     \
                const int pw = g ? 14 - i : i + 1; \
                if (((__ballot(xv[0] >= call && xv[1] >= call) >> (pw * 4)) & 15) != 15) { if (!wave_wait<false, 4>(xf + pw * 32 + (lane & 7) * 4, call, ws + WS_TMO)) bad = true; __builtin_amdgcn_s_waitcnt(0xC07F); } \
                SCHED(); \
            } } while (0)
        for (int jb = 0; i < nwin; ++jb) {
            const int iend = jb == 0 ? e0 : jb == 1 ? e1 : nwin;
            STEP_A(1);
            while (i + 1 < iend) { STEP_B(0); STEP_A(0); }
            const int o = column_of(jb, ni, role), pslot = jb == 0 ? role * 4 + kq : kq;
            ob = part_slot(o, mt) + pslot * PSTRIDE + wid * 65536u;
            STEP_B(2);
            pend = bad ? 0x80000000u : WS_PF + (unsigned)(((o * 4 + mt) * 4 + wid) * 128 + pslot * 4);
            STAMP(3 + jb);
        }
        STAMP(7); STAMPC(63);
        asm volatile("s_nop 15"); SCHED();
#pragma unroll
        for (int q = dfirst(); q < 64; ++q) DUMP(q, ob);
        WAIT_V(0);
        PUBLISH();
        STAMP(8);
        if (role && ni >= 3) {
            const int o = column_of(2, ni, 1);
            const __amdgpu_buffer_rsrc_t ro = __builtin_amdgcn_make_buffer_rsrc(A.out[rank & (NR - 1)], 0, 8192u * 1024 * 4, 0x00020000);
            if (!wave_wait<false, 4>(ws + WS_PF + ((o * 4 + mt) * 4 + wid) * 128 + (lane & 3) * 4, call, ws + WS_TMO)) bad = true;
            STAMP(10);
            reduce_run<4>(rp, ro, part_slot(o, mt) + wid * 65536u + kq * 16384u, (unsigned)(rank * 1024 + mt * 256 + kq * 64) * 4096u + ((o - 28) * 4 + wid) * 256u, 4, lane, bad);
        }
    } else {
        WAIT_V(0);
        if (lane == 0) st_agt(xf + XFW(pr1), call);
        if (blockIdx.x == 224 && wid == 0 && lane < NR) st_sys((unsigned char*)A.wsp[lane & (NR - 1)] + WS_START + rank * 128, call);
        if (!JIT) {
            f4 v2[8][2];
            XLD(v2, pr2);
            XST(v2, pr2);
            WAIT_V(0);
            if (lane == 0) st_agt(xf + XFW(pr2), call);
        } else {
            f4 u0[8][2], u1[8][2];
#define XPR(p) (g ? 31 - (p) : (p))
            XLD(u0, XPR(8)); XLD(u1, XPR(9));
#pragma nounroll
            for (int p = 8; p < 14; p += 2) {
                XST(u0, XPR(p)); SCHED(); XLD(u0, XPR(p + 2)); SCHED();
                XST(u1, XPR(p + 1)); SCHED(); XLD(u1, XPR(p + 3)); SCHED();
                WAIT_V(16);
                if (lane < 2) st_agt(xf + XFW(XPR(p) & ~1) + lane * 4, call);
            }
            XST(u0, XPR(14)); XST(u1, XPR(15));
            WAIT_V(0);
            if (lane < 2) st_agt(xf + XFW(XPR(14) & ~1) + lane * 4, call);
        }
        STAMP(38);
        __builtin_amdgcn_s_waitcnt(0x0F70);
    }
    if (ni == 7 || (role && ni < 3)) {
        const bool hlp = ni == 7;
        const int h = hlp ? wid * 32 + xcd * 4 + mt : wid * 48 + (ni * 4 + kq) * 4 + mt, pi = h % 7, hs = hlp ? 18 : TWV;
        const int hj = h / 7 + (hlp ? 0 : HB);
        if (h < 7 * hs) {
        if (!wave_wait<true, 8>(ws + WS_START + ((rank + 1 + pi) & (NR - 1)) * 128, call, ws + WS_TMO)) bad = true;
        const __amdgpu_buffer_rsrc_t ro = __builtin_amdgcn_make_buffer_rsrc(A.out[(rank + 1 + pi) & (NR - 1)], 0, NOSEND ? 0u : 8192u * 1024 * 4, 0x00020000);
#define IT_Q(t) (pi + 7 * ((hj + hs * (t)) >> 6))
#define IT_MT(t) (((hj + hs * (t)) >> 4) & 3)
#define IT_FLAG(t, hf) (ws + WS_PF + (((IT_Q(t) >> 2) * 4 + IT_MT(t)) * 4 + (IT_Q(t) & 3)) * 128 + ((hf) * 4 + (lane & 3)) * 4)
#define IT_HF(t) (IT_Q(t) < 28 ? 1 : 0)
#define IT_PB(t, hf) (part_slot(IT_Q(t) >> 2, IT_MT(t)) + (hf) * (4 * PSTRIDE) + (IT_Q(t) & 3) * 65536u + ((hj + hs * (t)) & 15) * 4096u)
        f4 v[8][4], sum[4], outv[4];
        asm volatile("" ::: ACLOB);
        if (hlp) {
#pragma unroll
        for (int t = 14; t >= 0; t -= 2) {
            const bool va = hj + 18 * t < 256, vb = t > 0;
            if (va && !wave_wait<PSYS, PS>(IT_FLAG(t, 0), call, ws + WS_TMO)) bad = true;
            const unsigned pa = va ? IT_PB(t, 0) : 0x80000000u;
#pragma unroll
            for (int i = 0; i < 4; ++i) PLOAD(v[i], pa + i * PSTRIDE);
            if (vb) {
                if (!wave_wait<PSYS, PS>(IT_FLAG(t - 1, 0), call, ws + WS_TMO)) bad = true;
#pragma unroll
                for (int i = 0; i < 4; ++i) PLOAD(v[4 + i], IT_PB(t - 1, 0) + i * PSTRIDE);
            }
            SCHED();
#pragma unroll
            for (int b = 0; b < (vb ? 2 : 1); ++b)
#pragma unroll
                for (int u = 0; u < 4; ++u) {
                    const f4 x = (v[b * 4][u] + v[b * 4 + 1][u]) + (v[b * 4 + 2][u] + v[b * 4 + 3][u]);
#pragma unroll
                    for (int j = 0; j < 4; ++j) asm volatile("v_accvgpr_write_b32 a[%c1], %0" :: "v"(x[j]), "i"((t - b) * 16 + u * 4 + j));
                }
            SCHED();
        }
        }
        STAMP(39);
        const unsigned voffo = (unsigned)(lane >> 4) * 16384u + (unsigned)(lane & 15) * 16u;
        const int nit = ((hlp ? HB : 1024) - hj + hs - 1) / hs, tpre = hlp ? 0 : 64;
        bool h0 = false, h1 = false;
        unsigned f0 = 0, f1 = 0;
        unsigned oprev = 0x80000000u;
#define PRE_CASE(T) case T: _Pragma("unroll") for (int u = 0; u < 16; ++u) { float f; asm volatile("v_accvgpr_read_b32 %0, a[%c1]" : "=v"(f) : "i"((T) * 16 + u)); sum[u >> 2][u & 3] = f; } break;
#define ITEM(B, hv, fv) do { \
            if (!hv) { \
                _Pragma("unroll") for (int j = 0; j < 4; ++j) __builtin_amdgcn_raw_buffer_store_b128(__builtin_bit_cast(u4, outv[j]), ro, voffo, oprev + j * 4096u, OAUX); \
                oprev = 0x80000000u; \
                if (!wave_wait<PSYS, PS>(IT_FLAG(t, IT_HF(t)), call, ws + WS_TMO)) bad = true; \
                fv = ld_agt(IT_FLAG(t + 2, IT_HF(t + 2))); \
                const unsigned pb = IT_PB(t, IT_HF(t)); \
                _Pragma("unroll") for (int i = 0; i < 4; ++i) PLOAD(v[(B) * 4 + i], pb + i * PSTRIDE); \
                __builtin_amdgcn_s_waitcnt(0x0F70); \
            } \
            SCHED(); \
            switch (t + tpre) { \
                PRE_CASE(B) PRE_CASE(2 + B) PRE_CASE(4 + B) PRE_CASE(6 + B) PRE_CASE(8 + B) PRE_CASE(10 + B) PRE_CASE(12 + B) PRE_CASE(14 + B) \
                default: _Pragma("unroll") for (int u = 0; u < 4; ++u) sum[u] = (f4){0.f, 0.f, 0.f, 0.f}; \
            } \
            hv = t + 2 >= nit || __all(fv >= call); \
            __builtin_amdgcn_fence(__ATOMIC_ACQUIRE, "wavefront"); \
            const unsigned nb = (hv && t + 2 < nit) ? IT_PB(t + 2, IT_HF(t + 2)) : 0x80000000u;     \
            fv = ld_agt(IT_FLAG(t + 4, IT_HF(t + 4))); \
            _Pragma("unroll") for (int c = 0; c < 4; ++c) { \
                _Pragma("unroll") for (int u = 0; u < 4; ++u) sum[u] += v[(B) * 4 + c][u]; \
                _Pragma("unroll") for (int u = 0; u < 4; ++u) asm volatile("" : "+v"(sum[u])); \
                SCHED(); \
                PLOAD(v[(B) * 4 + c], nb + c * PSTRIDE); \
                __builtin_amdgcn_raw_buffer_store_b128(__builtin_bit_cast(u4, outv[c]), ro, voffo, oprev + c * 4096u, OAUX); \
                SCHED(); \
            } \
            _Pragma("unroll") for (int j = 0; j < 4; ++j) { \
                outv[j] = (f4){silu(sum[0][j]), silu(sum[1][j]), silu(sum[2][j]), silu(sum[3][j])}; \
                if (bad) outv[j] = (f4){__builtin_nanf(""), __builtin_nanf(""), __builtin_nanf(""), __builtin_nanf("")}; \
            } \
            oprev = t < nit ? (unsigned)(rank * 1024 + ((hj + hs * t) & 63) * 16) * 4096u + (((((hj + hs * t) >> 6) ^ em) + pi) & 15) * 256u : 0x80000000u; \
            SCHED(); \
            ++t; \
        } while (0)
        asm volatile("v_accvgpr_write_b32 a[240], 0\n\tv_accvgpr_write_b32 a[241], 0\n\tv_accvgpr_write_b32 a[242], 0\n\tv_accvgpr_write_b32 a[243], 0\n\t"
                     "v_accvgpr_write_b32 a[244], 0\n\tv_accvgpr_write_b32 a[245], 0\n\tv_accvgpr_write_b32 a[246], 0\n\tv_accvgpr_write_b32 a[247], 0\n\t"
                     "v_accvgpr_write_b32 a[248], 0\n\tv_accvgpr_write_b32 a[249], 0\n\tv_accvgpr_write_b32 a[250], 0\n\tv_accvgpr_write_b32 a[251], 0\n\t"
                     "v_accvgpr_write_b32 a[252], 0\n\tv_accvgpr_write_b32 a[253], 0\n\tv_accvgpr_write_b32 a[254], 0\n\tv_accvgpr_write_b32 a[255], 0");
#pragma unroll
        for (int j = 0; j < 4; ++j) outv[j] = (f4){0.f, 0.f, 0.f, 0.f};
        for (int t = 0; t < nit;) {
            ITEM(0, h0, f0);
            ITEM(1, h1, f1);
            STAMP(40 + ((hj + hs * (t - 1)) >> 6));
        }
#pragma unroll
        for (int j = 0; j < 4; ++j) __builtin_amdgcn_raw_buffer_store_b128(__builtin_bit_cast(u4, outv[j]), ro, voffo, oprev + j * 4096u, OAUX);
        WAIT_V(0);
        if (lane == 0) __hip_atomic_fetch_add((GAS unsigned*)((unsigned char*)A.wsp[(rank + 1 + pi) & (NR - 1)] + WS_DONE + rank * 128), 1u, __ATOMIC_RELAXED, __HIP_MEMORY_SCOPE_SYSTEM);
        } else if (hlp && h == 127) {
            if (!wave_wait<true, 16>(ws + WS_DONE + ((rank + 1 + lane % 7) & (NR - 1)) * 128, call * (18 + TWV), ws + WS_TMO, call) && lane == 0)
                ((float*)A.out[rank & (NR - 1)])[(size_t)rank * 1024 * 1024] = __builtin_nanf("");
            STAMP(31);
        }
        __builtin_amdgcn_s_waitcnt(0x0F70);
        STAMP(ni == 7 ? 60 : 12);
    }
    WAIT_V(0);
    STAMP(30);
    STAMP_OUT();
    __syncthreads();
    if (tid == 0) __hip_atomic_fetch_add((GAS unsigned*)(ws + WS_TICKET), 1u, __ATOMIC_RELAXED, __HIP_MEMORY_SCOPE_AGENT);
}

extern "C" void kernel_launch(int rank, int nranks, ncclComm_t comm, hipStream_t stream,
                              void* const* d_in, const int* in_sizes, int n_in,
                              void* d_out, int out_size, const int* in_dtypes, int out_dtype,
                              void* const* d_out_all, void* const* d_ws_all, size_t ws_bytes) {
    if (nranks != NR || ws_bytes < WS_NEED || n_in < 2 || in_sizes[0] != M * K || in_sizes[1] != K * N || out_size != 8192 * 1024) { fprintf(stderr, "unexpected problem shape\n"); return; }
    static bool attr = false;
    if (!attr) { HIP_CHECK(hipFuncSetAttribute((const void*)gemm_a2a, hipFuncAttributeMaxDynamicSharedMemorySize, LDS_BYTES)); attr = true; }
    Args a; a.x = (const float*)d_in[0]; a.w = (const float*)d_in[1]; a.ws = d_ws_all[rank]; a.rank = rank; a.pad = 0;
    for (int p = 0; p < NR; ++p) { a.out[p] = d_out_all[p]; a.wsp[p] = d_ws_all[p]; }
    hipLaunchKernelGGL(gemm_a2a, dim3(256), dim3(256), LDS_BYTES, stream, a);
    HIP_CHECK(hipGetLastError());
}
```

```cpp
#include <hip/hip_runtime.h>
#include <rccl/rccl.h>
#include <cstdio>

#define HIP_CHECK(expr) do { hipError_t _e = (expr); if (_e != hipSuccess) { \
    fprintf(stderr, "HIP error %s at %s:%d\n", hipGetErrorString(_e), __FILE__, __LINE__); \
    return; } } while (0)

typedef _Float16 h2 __attribute__((ext_vector_type(2)));
typedef float f4 __attribute__((ext_vector_type(4)));
typedef float f2 __attribute__((ext_vector_type(2)));
typedef unsigned u4 __attribute__((ext_vector_type(4)));
typedef unsigned u2 __attribute__((ext_vector_type(2)));
#define LAS __attribute__((address_space(3)))
#define GAS __attribute__((address_space(1)))

constexpr int NR = 8;
constexpr int M = 1024, N = 8192, K = 8192;
constexpr int XS = 16384;
constexpr int LDS_BYTES = 8 * XS;
constexpr unsigned WS_TMO = 128, WS_TICKET = 256, WS_START = 1024, WS_DONE = 2048;
constexpr unsigned WS_XF = 4096;
constexpr unsigned WS_PF = 16384;
constexpr size_t WS_XS = 1u << 20, WS_PART = 17u << 20;
constexpr unsigned PSTRIDE = 262144u;
constexpr unsigned PART_BYTES = 624u * PSTRIDE;
constexpr size_t WS_NEED = WS_PART + PART_BYTES;
constexpr unsigned OOB = 0x80000000u;
constexpr unsigned long long DEADLINE = 40000000ull;
constexpr int HWV = 18, TWV = 20, HB = 704;
constexpr int PACE0 = 430, PACED = 11, PACEF = 275, PACEN = 30;

#define ACLOB "a0","a1","a2","a3","a4","a5","a6","a7","a8","a9","a10","a11","a12","a13","a14","a15", \
    "a16","a17","a18","a19","a20","a21","a22","a23","a24","a25","a26","a27","a28","a29","a30","a31", \
    "a32","a33","a34","a35","a36","a37","a38","a39","a40","a41","a42","a43","a44","a45","a46","a47", \
    "a48","a49","a50","a51","a52","a53","a54","a55","a56","a57","a58","a59","a60","a61","a62","a63", \
    "a64","a65","a66","a67","a68","a69","a70","a71","a72","a73","a74","a75","a76","a77","a78","a79", \
    "a80","a81","a82","a83","a84","a85","a86","a87","a88","a89","a90","a91","a92","a93","a94","a95", \
    "a96","a97","a98","a99","a100","a101","a102","a103","a104","a105","a106","a107","a108","a109","a110","a111", \
    "a112","a113","a114","a115","a116","a117","a118","a119","a120","a121","a122","a123","a124","a125","a126","a127", \
    "a128","a129","a130","a131","a132","a133","a134","a135","a136","a137","a138","a139","a140","a141","a142","a143", \
    "a144","a145","a146","a147","a148","a149","a150","a151","a152","a153","a154","a155","a156","a157","a158","a159", \
    "a160","a161","a162","a163","a164","a165","a166","a167","a168","a169","a170","a171","a172","a173","a174","a175", \
    "a176","a177","a178","a179","a180","a181","a182","a183","a184","a185","a186","a187","a188","a189","a190","a191", \
    "a192","a193","a194","a195","a196","a197","a198","a199","a200","a201","a202","a203","a204","a205","a206","a207", \
    "a208","a209","a210","a211","a212","a213","a214","a215","a216","a217","a218","a219","a220","a221","a222","a223", \
    "a224","a225","a226","a227","a228","a229","a230","a231","a232","a233","a234","a235","a236","a237","a238","a239", \
    "a240","a241","a242","a243","a244","a245","a246","a247","a248","a249","a250","a251","a252","a253","a254","a255"
#define BAR() do { asm volatile("" ::: "memory"); __builtin_amdgcn_s_barrier(); asm volatile("" ::: "memory"); } while (0)
#define SCHED() __builtin_amdgcn_sched_barrier(0)
#define WAIT_V(n) asm volatile("s_waitcnt vmcnt(%0)" ::"n"(n) : "memory")
#define KNOWN_LGKM0() __builtin_amdgcn_s_waitcnt(0xC07F)
#define KNOWN_VM0() __builtin_amdgcn_s_waitcnt(0x0F70)

__device__ __forceinline__ unsigned cvt2(float a, float b) { f2 v = {a, b}; h2 h = __builtin_convertvector(v, h2); return __builtin_bit_cast(unsigned, h); }
__device__ __forceinline__ float silu(float y) {
    const float yc = __builtin_amdgcn_fmed3f(y, -60.f, 60.f);
    const float e = __builtin_amdgcn_exp2f(yc * -1.44269504088896f);
    return y * __builtin_amdgcn_rcpf(1.0f + e);
}
__device__ __forceinline__ unsigned ld_sys(const void* p) { return __hip_atomic_load((GAS unsigned*)p, __ATOMIC_RELAXED, __HIP_MEMORY_SCOPE_SYSTEM); }
__device__ __forceinline__ unsigned ld_agt(const void* p) { return __hip_atomic_load((GAS unsigned*)p, __ATOMIC_RELAXED, __HIP_MEMORY_SCOPE_AGENT); }
__device__ __forceinline__ void st_sys(void* p, unsigned v) { __hip_atomic_store((GAS unsigned*)p, v, __ATOMIC_RELAXED, __HIP_MEMORY_SCOPE_SYSTEM); }
__device__ __forceinline__ void st_agt(void* p, unsigned v) { __hip_atomic_store((GAS unsigned*)p, v, __ATOMIC_RELAXED, __HIP_MEMORY_SCOPE_AGENT); }
template <bool SYS, int NAP>
__device__ __forceinline__ bool wave_wait(const void* p, unsigned want, void* tmo, unsigned call) {
    const unsigned long long t0 = __builtin_amdgcn_s_memrealtime();
    for (unsigned it = 1;; ++it) {
        if (__all((SYS ? ld_sys(p) : ld_agt(p)) >= want)) break;
        if ((it & 31u) == 0u && (ld_agt(tmo) == call || __builtin_amdgcn_s_memrealtime() - t0 > DEADLINE)) { st_agt(tmo, call); return false; }
        __builtin_amdgcn_s_sleep(NAP);
    }
    __builtin_amdgcn_fence(__ATOMIC_ACQUIRE, "wavefront");
    return true;
}

struct Args { const float* x; const float* w; void* ws; void* out[NR]; void* wsp[NR]; int rank; int pad; };

__device__ __forceinline__ int column_of(int jb, int ni, int role) {
    return jb == 0 ? 6 - ni : jb == 1 ? (role ? 20 - ni : 7 + ni) : role ? 28 + ((ni + 1) & 3) : 21 + ni;
}
__device__ __forceinline__ int strip_of(int o, int wv, int rank, int em) {
    const int q = o * 4 + wv, pi = q % 7, e = q / 7;
    return q < 112 ? ((rank + 1 + pi) & 7) * 16 + (((e ^ em) + pi) & 15) : rank * 16 + (q - 112);
}
__device__ __forceinline__ unsigned part_slot(int o, int mt) { return (unsigned)(o < 7 ? o * 32 + mt * 8 : 112 + o * 16 + mt * 4) * PSTRIDE; }

constexpr int lpos(int j) { return 222 - (47 - j) * 4; }
constexpr int lat(int p) { for (int j = 0; j < 48; ++j) if (lpos(j) == p) return j; return -1; }
constexpr int dpos(int q) { return 141 + q * 13 / 4; }
constexpr int dat(int p) { for (int q = 0; q < 64; ++q) if (dpos(q) == p) return q; return -1; }
constexpr int dfirst() { int q = 0; while (dpos(q) < 256) ++q; return q; }
constexpr int vwait(int j, int p, int mode) {
    const int a = lpos(j) < p ? lpos(j) : lpos(j) - 256;
    int n = 0;
    for (int k = 0; k < 48; ++k) { if (lpos(k) > a && lpos(k) < p) ++n; if (lpos(k) - 256 > a) ++n; }
    for (int q = 0; q < 64; ++q) { const int d = dpos(q) - (mode == 1 ? 256 : 0); if (mode && d >= a && d < p) ++n; }
    return n < 63 ? n : 63;
}
constexpr int lafter(int p, int mode) { int n = mode == 2 ? dfirst() : 0; for (int k = 0; k < 48; ++k) if (lpos(k) > p) ++n; return n < 63 ? n : 63; }
constexpr bool sched_ok() {
    for (int q = 0; q < 64; ++q) if (dpos(q) < 128 + ((q >> 2) + 1) * 8 + 4 || dpos(q) >= 256 + (q >> 2) * 8 - 2) return false;
    for (int j = 0; j < 8; ++j) if (lpos(16 + j) < (j | 1) * 8 + 5 || lpos(24 + j) < 64 + (j | 1) * 8 + 5 || lpos(40 + j) < 128 + (j | 1) * 8 + 5) return false;
    return lpos(0) >= 32 && lpos(32) > 113 && lpos(47) < 256 && dpos(63) - 256 < lpos(15);
}
static_assert(sched_ok(), "schedule");

#define PLOAD(dst, soff) _Pragma("unroll") for (int t_ = 0; t_ < 4; ++t_) \
        dst[t_] = __builtin_bit_cast(f4, __builtin_amdgcn_raw_buffer_load_b128(rp, lane * 16 + t_ * 1024, (soff), 18))
#define OSTORE(j) __builtin_amdgcn_raw_buffer_store_b128(__builtin_bit_cast(u4, outv[j]), ro, voffo, oprev + (j) * 4096u, 17)
#define ACTIVATE() _Pragma("unroll") for (int j = 0; j < 4; ++j) { \
        outv[j] = (f4){silu(sum[0][j]), silu(sum[1][j]), silu(sum[2][j]), silu(sum[3][j])}; \
        if (bad) outv[j] = (f4){__builtin_nanf(""), __builtin_nanf(""), __builtin_nanf(""), __builtin_nanf("")}; }

__global__ __launch_bounds__(256, 1) void gemm_a2a(Args A) {
    extern __shared__ __attribute__((aligned(16))) unsigned char smem_[];
    LAS unsigned char* smem = (LAS unsigned char*)smem_;
    const int tid = threadIdx.x, lane = tid & 63, wid = __builtin_amdgcn_readfirstlane(tid >> 6);
    const int rank = A.rank & (NR - 1);
    unsigned char* ws = (unsigned char*)A.ws;
    const int xcd = blockIdx.x & 7, idx = blockIdx.x >> 3;
    const int mt = idx & 3, ni = idx >> 2, kq = xcd & 3, g = xcd >> 2;
    const __amdgpu_buffer_rsrc_t rx = __builtin_amdgcn_make_buffer_rsrc((void*)(ws + WS_XS), 0, (unsigned)(M * K * 2), 0x00020000);
    const __amdgpu_buffer_rsrc_t rp = __builtin_amdgcn_make_buffer_rsrc((void*)(ws + WS_PART), 0, PART_BYTES, 0x00020000);
    const unsigned lane16 = lane * 16;
    const unsigned voffo = (unsigned)(lane >> 4) * 16384u + (unsigned)(lane & 15) * 16u;
    const unsigned tk0 = ld_agt(ws + WS_TICKET);
    const __amdgpu_buffer_rsrc_t rxi = __builtin_amdgcn_make_buffer_rsrc((void*)A.x, 0, (unsigned)(M * K * 4), 0x00020000);
    const unsigned xvi = (unsigned)(lane >> 3) * (K * 4) + (lane & 7) * 16, xvo = ((lane >> 2) & 1) * XS + (lane & 3) * 256 + (lane >> 3) * 16;
    const unsigned xbi = (unsigned)(mt * 256 + wid * 64) * (K * 4) + kq * 8192, xbo = (unsigned)(mt * 256 + kq * 64) * XS + wid * 4096;
#define XLD(dst, pr) _Pragma("unroll") for (int f_ = 0; f_ < 8; ++f_) _Pragma("unroll") for (int e_ = 0; e_ < 2; ++e_) \
        dst[f_][e_] = __builtin_bit_cast(f4, __builtin_amdgcn_raw_buffer_load_b128(rxi, xvi + e_ * 128, xbi + f_ * (8 * K * 4) + (pr) * 256, 0))
#define XST(src, pr) _Pragma("unroll") for (int f_ = 0; f_ < 8; ++f_) { \
        const u4 o_ = {cvt2(src[f_][0][0], src[f_][0][1]), cvt2(src[f_][0][2], src[f_][0][3]), cvt2(src[f_][1][0], src[f_][1][1]), cvt2(src[f_][1][2], src[f_][1][3])}; \
        __builtin_amdgcn_raw_buffer_store_b128(o_, rx, xvo + (f_ >> 1) * 1024 + (f_ & 1) * 128, xbo + (pr) * (2 * XS), 16); }
#define XPR(p) (g ? 31 - (p) : (p))
#define XFW(pr) ((((pr) >> 1) * 4 + wid) * 8 + ((pr) & 1) * 4)
    {
        f4 v[8][2];
        XLD(v, XPR(ni));
        XST(v, XPR(ni));
    }
    const unsigned call = (__builtin_amdgcn_readfirstlane(tk0) >> 8) + 1u;
    unsigned char* xf = ws + WS_XF + (mt * 4 + kq) * 512;
    unsigned char* tmo = ws + WS_TMO;
    bool bad = false;
    const int role = g ^ (ni & 1);
    const int em = (call & 1) ? 15 : 0;

    if (ni < 7) {
        const __amdgpu_buffer_rsrc_t rw = __builtin_amdgcn_make_buffer_rsrc((void*)A.w, 0, (unsigned)((size_t)K * N * 4), 0x00020000);
        const __amdgpu_buffer_rsrc_t rf = __builtin_amdgcn_make_buffer_rsrc((void*)(ws + WS_PF), 0, 65536u, 0x00020000);
        unsigned voffw[8];
#pragma unroll
        for (int j = 0; j < 8; ++j) { voffw[j] = (unsigned)((lane >> 4) * 4 + (j & 3) + 32 * (j >> 2)) * (N * 4) + (unsigned)(lane & 15) * 16; asm volatile("" : "+v"(voffw[j])); }
        const unsigned voffx = wid * 4096 + lane16;
        const unsigned ldsx = (unsigned)(size_t)smem + wid * 4096;
        const int e0 = role ? 15 - ni : ni + 1, e1 = e0 + 16, nwin = (role && ni < 3) ? e1 : e0 + 32;
        const int c0 = strip_of(column_of(0, ni, role), wid, rank, em) * 256, c1 = strip_of(column_of(1, ni, role), wid, rank, em) * 256,
                  c2 = strip_of(column_of(2, ni, role), wid, rank, em) * 256;
        unsigned wrel[4], xrel[4];
#pragma unroll
        for (int s = 0; s < 4; ++s) { const int st = ((g ? 3 - s : s) + mt) & 3; wrel[s] = (unsigned)((st >> 1) * 64 + (st & 1) * 16) * (N * 4); xrel[s] = (unsigned)(mt * 256 + st) * XS; }
#define WIN(i) (kq * 16 + (g ? 15 - (i) : (i)))
#define PIN(x) asm volatile("" : "+s"(x))
#define GLDS(so, i) asm volatile("buffer_load_dwordx4 %0, %1, %2 offen offset:%c3 sc1 lds" :: "v"(voffx), "s"(rx), "s"(so), "i"((i) * 1024) : "memory")
#define WLD(rr, h, j, so) asm volatile("buffer_load_dwordx4 %0, %1, %2, %3 offen" : "=v"(rr[h][j]) : "v"(voffw[j]), "s"(rw), "s"(so) : "memory")
#define CVT(dst, rr, h, t, q) do { unsigned c_; asm volatile("v_cvt_pk_f16_f32 %0, %1, %2" : "=v"(c_) : "v"(rr[h][2 * (q)][t]), "v"(rr[h][2 * (q) + 1][t])); dst[h][t][q] = c_; } while (0)
#define MFMA(m, t, av, bv) asm volatile("v_mfma_f32_16x16x32_f16 a[%c2:%c3], %0, %1, a[%c2:%c3]" :: "v"(av), "v"(bv), "i"(((m) * 4 + (t)) * 4), "i"(((m) * 4 + (t)) * 4 + 3))
#define MFMA0(m, t, av, bv) asm volatile("v_mfma_f32_16x16x32_f16 a[%c2:%c3], %0, %1, 0" :: "v"(av), "v"(bv), "i"(((m) * 4 + (t)) * 4), "i"(((m) * 4 + (t)) * 4 + 3))
#define LDA(dst, aaddr, off) dst = *(const LAS u4*)(smem + (aaddr) + (off))
        u4 a[4][2], b0[2][4], b1[2][4], r0[2][8], r1[2][8];
        u2 xv; const unsigned lane8 = lane * 8;
        asm volatile("" ::: ACLOB);
        unsigned wb1 = ((unsigned)WIN(1) << 22) + (unsigned)(e0 > 1 ? c0 : c1), xb1 = (unsigned)WIN(1) << 16;
        {
            const unsigned wb0 = ((unsigned)WIN(0) << 22) + (unsigned)c0, xb0 = (unsigned)WIN(0) << 16;
#pragma unroll
            for (int j = 0; j < 8; ++j) WLD(r0, 0, j, wb0 + wrel[0]);
            WAIT_V(8);
            if (lane == 0) st_agt(xf + XFW(XPR(ni)), call);
#pragma unroll
            for (int j = 0; j < 8; ++j) WLD(r0, 1, j, wb0 + wrel[1]);
#pragma unroll
            for (int h = 0; h < 2; ++h)
#pragma unroll
                for (int j = 0; j < 8; ++j) WLD(r1, h, j, wb0 + wrel[2 + h]);
            if (!wave_wait<false, 2>(xf + (g ? 448 : 0) + (lane & 2) * 16 + wid * 8 + (lane & 1) * 4, call, tmo, call)) bad = true;
            KNOWN_VM0();
            WAIT_V(0);
#pragma unroll
            for (int s = 0; s < 4; ++s) {
                const unsigned so = xb0 + xrel[s];
                asm volatile("s_mov_b32 m0, %0\n\ts_nop 0" :: "s"(ldsx + s * XS) : "memory");
                GLDS(so, 0); GLDS(so, 1); GLDS(so, 2); GLDS(so, 3);
            }
            SCHED();
#pragma unroll
            for (int h = 0; h < 2; ++h)
#pragma unroll
                for (int q = 0; q < 4; ++q)
#pragma unroll
                    for (int t = 0; t < 4; ++t) CVT(b0, r0, h, t, q);
            SCHED();
#pragma unroll
            for (int j = 0; j < 8; ++j) WLD(r0, 0, j, wb1 + wrel[0]);
            KNOWN_LGKM0();
            WAIT_V(8); BAR();
#pragma unroll
            for (int m = 0; m < 2; ++m) { LDA(a[m][0], lane16, m * 1024); LDA(a[m][1], lane16, XS + m * 1024); }
        }
#define DUMP(q) asm volatile("buffer_store_dwordx4 a[%c3:%c4], %0, %1, %2 offen offset:%c5 sc1" :: "v"(lane16), "s"(rp), "s"(ob + (((q) & 63) >> 2) * 4096u), \
            "i"(((q) & 63) * 4), "i"(((q) & 63) * 4 + 3), "i"(((q) & 3) * 1024) : "memory")
#define DUMPC(q) asm volatile("s_bitcmp1_b32 %2, 31\n\ts_cbranch_scc1 1f\n\tbuffer_store_dwordx4 a[%c3:%c4], %0, %1, %2 offen offset:%c5 sc1\n1:" \
            :: "v"(lane16), "s"(rp), "s"(ob + (((q) & 63) >> 2) * 4096u), "i"(((q) & 63) * 4), "i"(((q) & 63) * 4 + 3), "i"(((q) & 3) * 1024) : "memory", "scc")
#define PUBLISH() asm volatile("buffer_store_dword %0, %1, %2, %3 offen sc1" :: "v"(callv), "v"(zerov), "s"(rf), "s"(pend) : "memory")
#define GAP(MODE, B, m, e) do { constexpr int p_ = (B) * 128 + (m) * 8 + (e), j_ = lat(p_), jn_ = lat(p_ + 1), q_ = dat(p_ + ((MODE) == 1 ? 256 : 0)); \
            if (j_ >= 40) WLD(r0, 0, j_ & 7, so); else if (j_ >= 32) GLDS(so, j_ & 3); \
            else if (j_ >= 24) WLD(r1, 1, j_ & 7, so); else if (j_ >= 16) WLD(r1, 0, j_ & 7, so); \
            else if (j_ >= 8) GLDS(so, j_ & 3); else if (j_ >= 0) WLD(r0, 1, j_ & 7, so); \
            if (jn_ == 0 || jn_ == 16 || jn_ == 24) { so = wb1 + wrel[jn_ == 0 ? 1 : jn_ >> 3]; PIN(so); } \
            if (jn_ == 40) { so = wb2 + wrel[0]; PIN(so); } \
            if (jn_ == 8 || jn_ == 12 || jn_ == 32 || jn_ == 36) { so = xb1 + xrel[(jn_ >= 32 ? 2 : 0) + ((jn_ >> 2) & 1)]; PIN(so); \
                asm volatile("s_add_u32 m0, %0, %1" :: "s"(lds2), "i"(((jn_ >= 32 ? 2 : 0) + ((jn_ >> 2) & 1)) * XS) : "memory", "scc"); } \
              \
            if (p_ == 2) { t2 = i + 2; PIN(t2); } \
            if (p_ == 5) { cc = t2 >= e0 ? c1 : c0; PIN(cc); } \
            if (p_ == 6) { cc = t2 >= e1 ? c2 : cc; PIN(cc); } \
            if (p_ == 14) { kp = (kp + kstep) & 15; PIN(kp); } \
            if (p_ == 18) { kw = kp | (kq * 16); PIN(kw); } \
            if (p_ == 21) { wb2 = ((unsigned)kw << 22) + (unsigned)cc; PIN(wb2); } \
            if (p_ == 22) { xb2 = (unsigned)kw << 16; PIN(xb2); } \
            if (p_ == 26) { lds2 ^= 4 * XS; PIN(lds2); } \
            if (p_ == 109 || p_ == 237) { an ^= 4 * XS; asm volatile("" : "+v"(an)); } \
            if ((MODE) == 1 && q_ >= 0) DUMPC(q_); else if ((MODE) == 2 && q_ >= 0) DUMP(q_); \
            SCHED(); } while (0)
#define ITER(MODE, B, bc, bn, rc, m, ac) do { \
            constexpr bool FIRST = (MODE) == 1; \
            constexpr int t0 = (m & 1) ? 3 : 0, dt = (m & 1) ? -1 : 1, hh = m >> 3, qq = (m >> 1) & 3; \
            if (m == 14) { WAIT_V(vwait((B) ? 15 : 39, (B) * 128 + 112, MODE)); BAR(); } \
            if (FIRST) MFMA0(m, t0, a[m & 3][0], bc[0][t0]); else MFMA(m, t0, a[m & 3][0], bc[0][t0]); \
            if (m < 14) LDA(a[(m + 2) & 3][0], ac, (m + 2) * 1024); else LDA(a[(m + 2) & 3][0], an, (m - 14) * 1024); \
            GAP(MODE, B, m, 0); \
            MFMA(m, t0, a[m & 3][1], bc[1][t0]); \
            if (m == 0) WAIT_V(vwait((B) ? 47 : 23, (B) * 128 + 1, MODE)); \
            if (m == 8) WAIT_V(vwait((B) ? 7 : 31, (B) * 128 + 65, MODE)); \
            CVT(bn, rc, hh, (m & 1) * 2, qq); \
            GAP(MODE, B, m, 1); \
            if (FIRST) MFMA0(m, t0 + dt, a[m & 3][1], bc[1][t0 + dt]); else MFMA(m, t0 + dt, a[m & 3][1], bc[1][t0 + dt]); \
            if ((B) && m == 14) PUBLISH(); \
            GAP(MODE, B, m, 2); \
            MFMA(m, t0 + dt, a[m & 3][0], bc[0][t0 + dt]); \
            if (m < 14) LDA(a[(m + 2) & 3][1], ac, XS + (m + 2) * 1024); else LDA(a[(m + 2) & 3][1], an, XS + (m - 14) * 1024); \
            GAP(MODE, B, m, 3); \
            if (FIRST) MFMA0(m, t0 + 2 * dt, a[m & 3][0], bc[0][t0 + 2 * dt]); else MFMA(m, t0 + 2 * dt, a[m & 3][0], bc[0][t0 + 2 * dt]); \
            CVT(bn, rc, hh, (m & 1) * 2 + 1, qq); \
            GAP(MODE, B, m, 4); \
            if ((B) && m == 0) asm volatile("global_load_dwordx2 %0, %1, %2 sc1" : "=v"(xv) : "v"(lane8), "s"(xf) : "memory"); \
            MFMA(m, t0 + 2 * dt, a[m & 3][1], bc[1][t0 + 2 * dt]); \
            GAP(MODE, B, m, 5); \
            if (FIRST) MFMA0(m, t0 + 3 * dt, a[m & 3][1], bc[1][t0 + 3 * dt]); else MFMA(m, t0 + 3 * dt, a[m & 3][1], bc[1][t0 + 3 * dt]); \
            GAP(MODE, B, m, 6); \
            MFMA(m, t0 + 3 * dt, a[m & 3][0], bc[0][t0 + 3 * dt]); \
            GAP(MODE, B, m, 7); \
        } while (0)
#define STEP(MODE, B, bc, bn, rc, ac, an_) do { unsigned& an = an_; \
        ITER(MODE, B, bc, bn, rc, 0, ac); ITER(MODE, B, bc, bn, rc, 1, ac); ITER(MODE, B, bc, bn, rc, 2, ac); ITER(MODE, B, bc, bn, rc, 3, ac); \
        ITER(MODE, B, bc, bn, rc, 4, ac); ITER(MODE, B, bc, bn, rc, 5, ac); ITER(MODE, B, bc, bn, rc, 6, ac); ITER(MODE, B, bc, bn, rc, 7, ac); \
        ITER(MODE, B, bc, bn, rc, 8, ac); ITER(MODE, B, bc, bn, rc, 9, ac); ITER(MODE, B, bc, bn, rc, 10, ac); ITER(MODE, B, bc, bn, rc, 11, ac); \
        ITER(MODE, B, bc, bn, rc, 12, ac); ITER(MODE, B, bc, bn, rc, 13, ac); ITER(MODE, B, bc, bn, rc, 14, ac); ITER(MODE, B, bc, bn, rc, 15, ac); \
    } while (0)
        int i = 0;
        unsigned pend = OOB, ob = OOB;
        unsigned so = 0, wb2 = 0, xb2 = 0, lds2 = ldsx, aA = lane16, aB = lane16 + 6 * XS, callv = call, zerov = 0;
        unsigned tprev = (unsigned)__builtin_amdgcn_s_memrealtime();
        KNOWN_LGKM0();
        int t2 = 0, cc = 0, kw = 0, kp = WIN(1) & 15;
        const int kstep = g ? -1 : 1;
        asm volatile("" : "+v"(callv), "+v"(zerov));
#define STEP_A(MODE) STEP(MODE, 0, b0, b1, r1, aA, aB)
#define STEP_B(MODE) do { STEP(MODE, 1, b1, b0, r0, aB, aA); \
            pend = OOB; wb1 = wb2; xb1 = xb2; ++i; \
            if (i <= PACEN) { const int pc_ = PACE0 - i * PACED; const unsigned due = tprev + (pc_ > PACEF ? pc_ : PACEF); \
                while ((int)((tprev = (unsigned)__builtin_amdgcn_s_memrealtime()) - due) < 0) __builtin_amdgcn_s_sleep(1); \
                KNOWN_LGKM0(); } \
            asm volatile("s_waitcnt vmcnt(%1)" : "+v"(xv) : "n"(lafter(132, MODE)) : "memory"); \
            if (i < 15) {     \
                const int pw = g ? 14 - i : i + 1; \
                if (((__ballot(xv[0] >= call && xv[1] >= call) >> (pw * 4 + wid)) & 1) == 0) { \
                    if (!wave_wait<false, 4>(xf + pw * 32 + wid * 8 + (lane & 1) * 4, call, tmo, call)) bad = true; \
                    KNOWN_LGKM0(); } \
                SCHED(); \
            } } while (0)
        for (int jb = 0; i < nwin; ++jb) {
            const int iend = jb == 0 ? e0 : jb == 1 ? e1 : nwin;
            STEP_A(1);
            while (i + 1 < iend) { STEP_B(0); STEP_A(0); }
            const int o = column_of(jb, ni, role), pslot = jb == 0 ? role * 4 + kq : kq;
            ob = part_slot(o, mt) + pslot * PSTRIDE + wid * 65536u;
            STEP_B(2);
            pend = bad ? OOB : (unsigned)(((o * 4 + mt) * 4 + wid) * 128 + pslot * 4);
        }
        asm volatile("s_nop 15"); SCHED();
#pragma unroll
        for (int q = dfirst(); q < 64; ++q) DUMP(q);
        WAIT_V(0);
        PUBLISH();
        if (role && ni >= 3) {
            const int o = column_of(2, ni, 1);
            const __amdgpu_buffer_rsrc_t ro = __builtin_amdgcn_make_buffer_rsrc(A.out[rank], 0, 8192u * 1024 * 4, 0x00020000);
            if (!wave_wait<false, 4>(ws + WS_PF + ((o * 4 + mt) * 4 + wid) * 128 + (lane & 3) * 4, call, tmo, call)) bad = true;
            const unsigned pb = part_slot(o, mt) + wid * 65536u + kq * 16384u;
            f4 v[4][4], sum[4], outv[4];
#pragma unroll
            for (int c = 0; c < 4; ++c) { PLOAD(v[c], pb + c * PSTRIDE); outv[c] = (f4){0.f, 0.f, 0.f, 0.f}; }
            unsigned oprev = OOB;
#pragma nounroll
            for (int b = 0; b < 4; ++b) {
                const unsigned nx = b < 3 ? pb + (b + 1) * 4096u : OOB;
#pragma unroll
                for (int c = 1; c < 4; ++c) {
#pragma unroll
                    for (int u = 0; u < 4; ++u) { if (c == 1) sum[u] = v[0][u] + v[1][u]; else sum[u] += v[c][u]; }
#pragma unroll
                    for (int u = 0; u < 4; ++u) asm volatile("" : "+v"(sum[u]));
                    SCHED();
                    if (c == 1) PLOAD(v[0], nx);
                    PLOAD(v[c], nx + c * PSTRIDE);
                    if (c == 1) OSTORE(0);
                    OSTORE(c);
                    SCHED();
                }
                ACTIVATE();
                oprev = (unsigned)(rank * 1024 + mt * 256 + kq * 64 + b * 16) * 4096u + ((o - 28) * 4 + wid) * 256u;
                SCHED();
            }
#pragma unroll
            for (int j = 0; j < 4; ++j) OSTORE(j);
        }
    } else {
        WAIT_V(0);
        if (lane == 0) st_agt(xf + XFW(XPR(ni)), call);
        if (blockIdx.x == 224 && wid == 0 && lane < NR) st_sys((unsigned char*)A.wsp[lane & (NR - 1)] + WS_START + rank * 128, call);
        f4 u0[8][2], u1[8][2];
        XLD(u0, XPR(8)); XLD(u1, XPR(9));
#pragma nounroll
        for (int p = 8; p < 14; p += 2) {
            XST(u0, XPR(p)); SCHED(); XLD(u0, XPR(p + 2)); SCHED();
            XST(u1, XPR(p + 1)); SCHED(); XLD(u1, XPR(p + 3)); SCHED();
            WAIT_V(16);
            if (lane < 2) st_agt(xf + XFW(XPR(p) & ~1) + lane * 4, call);
        }
        XST(u0, XPR(14)); XST(u1, XPR(15));
        WAIT_V(0);
        if (lane < 2) st_agt(xf + XFW(XPR(14) & ~1) + lane * 4, call);
        KNOWN_VM0();
    }
    if (ni == 7 || (role && ni < 3)) {
        const bool hlp = ni == 7;
        const int h = hlp ? wid * 32 + xcd * 4 + mt : wid * 48 + (ni * 4 + kq) * 4 + mt, pi = h % 7, hs = hlp ? HWV : TWV;
        const int hj = h / 7 + (hlp ? 0 : HB), dst = (rank + 1 + pi) & (NR - 1);
        if (h < 7 * hs) {
            if (!wave_wait<true, 8>(ws + WS_START + dst * 128, call, tmo, call)) bad = true;
            const __amdgpu_buffer_rsrc_t ro = __builtin_amdgcn_make_buffer_rsrc(A.out[dst], 0, 8192u * 1024 * 4, 0x00020000);
#define IT_Q(t) (pi + 7 * ((hj + hs * (t)) >> 6))
#define IT_MT(t) (((hj + hs * (t)) >> 4) & 3)
#define IT_FLAG(t, hf) (ws + WS_PF + (((IT_Q(t) >> 2) * 4 + IT_MT(t)) * 4 + (IT_Q(t) & 3)) * 128 + ((hf) * 4 + (lane & 3)) * 4)
#define IT_HF(t) (IT_Q(t) < 28 ? 1 : 0)
#define IT_PB(t, hf) (part_slot(IT_Q(t) >> 2, IT_MT(t)) + (hf) * (4 * PSTRIDE) + (IT_Q(t) & 3) * 65536u + ((hj + hs * (t)) & 15) * 4096u)
            f4 v[8][4], sum[4], outv[4];
            asm volatile("" ::: ACLOB);
            if (hlp) {
#pragma unroll
                for (int t = 14; t >= 0; t -= 2) {
                    const bool va = hj + HWV * t < 256, vb = t > 0;
                    if (va && !wave_wait<false, 24>(IT_FLAG(t, 0), call, tmo, call)) bad = true;
                    const unsigned pa = va ? IT_PB(t, 0) : OOB;
#pragma unroll
                    for (int i = 0; i < 4; ++i) PLOAD(v[i], pa + i * PSTRIDE);
                    if (vb) {
                        if (!wave_wait<false, 24>(IT_FLAG(t - 1, 0), call, tmo, call)) bad = true;
#pragma unroll
                        for (int i = 0; i < 4; ++i) PLOAD(v[4 + i], IT_PB(t - 1, 0) + i * PSTRIDE);
                    }
                    SCHED();
#pragma unroll
                    for (int b = 0; b < (vb ? 2 : 1); ++b)
#pragma unroll
                        for (int u = 0; u < 4; ++u) {
                            const f4 x = (v[b * 4][u] + v[b * 4 + 1][u]) + (v[b * 4 + 2][u] + v[b * 4 + 3][u]);
#pragma unroll
                            for (int j = 0; j < 4; ++j) asm volatile("v_accvgpr_write_b32 a[%c1], %0" :: "v"(x[j]), "i"((t - b) * 16 + u * 4 + j));
                        }
                    SCHED();
                }
            }
            const int nit = ((hlp ? HB : 1024) - hj + hs - 1) / hs, tpre = hlp ? 0 : 64;
            bool h0 = false, h1 = false;
            unsigned f0 = 0, f1 = 0;
            unsigned oprev = OOB;
#define PRE_CASE(T) case T: _Pragma("unroll") for (int u = 0; u < 16; ++u) { float f; asm volatile("v_accvgpr_read_b32 %0, a[%c1]" : "=v"(f) : "i"((T) * 16 + u)); sum[u >> 2][u & 3] = f; } break;
#define ITEM(B, hv, fv) do { \
                if (!hv) { \
                    _Pragma("unroll") for (int j = 0; j < 4; ++j) OSTORE(j); \
                    oprev = OOB; \
                    if (!wave_wait<false, 24>(IT_FLAG(t, IT_HF(t)), call, tmo, call)) bad = true; \
                    fv = ld_agt(IT_FLAG(t + 2, IT_HF(t + 2))); \
                    const unsigned pb = IT_PB(t, IT_HF(t)); \
                    _Pragma("unroll") for (int i = 0; i < 4; ++i) PLOAD(v[(B) * 4 + i], pb + i * PSTRIDE); \
                    KNOWN_VM0(); \
                } \
                SCHED(); \
                switch (t + tpre) { \
                    PRE_CASE(B) PRE_CASE(2 + B) PRE_CASE(4 + B) PRE_CASE(6 + B) PRE_CASE(8 + B) PRE_CASE(10 + B) PRE_CASE(12 + B) PRE_CASE(14 + B) \
                    default: _Pragma("unroll") for (int u = 0; u < 4; ++u) sum[u] = (f4){0.f, 0.f, 0.f, 0.f}; \
                } \
                hv = t + 2 >= nit || __all(fv >= call); \
                __builtin_amdgcn_fence(__ATOMIC_ACQUIRE, "wavefront"); \
                const unsigned nb = (hv && t + 2 < nit) ? IT_PB(t + 2, IT_HF(t + 2)) : OOB;     \
                fv = ld_agt(IT_FLAG(t + 4, IT_HF(t + 4))); \
                _Pragma("unroll") for (int c = 0; c < 4; ++c) { \
                    _Pragma("unroll") for (int u = 0; u < 4; ++u) sum[u] += v[(B) * 4 + c][u]; \
                    _Pragma("unroll") for (int u = 0; u < 4; ++u) asm volatile("" : "+v"(sum[u])); \
                    SCHED(); \
                    PLOAD(v[(B) * 4 + c], nb + c * PSTRIDE); \
                    OSTORE(c); \
                    SCHED(); \
                } \
                ACTIVATE(); \
                oprev = t < nit ? (unsigned)(rank * 1024 + ((hj + hs * t) & 63) * 16) * 4096u + (((((hj + hs * t) >> 6) ^ em) + pi) & 15) * 256u : OOB; \
                SCHED(); \
                ++t; \
            } while (0)
#pragma unroll
            for (int u = 240; u < 256; ++u) asm volatile("v_accvgpr_write_b32 a[%c0], 0" :: "i"(u));
#pragma unroll
            for (int j = 0; j < 4; ++j) outv[j] = (f4){0.f, 0.f, 0.f, 0.f};
            for (int t = 0; t < nit;) {
                ITEM(0, h0, f0);
                ITEM(1, h1, f1);
            }
#pragma unroll
            for (int j = 0; j < 4; ++j) OSTORE(j);
            WAIT_V(0);
            if (lane == 0) __hip_atomic_fetch_add((GAS unsigned*)((unsigned char*)A.wsp[dst] + WS_DONE + rank * 128), 1u, __ATOMIC_RELAXED, __HIP_MEMORY_SCOPE_SYSTEM);
        } else if (hlp && h == 127) {
            if (!wave_wait<true, 16>(ws + WS_DONE + ((rank + 1 + lane % 7) & (NR - 1)) * 128, call * (HWV + TWV), tmo, call) && lane == 0)
                ((float*)A.out[rank])[(size_t)rank * 1024 * 1024] = __builtin_nanf("");
        }
        KNOWN_VM0();
    }
    WAIT_V(0);
    __syncthreads();
    if (tid == 0) __hip_atomic_fetch_add((GAS unsigned*)(ws + WS_TICKET), 1u, __ATOMIC_RELAXED, __HIP_MEMORY_SCOPE_AGENT);
}

extern "C" void kernel_launch(int rank, int nranks, ncclComm_t comm, hipStream_t stream,
                              void* const* d_in, const int* in_sizes, int n_in,
                              void* d_out, int out_size, const int* in_dtypes, int out_dtype,
                              void* const* d_out_all, void* const* d_ws_all, size_t ws_bytes) {
    if (nranks != NR || (unsigned)rank >= (unsigned)NR || ws_bytes < WS_NEED || n_in < 2 || in_sizes[0] != M * K || in_sizes[1] != K * N || out_size != 8192 * 1024) {
        fprintf(stderr, "unexpected problem shape\n");
        return;
    }
    static bool attr = false;
    if (!attr) { HIP_CHECK(hipFuncSetAttribute((const void*)gemm_a2a, hipFuncAttributeMaxDynamicSharedMemorySize, LDS_BYTES)); attr = true; }
    Args a; a.x = (const float*)d_in[0]; a.w = (const float*)d_in[1]; a.ws = d_ws_all[rank]; a.rank = rank; a.pad = 0;
    for (int p = 0; p < NR; ++p) { a.out[p] = d_out_all[p]; a.wsp[p] = d_ws_all[p]; }
    hipLaunchKernelGGL(gemm_a2a, dim3(256), dim3(256), LDS_BYTES, stream, a);
    HIP_CHECK(hipGetLastError());
}
```

```cpp
#include <hip/hip_runtime.h>
#include <rccl/rccl.h>
#include <cstdio>

#define HIP_CHECK(expr) do { hipError_t _e = (expr); if (_e != hipSuccess) { \
    fprintf(stderr, "HIP error %s at %s:%d\n", hipGetErrorString(_e), __FILE__, __LINE__); \
    return; } } while (0)

typedef _Float16 h2 __attribute__((ext_vector_type(2)));
typedef float f4 __attribute__((ext_vector_type(4)));
typedef float f2 __attribute__((ext_vector_type(2)));
typedef unsigned u4 __attribute__((ext_vector_type(4)));
typedef unsigned u2 __attribute__((ext_vector_type(2)));
#define LAS __attribute__((address_space(3)))
#define GAS __attribute__((address_space(1)))

constexpr int NR = 8;
constexpr int M = 1024, N = 8192, K = 8192;
constexpr int XS = 16384;
constexpr int LDS_BYTES = 8 * XS;
constexpr unsigned WS_TMO = 128, WS_TICKET = 256, WS_START = 1024, WS_DONE = 2048;
constexpr unsigned WS_XF = 4096;
constexpr unsigned WS_PF = 16384;
constexpr size_t WS_XS = 1u << 20, WS_PART = 17u << 20;
constexpr unsigned PSTRIDE = 262144u;
constexpr unsigned PART_BYTES = 624u * PSTRIDE;
constexpr size_t WS_NEED = WS_PART + PART_BYTES;
constexpr unsigned OOB = 0x80000000u;
constexpr unsigned long long DEADLINE = 40000000ull;
constexpr int HWV = 18, TWV = 20, HB = 704;
constexpr int PACE0 = 420, PACED = 12, PACEF = 295, PACEN = 30;

#define ACLOB "a0","a1","a2","a3","a4","a5","a6","a7","a8","a9","a10","a11","a12","a13","a14","a15", \
    "a16","a17","a18","a19","a20","a21","a22","a23","a24","a25","a26","a27","a28","a29","a30","a31", \
    "a32","a33","a34","a35","a36","a37","a38","a39","a40","a41","a42","a43","a44","a45","a46","a47", \
    "a48","a49","a50","a51","a52","a53","a54","a55","a56","a57","a58","a59","a60","a61","a62","a63", \
    "a64","a65","a66","a67","a68","a69","a70","a71","a72","a73","a74","a75","a76","a77","a78","a79", \
    "a80","a81","a82","a83","a84","a85","a86","a87","a88","a89","a90","a91","a92","a93","a94","a95", \
    "a96","a97","a98","a99","a100","a101","a102","a103","a104","a105","a106","a107","a108","a109","a110","a111", \
    "a112","a113","a114","a115","a116","a117","a118","a119","a120","a121","a122","a123","a124","a125","a126","a127", \
    "a128","a129","a130","a131","a132","a133","a134","a135","a136","a137","a138","a139","a140","a141","a142","a143", \
    "a144","a145","a146","a147","a148","a149","a150","a151","a152","a153","a154","a155","a156","a157","a158","a159", \
    "a160","a161","a162","a163","a164","a165","a166","a167","a168","a169","a170","a171","a172","a173","a174","a175", \
    "a176","a177","a178","a179","a180","a181","a182","a183","a184","a185","a186","a187","a188","a189","a190","a191", \
    "a192","a193","a194","a195","a196","a197","a198","a199","a200","a201","a202","a203","a204","a205","a206","a207", \
    "a208","a209","a210","a211","a212","a213","a214","a215","a216","a217","a218","a219","a220","a221","a222","a223", \
    "a224","a225","a226","a227","a228","a229","a230","a231","a232","a233","a234","a235","a236","a237","a238","a239", \
    "a240","a241","a242","a243","a244","a245","a246","a247","a248","a249","a250","a251","a252","a253","a254","a255"
#define BAR() do { asm volatile("" ::: "memory"); __builtin_amdgcn_s_barrier(); asm volatile("" ::: "memory"); } while (0)
#define SCHED() __builtin_amdgcn_sched_barrier(0)
#define WAIT_V(n) asm volatile("s_waitcnt vmcnt(%0)" ::"n"(n) : "memory")
#define KNOWN_LGKM0() __builtin_amdgcn_s_waitcnt(0xC07F)
#define KNOWN_VM0() __builtin_amdgcn_s_waitcnt(0x0F70)

__device__ __forceinline__ unsigned cvt2(float a, float b) { f2 v = {a, b}; h2 h = __builtin_convertvector(v, h2); return __builtin_bit_cast(unsigned, h); }
__device__ __forceinline__ float silu(float y) {
    const float yc = __builtin_amdgcn_fmed3f(y, -60.f, 60.f);
    const float e = __builtin_amdgcn_exp2f(yc * -1.44269504088896f);
    return y * __builtin_amdgcn_rcpf(1.0f + e);
}
__device__ __forceinline__ unsigned ld_sys(const void* p) { return __hip_atomic_load((GAS unsigned*)p, __ATOMIC_RELAXED, __HIP_MEMORY_SCOPE_SYSTEM); }
__device__ __forceinline__ unsigned ld_agt(const void* p) { return __hip_atomic_load((GAS unsigned*)p, __ATOMIC_RELAXED, __HIP_MEMORY_SCOPE_AGENT); }
__device__ __forceinline__ void st_sys(void* p, unsigned v) { __hip_atomic_store((GAS unsigned*)p, v, __ATOMIC_RELAXED, __HIP_MEMORY_SCOPE_SYSTEM); }
__device__ __forceinline__ void st_agt(void* p, unsigned v) { __hip_atomic_store((GAS unsigned*)p, v, __ATOMIC_RELAXED, __HIP_MEMORY_SCOPE_AGENT); }
template <bool SYS, int NAP>
__device__ __forceinline__ bool wave_wait(const void* p, unsigned want, void* tmo, unsigned call) {
    const unsigned long long t0 = __builtin_amdgcn_s_memrealtime();
    for (unsigned it = 1;; ++it) {
        if (__all((SYS ? ld_sys(p) : ld_agt(p)) >= want)) break;
        if ((it & 31u) == 0u && (ld_agt(tmo) == call || __builtin_amdgcn_s_memrealtime() - t0 > DEADLINE)) { st_agt(tmo, call); return false; }
        __builtin_amdgcn_s_sleep(NAP);
    }
    __builtin_amdgcn_fence(__ATOMIC_ACQUIRE, "wavefront");
    return true;
}

struct Args { const float* x; const float* w; void* ws; void* out[NR]; void* wsp[NR]; int rank; int pad; };

__device__ __forceinline__ int column_of(int jb, int ni, int role) {
    return jb == 0 ? 6 - ni : jb == 1 ? (role ? 20 - ni : 7 + ni) : role ? 28 + ((ni + 1) & 3) : 21 + ni;
}
__device__ __forceinline__ int strip_of(int o, int wv, int rank) {
    const int q = o * 4 + wv, pi = q % 7, e = q / 7;
    return q < 112 ? ((rank + 1 + pi) & 7) * 16 + ((e + pi) & 15) : rank * 16 + (q - 112);
}
__device__ __forceinline__ unsigned part_slot(int o, int mt) { return (unsigned)(o < 7 ? o * 32 + mt * 8 : 112 + o * 16 + mt * 4) * PSTRIDE; }

constexpr int lpos(int j) { return 222 - (47 - j) * 4; }
constexpr int lat(int p) { for (int j = 0; j < 48; ++j) if (lpos(j) == p) return j; return -1; }
constexpr int dpos(int q) { return 141 + q * 13 / 4; }
constexpr int dat(int p) { for (int q = 0; q < 64; ++q) if (dpos(q) == p) return q; return -1; }
constexpr int dfirst() { int q = 0; while (dpos(q) < 256) ++q; return q; }
constexpr int vwait(int j, int p, int mode) {
    const int a = lpos(j) < p ? lpos(j) : lpos(j) - 256;
    int n = 0;
    for (int k = 0; k < 48; ++k) { if (lpos(k) > a && lpos(k) < p) ++n; if (lpos(k) - 256 > a) ++n; }
    for (int q = 0; q < 64; ++q) { const int d = dpos(q) - (mode == 1 ? 256 : 0); if (mode && d >= a && d < p) ++n; }
    return n < 63 ? n : 63;
}
constexpr int lafter(int p, int mode) { int n = mode == 2 ? dfirst() : 0; for (int k = 0; k < 48; ++k) if (lpos(k) > p) ++n; return n < 63 ? n : 63; }
constexpr bool sched_ok() {
    for (int q = 0; q < 64; ++q) if (dpos(q) < 128 + ((q >> 2) + 1) * 8 + 4 || dpos(q) >= 256 + (q >> 2) * 8 - 2) return false;
    for (int j = 0; j < 8; ++j) if (lpos(16 + j) < (j | 1) * 8 + 5 || lpos(24 + j) < 64 + (j | 1) * 8 + 5 || lpos(40 + j) < 128 + (j | 1) * 8 + 5) return false;
    return lpos(0) >= 32 && lpos(32) > 113 && lpos(47) < 256 && dpos(63) - 256 < lpos(15);
}
static_assert(sched_ok(), "schedule");

#define PLOAD(dst, soff) _Pragma("unroll") for (int t_ = 0; t_ < 4; ++t_) \
        dst[t_] = __builtin_bit_cast(f4, __builtin_amdgcn_raw_buffer_load_b128(rp, lane * 16 + t_ * 1024, (soff), 18))
#define OSTORE(j) __builtin_amdgcn_raw_buffer_store_b128(__builtin_bit_cast(u4, outv[j]), ro, voffo, oprev + (j) * 4096u, 17)
#define ACTIVATE() _Pragma("unroll") for (int j = 0; j < 4; ++j) { \
        outv[j] = (f4){silu(sum[0][j]), silu(sum[1][j]), silu(sum[2][j]), silu(sum[3][j])}; \
        if (bad) outv[j] = (f4){__builtin_nanf(""), __builtin_nanf(""), __builtin_nanf(""), __builtin_nanf("")}; }

__global__ __launch_bounds__(256, 1) void gemm_a2a(Args A) {
    extern __shared__ __attribute__((aligned(16))) unsigned char smem_[];
    LAS unsigned char* smem = (LAS unsigned char*)smem_;
    const int tid = threadIdx.x, lane = tid & 63, wid = __builtin_amdgcn_readfirstlane(tid >> 6);
    const int rank = A.rank & (NR - 1);
    unsigned char* ws = (unsigned char*)A.ws;
    const int xcd = blockIdx.x & 7, idx = blockIdx.x >> 3;
    const int mt = idx & 3, ni = idx >> 2, kq = xcd & 3, g = xcd >> 2;
    const __amdgpu_buffer_rsrc_t rx = __builtin_amdgcn_make_buffer_rsrc((void*)(ws + WS_XS), 0, (unsigned)(M * K * 2), 0x00020000);
    const __amdgpu_buffer_rsrc_t rp = __builtin_amdgcn_make_buffer_rsrc((void*)(ws + WS_PART), 0, PART_BYTES, 0x00020000);
    const unsigned lane16 = lane * 16;
    const unsigned voffo = (unsigned)(lane >> 4) * 16384u + (unsigned)(lane & 15) * 16u;
    const unsigned tk0 = ld_agt(ws + WS_TICKET);
    const __amdgpu_buffer_rsrc_t rxi = __builtin_amdgcn_make_buffer_rsrc((void*)A.x, 0, (unsigned)(M * K * 4), 0x00020000);
    const unsigned xvi = (unsigned)(lane >> 3) * (K * 4) + (lane & 7) * 16, xvo = ((lane >> 2) & 1) * XS + (lane & 3) * 256 + (lane >> 3) * 16;
    const unsigned xbi = (unsigned)(mt * 256 + wid * 64) * (K * 4) + kq * 8192, xbo = (unsigned)(mt * 256 + kq * 64) * XS + wid * 4096;
#define XLD(dst, pr) _Pragma("unroll") for (int f_ = 0; f_ < 8; ++f_) _Pragma("unroll") for (int e_ = 0; e_ < 2; ++e_) \
        dst[f_][e_] = __builtin_bit_cast(f4, __builtin_amdgcn_raw_buffer_load_b128(rxi, xvi + e_ * 128, xbi + f_ * (8 * K * 4) + (pr) * 256, 0))
#define XST(src, pr) _Pragma("unroll") for (int f_ = 0; f_ < 8; ++f_) { \
        const u4 o_ = {cvt2(src[f_][0][0], src[f_][0][1]), cvt2(src[f_][0][2], src[f_][0][3]), cvt2(src[f_][1][0], src[f_][1][1]), cvt2(src[f_][1][2], src[f_][1][3])}; \
        __builtin_amdgcn_raw_buffer_store_b128(o_, rx, xvo + (f_ >> 1) * 1024 + (f_ & 1) * 128, xbo + (pr) * (2 * XS), 16); }
#define XPR(p) (g ? 31 - (p) : (p))
#define XFW(pr) ((((pr) >> 1) * 4 + wid) * 8 + ((pr) & 1) * 4)
    {
        f4 v[8][2];
        XLD(v, XPR(ni));
        XST(v, XPR(ni));
    }
    const unsigned call = (__builtin_amdgcn_readfirstlane(tk0) >> 8) + 1u;
    unsigned char* xf = ws + WS_XF + (mt * 4 + kq) * 512;
    unsigned char* tmo = ws + WS_TMO;
    bool bad = false;
    const int role = g ^ (ni & 1);

    if (ni < 7) {
        const __amdgpu_buffer_rsrc_t rw = __builtin_amdgcn_make_buffer_rsrc((void*)A.w, 0, (unsigned)((size_t)K * N * 4), 0x00020000);
        const __amdgpu_buffer_rsrc_t rf = __builtin_amdgcn_make_buffer_rsrc((void*)(ws + WS_PF), 0, 65536u, 0x00020000);
        unsigned voffw[8];
#pragma unroll
        for (int j = 0; j < 8; ++j) { voffw[j] = (unsigned)((lane >> 4) * 4 + (j & 3) + 32 * (j >> 2)) * (N * 4) + (unsigned)(lane & 15) * 16; asm volatile("" : "+v"(voffw[j])); }
        const unsigned voffx = wid * 4096 + lane16;
        const unsigned ldsx = (unsigned)(size_t)smem + wid * 4096;
        const int e0 = role ? 15 - ni : ni + 1, e1 = e0 + 16, nwin = (role && ni < 3) ? e1 : e0 + 32;
        const int c0 = strip_of(column_of(0, ni, role), wid, rank) * 256, c1 = strip_of(column_of(1, ni, role), wid, rank) * 256,
                  c2 = strip_of(column_of(2, ni, role), wid, rank) * 256;
        unsigned wrel[4], xrel[4];
#pragma unroll
        for (int s = 0; s < 4; ++s) { const int st = ((g ? 3 - s : s) + mt) & 3; wrel[s] = (unsigned)((st >> 1) * 64 + (st & 1) * 16) * (N * 4); xrel[s] = (unsigned)(mt * 256 + st) * XS; }
#define WIN(i) (kq * 16 + (g ? 15 - (i) : (i)))
#define PIN(x) asm volatile("" : "+s"(x))
#define GLDS(so, i) asm volatile("buffer_load_dwordx4 %0, %1, %2 offen offset:%c3 sc1 lds" :: "v"(voffx), "s"(rx), "s"(so), "i"((i) * 1024) : "memory")
#define WLD(rr, h, j, so) asm volatile("buffer_load_dwordx4 %0, %1, %2, %3 offen" : "=v"(rr[h][j]) : "v"(voffw[j]), "s"(rw), "s"(so) : "memory")
#define CVT(dst, rr, h, t, q) do { unsigned c_; asm volatile("v_cvt_pk_f16_f32 %0, %1, %2" : "=v"(c_) : "v"(rr[h][2 * (q)][t]), "v"(rr[h][2 * (q) + 1][t])); dst[h][t][q] = c_; } while (0)
#define MFMA(m, t, av, bv) asm volatile("v_mfma_f32_16x16x32_f16 a[%c2:%c3], %0, %1, a[%c2:%c3]" :: "v"(av), "v"(bv), "i"(((m) * 4 + (t)) * 4), "i"(((m) * 4 + (t)) * 4 + 3))
#define MFMA0(m, t, av, bv) asm volatile("v_mfma_f32_16x16x32_f16 a[%c2:%c3], %0, %1, 0" :: "v"(av), "v"(bv), "i"(((m) * 4 + (t)) * 4), "i"(((m) * 4 + (t)) * 4 + 3))
#define LDA(dst, aaddr, off) dst = *(const LAS u4*)(smem + (aaddr) + (off))
        u4 a[4][2], b0[2][4], b1[2][4], r0[2][8], r1[2][8];
        u2 xv; const unsigned lane8 = lane * 8;
        asm volatile("" ::: ACLOB);
        unsigned wb1 = ((unsigned)WIN(1) << 22) + (unsigned)(e0 > 1 ? c0 : c1), xb1 = (unsigned)WIN(1) << 16;
        {
            const unsigned wb0 = ((unsigned)WIN(0) << 22) + (unsigned)c0, xb0 = (unsigned)WIN(0) << 16;
#pragma unroll
            for (int j = 0; j < 8; ++j) WLD(r0, 0, j, wb0 + wrel[0]);
            WAIT_V(8);
            if (lane == 0) st_agt(xf + XFW(XPR(ni)), call);
#pragma unroll
            for (int j = 0; j < 8; ++j) WLD(r0, 1, j, wb0 + wrel[1]);
#pragma unroll
            for (int h = 0; h < 2; ++h)
#pragma unroll
                for (int j = 0; j < 8; ++j) WLD(r1, h, j, wb0 + wrel[2 + h]);
            if (!wave_wait<false, 2>(xf + (g ? 448 : 0) + (lane & 2) * 16 + wid * 8 + (lane & 1) * 4, call, tmo, call)) bad = true;
            KNOWN_VM0();
            WAIT_V(0);
#pragma unroll
            for (int s = 0; s < 4; ++s) {
                const unsigned so = xb0 + xrel[s];
                asm volatile("s_mov_b32 m0, %0\n\ts_nop 0" :: "s"(ldsx + s * XS) : "memory");
                GLDS(so, 0); GLDS(so, 1); GLDS(so, 2); GLDS(so, 3);
            }
            SCHED();
#pragma unroll
            for (int h = 0; h < 2; ++h)
#pragma unroll
                for (int q = 0; q < 4; ++q)
#pragma unroll
                    for (int t = 0; t < 4; ++t) CVT(b0, r0, h, t, q);
            SCHED();
#pragma unroll
            for (int j = 0; j < 8; ++j) WLD(r0, 0, j, wb1 + wrel[0]);
            KNOWN_LGKM0();
            WAIT_V(8); BAR();
#pragma unroll
            for (int m = 0; m < 2; ++m) { LDA(a[m][0], lane16, m * 1024); LDA(a[m][1], lane16, XS + m * 1024); }
        }
#define DUMP(q) asm volatile("buffer_store_dwordx4 a[%c3:%c4], %0, %1, %2 offen offset:%c5 sc1" :: "v"(lane16), "s"(rp), "s"(ob + (((q) & 63) >> 2) * 4096u), \
            "i"(((q) & 63) * 4), "i"(((q) & 63) * 4 + 3), "i"(((q) & 3) * 1024) : "memory")
#define DUMPC(q) asm volatile("s_bitcmp1_b32 %2, 31\n\ts_cbranch_scc1 1f\n\tbuffer_store_dwordx4 a[%c3:%c4], %0, %1, %2 offen offset:%c5 sc1\n1:" \
            :: "v"(lane16), "s"(rp), "s"(ob + (((q) & 63) >> 2) * 4096u), "i"(((q) & 63) * 4), "i"(((q) & 63) * 4 + 3), "i"(((q) & 3) * 1024) : "memory", "scc")
#define PUBLISH() asm volatile("buffer_store_dword %0, %1, %2, %3 offen sc1" :: "v"(callv), "v"(zerov), "s"(rf), "s"(pend) : "memory")
#define GAP(MODE, B, m, e) do { constexpr int p_ = (B) * 128 + (m) * 8 + (e), j_ = lat(p_), jn_ = lat(p_ + 1), q_ = dat(p_ + ((MODE) == 1 ? 256 : 0)); \
            if (j_ >= 40) WLD(r0, 0, j_ & 7, so); else if (j_ >= 32) GLDS(so, j_ & 3); \
            else if (j_ >= 24) WLD(r1, 1, j_ & 7, so); else if (j_ >= 16) WLD(r1, 0, j_ & 7, so); \
            else if (j_ >= 8) GLDS(so, j_ & 3); else if (j_ >= 0) WLD(r0, 1, j_ & 7, so); \
            if (jn_ == 0 || jn_ == 16 || jn_ == 24) { so = wb1 + wrel[jn_ == 0 ? 1 : jn_ >> 3]; PIN(so); } \
            if (jn_ == 40) { so = wb2 + wrel[0]; PIN(so); } \
            if (jn_ == 8 || jn_ == 12 || jn_ == 32 || jn_ == 36) { so = xb1 + xrel[(jn_ >= 32 ? 2 : 0) + ((jn_ >> 2) & 1)]; PIN(so); \
                asm volatile("s_add_u32 m0, %0, %1" :: "s"(lds2), "i"(((jn_ >= 32 ? 2 : 0) + ((jn_ >> 2) & 1)) * XS) : "memory", "scc"); } \
              \
            if (p_ == 2) { t2 = i + 2; PIN(t2); } \
            if (p_ == 5) { cc = t2 >= e0 ? c1 : c0; PIN(cc); } \
            if (p_ == 6) { cc = t2 >= e1 ? c2 : cc; PIN(cc); } \
            if (p_ == 14) { kp = (kp + kstep) & 15; PIN(kp); } \
            if (p_ == 18) { kw = kp | (kq * 16); PIN(kw); } \
            if (p_ == 21) { wb2 = ((unsigned)kw << 22) + (unsigned)cc; PIN(wb2); } \
            if (p_ == 22) { xb2 = (unsigned)kw << 16; PIN(xb2); } \
            if (p_ == 26) { lds2 ^= 4 * XS; PIN(lds2); } \
            if (p_ == 109 || p_ == 237) { an ^= 4 * XS; asm volatile("" : "+v"(an)); } \
            if ((MODE) == 1 && q_ >= 0) DUMPC(q_); else if ((MODE) == 2 && q_ >= 0) DUMP(q_); \
            SCHED(); } while (0)
#define ITER(MODE, B, bc, bn, rc, m, ac) do { \
            constexpr bool FIRST = (MODE) == 1; \
            constexpr int t0 = (m & 1) ? 3 : 0, dt = (m & 1) ? -1 : 1, hh = m >> 3, qq = (m >> 1) & 3; \
            if (m == 14) { WAIT_V(vwait((B) ? 15 : 39, (B) * 128 + 112, MODE)); BAR(); } \
            if (FIRST) MFMA0(m, t0, a[m & 3][0], bc[0][t0]); else MFMA(m, t0, a[m & 3][0], bc[0][t0]); \
            if (m < 14) LDA(a[(m + 2) & 3][0], ac, (m + 2) * 1024); else LDA(a[(m + 2) & 3][0], an, (m - 14) * 1024); \
            GAP(MODE, B, m, 0); \
            MFMA(m, t0, a[m & 3][1], bc[1][t0]); \
            if (m == 0) WAIT_V(vwait((B) ? 47 : 23, (B) * 128 + 1, MODE)); \
            if (m == 8) WAIT_V(vwait((B) ? 7 : 31, (B) * 128 + 65, MODE)); \
            CVT(bn, rc, hh, (m & 1) * 2, qq); \
            GAP(MODE, B, m, 1); \
            if (FIRST) MFMA0(m, t0 + dt, a[m & 3][1], bc[1][t0 + dt]); else MFMA(m, t0 + dt, a[m & 3][1], bc[1][t0 + dt]); \
            if ((B) && m == 14) PUBLISH(); \
            GAP(MODE, B, m, 2); \
            MFMA(m, t0 + dt, a[m & 3][0], bc[0][t0 + dt]); \
            if (m < 14) LDA(a[(m + 2) & 3][1], ac, XS + (m + 2) * 1024); else LDA(a[(m + 2) & 3][1], an, XS + (m - 14) * 1024); \
            GAP(MODE, B, m, 3); \
            if (FIRST) MFMA0(m, t0 + 2 * dt, a[m & 3][0], bc[0][t0 + 2 * dt]); else MFMA(m, t0 + 2 * dt, a[m & 3][0], bc[0][t0 + 2 * dt]); \
            CVT(bn, rc, hh, (m & 1) * 2 + 1, qq); \
            GAP(MODE, B, m, 4); \
            if ((B) && m == 0) asm volatile("global_load_dwordx2 %0, %1, %2 sc1" : "=v"(xv) : "v"(lane8), "s"(xf) : "memory"); \
            MFMA(m, t0 + 2 * dt, a[m & 3][1], bc[1][t0 + 2 * dt]); \
            GAP(MODE, B, m, 5); \
            if (FIRST) MFMA0(m, t0 + 3 * dt, a[m & 3][1], bc[1][t0 + 3 * dt]); else MFMA(m, t0 + 3 * dt, a[m & 3][1], bc[1][t0 + 3 * dt]); \
            GAP(MODE, B, m, 6); \
            MFMA(m, t0 + 3 * dt, a[m & 3][0], bc[0][t0 + 3 * dt]); \
            GAP(MODE, B, m, 7); \
        } while (0)
#define STEP(MODE, B, bc, bn, rc, ac, an_) do { unsigned& an = an_; \
        ITER(MODE, B, bc, bn, rc, 0, ac); ITER(MODE, B, bc, bn, rc, 1, ac); ITER(MODE, B, bc, bn, rc, 2, ac); ITER(MODE, B, bc, bn, rc, 3, ac); \
        ITER(MODE, B, bc, bn, rc, 4, ac); ITER(MODE, B, bc, bn, rc, 5, ac); ITER(MODE, B, bc, bn, rc, 6, ac); ITER(MODE, B, bc, bn, rc, 7, ac); \
        ITER(MODE, B, bc, bn, rc, 8, ac); ITER(MODE, B, bc, bn, rc, 9, ac); ITER(MODE, B, bc, bn, rc, 10, ac); ITER(MODE, B, bc, bn, rc, 11, ac); \
        ITER(MODE, B, bc, bn, rc, 12, ac); ITER(MODE, B, bc, bn, rc, 13, ac); ITER(MODE, B, bc, bn, rc, 14, ac); ITER(MODE, B, bc, bn, rc, 15, ac); \
    } while (0)
        int i = 0;
        unsigned pend = OOB, ob = OOB;
        unsigned so = 0, wb2 = 0, xb2 = 0, lds2 = ldsx, aA = lane16, aB = lane16 + 6 * XS, callv = call, zerov = 0;
        unsigned tprev = (unsigned)__builtin_amdgcn_s_memrealtime();
        KNOWN_LGKM0();
        int t2 = 0, cc = 0, kw = 0, kp = WIN(1) & 15;
        const int kstep = g ? -1 : 1;
        asm volatile("" : "+v"(callv), "+v"(zerov));
#define STEP_A(MODE) STEP(MODE, 0, b0, b1, r1, aA, aB)
#define STEP_B(MODE) do { STEP(MODE, 1, b1, b0, r0, aB, aA); \
            pend = OOB; wb1 = wb2; xb1 = xb2; ++i; \
            if (i <= PACEN) { const int pc_ = PACE0 - i * PACED; const unsigned due = tprev + (pc_ > PACEF ? pc_ : PACEF); \
                while ((int)((tprev = (unsigned)__builtin_amdgcn_s_memrealtime()) - due) < 0) __builtin_amdgcn_s_sleep(1); \
                KNOWN_LGKM0(); } \
            asm volatile("s_waitcnt vmcnt(%1)" : "+v"(xv) : "n"(lafter(132, MODE)) : "memory"); \
            if (i < 15) {     \
                const int pw = g ? 14 - i : i + 1; \
                if (((__ballot(xv[0] >= call && xv[1] >= call) >> (pw * 4 + wid)) & 1) == 0) { \
                    if (!wave_wait<false, 4>(xf + pw * 32 + wid * 8 + (lane & 1) * 4, call, tmo, call)) bad = true; \
                    KNOWN_LGKM0(); } \
                SCHED(); \
            } } while (0)
        for (int jb = 0; i < nwin; ++jb) {
            const int iend = jb == 0 ? e0 : jb == 1 ? e1 : nwin;
            STEP_A(1);
            while (i + 1 < iend) { STEP_B(0); STEP_A(0); }
            const int o = column_of(jb, ni, role), pslot = jb == 0 ? role * 4 + kq : kq;
            ob = part_slot(o, mt) + pslot * PSTRIDE + wid * 65536u;
            STEP_B(2);
            pend = bad ? OOB : (unsigned)(((o * 4 + mt) * 4 + wid) * 128 + pslot * 4);
        }
        asm volatile("s_nop 15"); SCHED();
#pragma unroll
        for (int q = dfirst(); q < 64; ++q) DUMP(q);
        WAIT_V(0);
        PUBLISH();
        if (role && ni >= 3) {
            const int o = column_of(2, ni, 1);
            const __amdgpu_buffer_rsrc_t ro = __builtin_amdgcn_make_buffer_rsrc(A.out[rank], 0, 8192u * 1024 * 4, 0x00020000);
            if (!wave_wait<false, 4>(ws + WS_PF + ((o * 4 + mt) * 4 + wid) * 128 + (lane & 3) * 4, call, tmo, call)) bad = true;
            const unsigned pb = part_slot(o, mt) + wid * 65536u + kq * 16384u;
            f4 v[4][4], sum[4], outv[4];
#pragma unroll
            for (int c = 0; c < 4; ++c) { PLOAD(v[c], pb + c * PSTRIDE); outv[c] = (f4){0.f, 0.f, 0.f, 0.f}; }
            unsigned oprev = OOB;
#pragma nounroll
            for (int b = 0; b < 4; ++b) {
                const unsigned nx = b < 3 ? pb + (b + 1) * 4096u : OOB;
#pragma unroll
                for (int c = 1; c < 4; ++c) {
#pragma unroll
                    for (int u = 0; u < 4; ++u) { if (c == 1) sum[u] = v[0][u] + v[1][u]; else sum[u] += v[c][u]; }
#pragma unroll
                    for (int u = 0; u < 4; ++u) asm volatile("" : "+v"(sum[u]));
                    SCHED();
                    if (c == 1) PLOAD(v[0], nx);
                    PLOAD(v[c], nx + c * PSTRIDE);
                    if (c == 1) OSTORE(0);
                    OSTORE(c);
                    SCHED();
                }
                ACTIVATE();
                oprev = (unsigned)(rank * 1024 + mt * 256 + kq * 64 + b * 16) * 4096u + ((o - 28) * 4 + wid) * 256u;
                SCHED();
            }
#pragma unroll
            for (int j = 0; j < 4; ++j) OSTORE(j);
        }
    } else {
        WAIT_V(0);
        if (lane == 0) st_agt(xf + XFW(XPR(ni)), call);
        if (blockIdx.x == 224 && wid == 0 && lane < NR) st_sys((unsigned char*)A.wsp[lane & (NR - 1)] + WS_START + rank * 128, call);
        f4 u0[8][2], u1[8][2];
        XLD(u0, XPR(8)); XLD(u1, XPR(9));
#pragma nounroll
        for (int p = 8; p < 14; p += 2) {
            XST(u0, XPR(p)); SCHED(); XLD(u0, XPR(p + 2)); SCHED();
            XST(u1, XPR(p + 1)); SCHED(); XLD(u1, XPR(p + 3)); SCHED();
            WAIT_V(16);
            if (lane < 2) st_agt(xf + XFW(XPR(p) & ~1) + lane * 4, call);
        }
        XST(u0, XPR(14)); XST(u1, XPR(15));
        WAIT_V(0);
        if (lane < 2) st_agt(xf + XFW(XPR(14) & ~1) + lane * 4, call);
        KNOWN_VM0();
    }
    if (ni == 7 || (role && ni < 3)) {
        const bool hlp = ni == 7;
        const int h = hlp ? wid * 32 + xcd * 4 + mt : wid * 48 + (ni * 4 + kq) * 4 + mt, pi = h % 7, hs = hlp ? HWV : TWV;
        const int hj = h / 7 + (hlp ? 0 : HB), dst = (rank + 1 + pi) & (NR - 1);
        if (h < 7 * hs) {
            if (!wave_wait<true, 8>(ws + WS_START + dst * 128, call, tmo, call)) bad = true;
            const __amdgpu_buffer_rsrc_t ro = __builtin_amdgcn_make_buffer_rsrc(A.out[dst], 0, 8192u * 1024 * 4, 0x00020000);
#define IT_Q(t) (pi + 7 * ((hj + hs * (t)) >> 6))
#define IT_MT(t) (((hj + hs * (t)) >> 4) & 3)
#define IT_FLAG(t, hf) (ws + WS_PF + (((IT_Q(t) >> 2) * 4 + IT_MT(t)) * 4 + (IT_Q(t) & 3)) * 128 + ((hf) * 4 + (lane & 3)) * 4)
#define IT_HF(t) (IT_Q(t) < 28 ? 1 : 0)
#define IT_PB(t, hf) (part_slot(IT_Q(t) >> 2, IT_MT(t)) + (hf) * (4 * PSTRIDE) + (IT_Q(t) & 3) * 65536u + ((hj + hs * (t)) & 15) * 4096u)
            f4 v[8][4], sum[4], outv[4];
            asm volatile("" ::: ACLOB);
            if (hlp) {
#pragma unroll
                for (int t = 14; t >= 0; t -= 2) {
                    const bool va = hj + HWV * t < 256, vb = t > 0;
                    if (va && !wave_wait<false, 24>(IT_FLAG(t, 0), call, tmo, call)) bad = true;
                    const unsigned pa = va ? IT_PB(t, 0) : OOB;
#pragma unroll
                    for (int i = 0; i < 4; ++i) PLOAD(v[i], pa + i * PSTRIDE);
                    if (vb) {
                        if (!wave_wait<false, 24>(IT_FLAG(t - 1, 0), call, tmo, call)) bad = true;
#pragma unroll
                        for (int i = 0; i < 4; ++i) PLOAD(v[4 + i], IT_PB(t - 1, 0) + i * PSTRIDE);
                    }
                    SCHED();
#pragma unroll
                    for (int b = 0; b < (vb ? 2 : 1); ++b)
#pragma unroll
                        for (int u = 0; u < 4; ++u) {
                            const f4 x = (v[b * 4][u] + v[b * 4 + 1][u]) + (v[b * 4 + 2][u] + v[b * 4 + 3][u]);
#pragma unroll
                            for (int j = 0; j < 4; ++j) asm volatile("v_accvgpr_write_b32 a[%c1], %0" :: "v"(x[j]), "i"((t - b) * 16 + u * 4 + j));
                        }
                    SCHED();
                }
            }
            const int nit = ((hlp ? HB : 1024) - hj + hs - 1) / hs, tpre = hlp ? 0 : 64;
            bool h0 = false, h1 = false;
            unsigned f0 = 0, f1 = 0;
            unsigned oprev = OOB;
#define PRE_CASE(T) case T: _Pragma("unroll") for (int u = 0; u < 16; ++u) { float f; asm volatile("v_accvgpr_read_b32 %0, a[%c1]" : "=v"(f) : "i"((T) * 16 + u)); sum[u >> 2][u & 3] = f; } break;
#define ITEM(B, hv, fv) do { \
                if (!hv) { \
                    _Pragma("unroll") for (int j = 0; j < 4; ++j) OSTORE(j); \
                    oprev = OOB; \
                    if (!wave_wait<false, 24>(IT_FLAG(t, IT_HF(t)), call, tmo, call)) bad = true; \
                    fv = ld_agt(IT_FLAG(t + 2, IT_HF(t + 2))); \
                    const unsigned pb = IT_PB(t, IT_HF(t)); \
                    _Pragma("unroll") for (int i = 0; i < 4; ++i) PLOAD(v[(B) * 4 + i], pb + i * PSTRIDE); \
                    KNOWN_VM0(); \
                } \
                SCHED(); \
                switch (t + tpre) { \
                    PRE_CASE(B) PRE_CASE(2 + B) PRE_CASE(4 + B) PRE_CASE(6 + B) PRE_CASE(8 + B) PRE_CASE(10 + B) PRE_CASE(12 + B) PRE_CASE(14 + B) \
                    default: _Pragma("unroll") for (int u = 0; u < 4; ++u) sum[u] = (f4){0.f, 0.f, 0.f, 0.f}; \
                } \
                hv = t + 2 >= nit || __all(fv >= call); \
                __builtin_amdgcn_fence(__ATOMIC_ACQUIRE, "wavefront"); \
                const unsigned nb = (hv && t + 2 < nit) ? IT_PB(t + 2, IT_HF(t + 2)) : OOB;     \
                fv = ld_agt(IT_FLAG(t + 4, IT_HF(t + 4))); \
                _Pragma("unroll") for (int c = 0; c < 4; ++c) { \
                    _Pragma("unroll") for (int u = 0; u < 4; ++u) sum[u] += v[(B) * 4 + c][u]; \
                    _Pragma("unroll") for (int u = 0; u < 4; ++u) asm volatile("" : "+v"(sum[u])); \
                    SCHED(); \
                    PLOAD(v[(B) * 4 + c], nb + c * PSTRIDE); \
                    OSTORE(c); \
                    SCHED(); \
                } \
                ACTIVATE(); \
                oprev = t < nit ? (unsigned)(rank * 1024 + ((hj + hs * t) & 63) * 16) * 4096u + ((((hj + hs * t) >> 6) + pi) & 15) * 256u : OOB; \
                SCHED(); \
                ++t; \
            } while (0)
#pragma unroll
            for (int u = 240; u < 256; ++u) asm volatile("v_accvgpr_write_b32 a[%c0], 0" :: "i"(u));
#pragma unroll
            for (int j = 0; j < 4; ++j) outv[j] = (f4){0.f, 0.f, 0.f, 0.f};
            for (int t = 0; t < nit;) {
                ITEM(0, h0, f0);
                ITEM(1, h1, f1);
            }
#pragma unroll
            for (int j = 0; j < 4; ++j) OSTORE(j);
            WAIT_V(0);
            if (lane == 0) __hip_atomic_fetch_add((GAS unsigned*)((unsigned char*)A.wsp[dst] + WS_DONE + rank * 128), 1u, __ATOMIC_RELAXED, __HIP_MEMORY_SCOPE_SYSTEM);
        } else if (hlp && h == 127) {
            if (!wave_wait<true, 16>(ws + WS_DONE + ((rank + 1 + lane % 7) & (NR - 1)) * 128, call * (HWV + TWV), tmo, call) && lane == 0)
                ((float*)A.out[rank])[(size_t)rank * 1024 * 1024] = __builtin_nanf("");
        }
        KNOWN_VM0();
    }
    WAIT_V(0);
    __syncthreads();
    if (tid == 0) __hip_atomic_fetch_add((GAS unsigned*)(ws + WS_TICKET), 1u, __ATOMIC_RELAXED, __HIP_MEMORY_SCOPE_AGENT);
}

extern "C" void kernel_launch(int rank, int nranks, ncclComm_t comm, hipStream_t stream,
                              void* const* d_in, const int* in_sizes, int n_in,
                              void* d_out, int out_size, const int* in_dtypes, int out_dtype,
                              void* const* d_out_all, void* const* d_ws_all, size_t ws_bytes) {
    if (nranks != NR || (unsigned)rank >= (unsigned)NR || ws_bytes < WS_NEED || n_in < 2 || in_sizes[0] != M * K || in_sizes[1] != K * N || out_size != 8192 * 1024) {
        fprintf(stderr, "unexpected problem shape\n");
        return;
    }
    static bool attr = false;
    if (!attr) { HIP_CHECK(hipFuncSetAttribute((const void*)gemm_a2a, hipFuncAttributeMaxDynamicSharedMemorySize, LDS_BYTES)); attr = true; }
    Args a; a.x = (const float*)d_in[0]; a.w = (const float*)d_in[1]; a.ws = d_ws_all[rank]; a.rank = rank; a.pad = 0;
    for (int p = 0; p < NR; ++p) { a.out[p] = d_out_all[p]; a.wsp[p] = d_ws_all[p]; }
    hipLaunchKernelGGL(gemm_a2a, dim3(256), dim3(256), LDS_BYTES, stream, a);
    HIP_CHECK(hipGetLastError());
}
```

```cpp
#include <hip/hip_runtime.h>
#include <rccl/rccl.h>
#include <cstdio>

#define HIP_CHECK(expr) do { hipError_t _e = (expr); if (_e != hipSuccess) { \
    fprintf(stderr, "HIP error %s at %s:%d\n", hipGetErrorString(_e), __FILE__, __LINE__); \
    return; } } while (0)

typedef _Float16 h2 __attribute__((ext_vector_type(2)));
typedef float f4 __attribute__((ext_vector_type(4)));
typedef float f2 __attribute__((ext_vector_type(2)));
typedef unsigned u4 __attribute__((ext_vector_type(4)));
typedef unsigned u2 __attribute__((ext_vector_type(2)));
#define LAS __attribute__((address_space(3)))
#define GAS __attribute__((address_space(1)))

constexpr int NR = 8;
constexpr int M = 1024, N = 8192, K = 8192;
constexpr int XS = 16384;
constexpr int LDS_BYTES = 8 * XS;
constexpr unsigned WS_TMO = 128, WS_TICKET = 256, WS_START = 1024, WS_DONE = 2048;
constexpr unsigned WS_XF = 4096;
constexpr unsigned WS_PF = 16384;
constexpr size_t WS_XS = 1u << 20, WS_PART = 17u << 20;
constexpr unsigned PSTRIDE = 262144u;
constexpr unsigned PART_BYTES = 624u * PSTRIDE;
constexpr size_t WS_NEED = WS_PART + PART_BYTES;
constexpr unsigned OOB = 0x80000000u;
constexpr unsigned long long DEADLINE = 40000000ull;
constexpr int HWV = 18, TWV = 13, HB = 672;
constexpr int NTAB = 0x7665321;
constexpr int PACE0 = 420, PACED = 12, PACEF = 295, PACEN = 30;

#define ACLOB "a0","a1","a2","a3","a4","a5","a6","a7","a8","a9","a10","a11","a12","a13","a14","a15", \
    "a16","a17","a18","a19","a20","a21","a22","a23","a24","a25","a26","a27","a28","a29","a30","a31", \
    "a32","a33","a34","a35","a36","a37","a38","a39","a40","a41","a42","a43","a44","a45","a46","a47", \
    "a48","a49","a50","a51","a52","a53","a54","a55","a56","a57","a58","a59","a60","a61","a62","a63", \
    "a64","a65","a66","a67","a68","a69","a70","a71","a72","a73","a74","a75","a76","a77","a78","a79", \
    "a80","a81","a82","a83","a84","a85","a86","a87","a88","a89","a90","a91","a92","a93","a94","a95", \
    "a96","a97","a98","a99","a100","a101","a102","a103","a104","a105","a106","a107","a108","a109","a110","a111", \
    "a112","a113","a114","a115","a116","a117","a118","a119","a120","a121","a122","a123","a124","a125","a126","a127", \
    "a128","a129","a130","a131","a132","a133","a134","a135","a136","a137","a138","a139","a140","a141","a142","a143", \
    "a144","a145","a146","a147","a148","a149","a150","a151","a152","a153","a154","a155","a156","a157","a158","a159", \
    "a160","a161","a162","a163","a164","a165","a166","a167","a168","a169","a170","a171","a172","a173","a174","a175", \
    "a176","a177","a178","a179","a180","a181","a182","a183","a184","a185","a186","a187","a188","a189","a190","a191", \
    "a192","a193","a194","a195","a196","a197","a198","a199","a200","a201","a202","a203","a204","a205","a206","a207", \
    "a208","a209","a210","a211","a212","a213","a214","a215","a216","a217","a218","a219","a220","a221","a222","a223", \
    "a224","a225","a226","a227","a228","a229","a230","a231","a232","a233","a234","a235","a236","a237","a238","a239", \
    "a240","a241","a242","a243","a244","a245","a246","a247","a248","a249","a250","a251","a252","a253","a254","a255"
#define BAR() do { asm volatile("" ::: "memory"); __builtin_amdgcn_s_barrier(); asm volatile("" ::: "memory"); } while (0)
#define SCHED() __builtin_amdgcn_sched_barrier(0)
#define WAIT_V(n) asm volatile("s_waitcnt vmcnt(%0)" ::"n"(n) : "memory")
#define KNOWN_LGKM0() __builtin_amdgcn_s_waitcnt(0xC07F)
#define KNOWN_VM0() __builtin_amdgcn_s_waitcnt(0x0F70)

__device__ __forceinline__ unsigned cvt2(float a, float b) { f2 v = {a, b}; h2 h = __builtin_convertvector(v, h2); return __builtin_bit_cast(unsigned, h); }
__device__ __forceinline__ float silu(float y) {
    const float yc = __builtin_amdgcn_fmed3f(y, -60.f, 60.f);
    const float e = __builtin_amdgcn_exp2f(yc * -1.44269504088896f);
    return y * __builtin_amdgcn_rcpf(1.0f + e);
}
__device__ __forceinline__ unsigned ld_sys(const void* p) { return __hip_atomic_load((GAS unsigned*)p, __ATOMIC_RELAXED, __HIP_MEMORY_SCOPE_SYSTEM); }
__device__ __forceinline__ unsigned ld_agt(const void* p) { return __hip_atomic_load((GAS unsigned*)p, __ATOMIC_RELAXED, __HIP_MEMORY_SCOPE_AGENT); }
__device__ __forceinline__ void st_sys(void* p, unsigned v) { __hip_atomic_store((GAS unsigned*)p, v, __ATOMIC_RELAXED, __HIP_MEMORY_SCOPE_SYSTEM); }
__device__ __forceinline__ void st_agt(void* p, unsigned v) { __hip_atomic_store((GAS unsigned*)p, v, __ATOMIC_RELAXED, __HIP_MEMORY_SCOPE_AGENT); }
template <bool SYS, int NAP, bool WB = false>
__device__ __forceinline__ bool wave_wait(const void* p, unsigned want, void* tmo, unsigned call) {
    const unsigned long long t0 = __builtin_amdgcn_s_memrealtime();
    for (unsigned it = 1;; ++it) {
        if (__all((SYS ? ld_sys(p) : ld_agt(p)) >= want)) break;
        if ((it & 31u) == 0u && (ld_agt(tmo) == call || __builtin_amdgcn_s_memrealtime() - t0 > DEADLINE)) { st_agt(tmo, call); return false; }
        __builtin_amdgcn_s_sleep(NAP);
        if (WB) BAR();
    }
    __builtin_amdgcn_fence(__ATOMIC_ACQUIRE, "wavefront");
    return true;
}

struct Args { const float* x; const float* w; void* ws; void* out[NR]; void* wsp[NR]; int rank; int pad; };

__device__ __forceinline__ int column_of(int jb, int ni, int role) {
    return jb == 0 ? 6 - ni : jb == 1 ? (role ? 20 - ni : 7 + ni) : role ? 28 + ((ni + 1) & 3) : 21 + ni;
}
__device__ __forceinline__ int strip_of(int o, int wv, int rank) {
    const int q = o * 4 + wv, pi = q % 7, e = q / 7;
    return q < 112 ? ((rank + 1 + pi) & 7) * 16 + ((e + pi) & 15) : rank * 16 + (q - 112);
}
__device__ __forceinline__ unsigned part_slot(int o, int mt) { return (unsigned)(o < 7 ? o * 32 + mt * 8 : 112 + o * 16 + mt * 4) * PSTRIDE; }

constexpr int lpos(int j) { return 222 - (47 - j) * 4; }
constexpr int lat(int p) { for (int j = 0; j < 48; ++j) if (lpos(j) == p) return j; return -1; }
constexpr int dpos(int q) { return 141 + q * 11 / 4; }
constexpr int dat(int p) { for (int q = 0; q < 64; ++q) if (dpos(q) == p) return q; return -1; }
constexpr int dfirst() { int q = 0; while (dpos(q) < 256) ++q; return q; }
constexpr int vwait(int j, int p, int mode) {
    const int a = lpos(j) < p ? lpos(j) : lpos(j) - 256;
    int n = 0;
    for (int k = 0; k < 48; ++k) { if (lpos(k) > a && lpos(k) < p) ++n; if (lpos(k) - 256 > a) ++n; }
    for (int q = 0; q < 64; ++q) { const int d = dpos(q) - (mode == 1 ? 256 : 0); if (mode && d >= a && d < p) ++n; }
    return n < 63 ? n : 63;
}
constexpr int dwait(int p) { int n = 0; for (int k = 0; k < 48; ++k) if (lpos(k) > dpos(63) - 256 && lpos(k) < p) ++n; return n; }
constexpr int lafter(int p, int mode) { int n = mode == 2 ? dfirst() : 0; for (int k = 0; k < 48; ++k) if (lpos(k) > p) ++n; return n < 63 ? n : 63; }
constexpr bool sched_ok() {
    for (int q = 0; q < 64; ++q) if (dpos(q) < 128 + ((q >> 2) + 1) * 8 + 4 || dpos(q) >= 256 + (q >> 2) * 8 - 2) return false;
    for (int j = 0; j < 8; ++j) if (lpos(16 + j) < (j | 1) * 8 + 5 || lpos(24 + j) < 64 + (j | 1) * 8 + 5 || lpos(40 + j) < 128 + (j | 1) * 8 + 5) return false;
    return lpos(0) >= 32 && lpos(32) > 113 && lpos(47) < 256 && dpos(63) >= 256 && dpos(63) - 256 < 112;
}
static_assert(sched_ok(), "schedule");

#define PLOAD(dst, soff) _Pragma("unroll") for (int t_ = 0; t_ < 4; ++t_) \
        dst[t_] = __builtin_bit_cast(f4, __builtin_amdgcn_raw_buffer_load_b128(rp, lane * 16 + t_ * 1024, (soff), 18))
#define OSTORE(j) __builtin_amdgcn_raw_buffer_store_b128(__builtin_bit_cast(u4, outv[j]), ro, voffo, oprev + (j) * 4096u, 17)
#define ACTIVATE() _Pragma("unroll") for (int j = 0; j < 4; ++j) { \
        outv[j] = (f4){silu(sum[0][j]), silu(sum[1][j]), silu(sum[2][j]), silu(sum[3][j])}; \
        if (bad) outv[j] = (f4){__builtin_nanf(""), __builtin_nanf(""), __builtin_nanf(""), __builtin_nanf("")}; }

__global__ __launch_bounds__(256, 1) void gemm_a2a(Args A) {
    extern __shared__ __attribute__((aligned(16))) unsigned char smem_[];
    LAS unsigned char* smem = (LAS unsigned char*)smem_;
    const int tid = threadIdx.x, lane = tid & 63, wid = __builtin_amdgcn_readfirstlane(tid >> 6);
    const int rank = A.rank & (NR - 1);
    unsigned char* ws = (unsigned char*)A.ws;
    const int xcd = blockIdx.x & 7, idx = blockIdx.x >> 3;
    const int mt = idx & 3, ni = idx >> 2, kq = xcd & 3, g = xcd >> 2;
    const __amdgpu_buffer_rsrc_t rx = __builtin_amdgcn_make_buffer_rsrc((void*)(ws + WS_XS), 0, (unsigned)(M * K * 2), 0x00020000);
    const __amdgpu_buffer_rsrc_t rp = __builtin_amdgcn_make_buffer_rsrc((void*)(ws + WS_PART), 0, PART_BYTES, 0x00020000);
    const unsigned lane16 = lane * 16;
    const unsigned voffo = (unsigned)(lane >> 4) * 16384u + (unsigned)(lane & 15) * 16u;
    const unsigned tk0 = ld_agt(ws + WS_TICKET);
    const __amdgpu_buffer_rsrc_t rxi = __builtin_amdgcn_make_buffer_rsrc((void*)A.x, 0, (unsigned)(M * K * 4), 0x00020000);
    const unsigned xvi = (unsigned)(lane >> 3) * (K * 4) + (lane & 7) * 16, xvo = ((lane >> 2) & 1) * XS + (lane & 3) * 256 + (lane >> 3) * 16;
    const unsigned xbi = (unsigned)(mt * 256 + wid * 64) * (K * 4) + kq * 8192, xbo = (unsigned)(mt * 256 + kq * 64) * XS + wid * 4096;
#define XLD(dst, pr) _Pragma("unroll") for (int f_ = 0; f_ < 8; ++f_) _Pragma("unroll") for (int e_ = 0; e_ < 2; ++e_) \
        dst[f_][e_] = __builtin_bit_cast(f4, __builtin_amdgcn_raw_buffer_load_b128(rxi, xvi + e_ * 128, xbi + f_ * (8 * K * 4) + (pr) * 256, 0))
#define XST(src, pr) _Pragma("unroll") for (int f_ = 0; f_ < 8; ++f_) { \
        const u4 o_ = {cvt2(src[f_][0][0], src[f_][0][1]), cvt2(src[f_][0][2], src[f_][0][3]), cvt2(src[f_][1][0], src[f_][1][1]), cvt2(src[f_][1][2], src[f_][1][3])}; \
        __builtin_amdgcn_raw_buffer_store_b128(o_, rx, xvo + (f_ >> 1) * 1024 + (f_ & 1) * 128, xbo + (pr) * (2 * XS), 16); }
#define XPR(p) (g ? 31 - (p) : (p))
#define XFW(pr) ((((pr) >> 1) * 4 + wid) * 8 + ((pr) & 1) * 4)
    {
        f4 v[8][2];
        XLD(v, XPR(ni));
        XST(v, XPR(ni));
    }
    const unsigned call = (__builtin_amdgcn_readfirstlane(tk0) >> 8) + 1u;
    unsigned char* xf = ws + WS_XF + (mt * 4 + kq) * 512;
    unsigned char* tmo = ws + WS_TMO;
    bool bad = false;
    const int role = g ^ (ni & 1);

    if (ni < 7) {
        const __amdgpu_buffer_rsrc_t rw = __builtin_amdgcn_make_buffer_rsrc((void*)A.w, 0, (unsigned)((size_t)K * N * 4), 0x00020000);
        const __amdgpu_buffer_rsrc_t rf = __builtin_amdgcn_make_buffer_rsrc((void*)(ws + WS_PF), 0, 65536u, 0x00020000);
        unsigned voffw[8];
#pragma unroll
        for (int j = 0; j < 8; ++j) { voffw[j] = (unsigned)((lane >> 4) * 4 + (j & 3) + 32 * (j >> 2)) * (N * 4) + (unsigned)(lane & 15) * 16; asm volatile("" : "+v"(voffw[j])); }
        const unsigned voffx = wid * 4096 + lane16;
        const unsigned ldsx = (unsigned)(size_t)smem + wid * 4096;
        const int n = (NTAB >> (4 * ni)) & 15, e0 = role ? 16 - n : n, e1 = e0 + 16, nwin = (role && ni < 3) ? e1 : e0 + 32;
        const int c0 = strip_of(column_of(0, ni, role), wid, rank) * 256, c1 = strip_of(column_of(1, ni, role), wid, rank) * 256,
                  c2 = strip_of(column_of(2, ni, role), wid, rank) * 256;
        unsigned wrel[4], xrel[4];
#pragma unroll
        for (int s = 0; s < 4; ++s) { const int st = ((g ? 3 - s : s) + mt) & 3; wrel[s] = (unsigned)((st >> 1) * 64 + (st & 1) * 16) * (N * 4); xrel[s] = (unsigned)(mt * 256 + st) * XS; }
#define WIN(i) (kq * 16 + (g ? 15 - (i) : (i)))
#define PIN(x) asm volatile("" : "+s"(x))
#define GLDS(so, i) asm volatile("buffer_load_dwordx4 %0, %1, %2 offen offset:%c3 sc1 lds" :: "v"(voffx), "s"(rx), "s"(so), "i"((i) * 1024) : "memory")
#define WLD(rr, h, j, so) asm volatile("buffer_load_dwordx4 %0, %1, %2, %3 offen" : "=v"(rr[h][j]) : "v"(voffw[j]), "s"(rw), "s"(so) : "memory")
#define CVT(dst, rr, h, t, q) do { unsigned c_; asm volatile("v_cvt_pk_f16_f32 %0, %1, %2" : "=v"(c_) : "v"(rr[h][2 * (q)][t]), "v"(rr[h][2 * (q) + 1][t])); dst[h][t][q] = c_; } while (0)
#define MFMA(m, t, av, bv) asm volatile("v_mfma_f32_16x16x32_f16 a[%c2:%c3], %0, %1, a[%c2:%c3]" :: "v"(av), "v"(bv), "i"(((m) * 4 + (t)) * 4), "i"(((m) * 4 + (t)) * 4 + 3))
#define MFMA0(m, t, av, bv) asm volatile("v_mfma_f32_16x16x32_f16 a[%c2:%c3], %0, %1, 0" :: "v"(av), "v"(bv), "i"(((m) * 4 + (t)) * 4), "i"(((m) * 4 + (t)) * 4 + 3))
#define LDA(dst, aaddr, off) dst = *(const LAS u4*)(smem + (aaddr) + (off))
        u4 a[4][2], b0[2][4], b1[2][4], r0[2][8], r1[2][8];
        u2 xv; const unsigned lane8 = lane * 8;
        asm volatile("" ::: ACLOB);
        unsigned wb1 = ((unsigned)WIN(1) << 22) + (unsigned)(e0 > 1 ? c0 : c1), xb1 = (unsigned)WIN(1) << 16;
        {
            const unsigned wb0 = ((unsigned)WIN(0) << 22) + (unsigned)c0, xb0 = (unsigned)WIN(0) << 16;
#pragma unroll
            for (int j = 0; j < 8; ++j) WLD(r0, 0, j, wb0 + wrel[0]);
            WAIT_V(8);
            if (lane == 0) st_agt(xf + XFW(XPR(ni)), call);
#pragma unroll
            for (int j = 0; j < 8; ++j) WLD(r0, 1, j, wb0 + wrel[1]);
#pragma unroll
            for (int h = 0; h < 2; ++h)
#pragma unroll
                for (int j = 0; j < 8; ++j) WLD(r1, h, j, wb0 + wrel[2 + h]);
            if (!wave_wait<false, 2>(xf + (g ? 448 : 0) + (lane & 2) * 16 + wid * 8 + (lane & 1) * 4, call, tmo, call)) bad = true;
            KNOWN_VM0();
            WAIT_V(0);
#pragma unroll
            for (int s = 0; s < 4; ++s) {
                const unsigned so = xb0 + xrel[s];
                asm volatile("s_mov_b32 m0, %0\n\ts_nop 0" :: "s"(ldsx + s * XS) : "memory");
                GLDS(so, 0); GLDS(so, 1); GLDS(so, 2); GLDS(so, 3);
            }
            SCHED();
#pragma unroll
            for (int h = 0; h < 2; ++h)
#pragma unroll
                for (int q = 0; q < 4; ++q)
#pragma unroll
                    for (int t = 0; t < 4; ++t) CVT(b0, r0, h, t, q);
            SCHED();
#pragma unroll
            for (int j = 0; j < 8; ++j) WLD(r0, 0, j, wb1 + wrel[0]);
            KNOWN_LGKM0();
            WAIT_V(8); BAR();
#pragma unroll
            for (int m = 0; m < 2; ++m) { LDA(a[m][0], lane16, m * 1024); LDA(a[m][1], lane16, XS + m * 1024); }
        }
#define DUMP(q) asm volatile("buffer_store_dwordx4 a[%c3:%c4], %0, %1, %2 offen offset:%c5 sc1" :: "v"(lane16), "s"(rp), "s"(ob + (((q) & 63) >> 2) * 4096u), \
            "i"(((q) & 63) * 4), "i"(((q) & 63) * 4 + 3), "i"(((q) & 3) * 1024) : "memory")
#define DUMPC(q) asm volatile("s_bitcmp1_b32 %2, 31\n\ts_cbranch_scc1 1f\n\tbuffer_store_dwordx4 a[%c3:%c4], %0, %1, %2 offen offset:%c5 sc1\n1:" \
            :: "v"(lane16), "s"(rp), "s"(ob + (((q) & 63) >> 2) * 4096u), "i"(((q) & 63) * 4), "i"(((q) & 63) * 4 + 3), "i"(((q) & 3) * 1024) : "memory", "scc")
#define PUBLISH() asm volatile("buffer_store_dword %0, %1, %2, %3 offen sc1" :: "v"(callv), "v"(zerov), "s"(rf), "s"(pend) : "memory")
#define GAP(MODE, B, m, e) do { constexpr int p_ = (B) * 128 + (m) * 8 + (e), j_ = lat(p_), jn_ = lat(p_ + 1), q_ = dat(p_ + ((MODE) == 1 ? 256 : 0)); \
            if (j_ >= 40) WLD(r0, 0, j_ & 7, so); else if (j_ >= 32) GLDS(so, j_ & 3); \
            else if (j_ >= 24) WLD(r1, 1, j_ & 7, so); else if (j_ >= 16) WLD(r1, 0, j_ & 7, so); \
            else if (j_ >= 8) GLDS(so, j_ & 3); else if (j_ >= 0) WLD(r0, 1, j_ & 7, so); \
            if (jn_ == 0 || jn_ == 16 || jn_ == 24) { so = wb1 + wrel[jn_ == 0 ? 1 : jn_ >> 3]; PIN(so); } \
            if (jn_ == 40) { so = wb2 + wrel[0]; PIN(so); } \
            if (jn_ == 8 || jn_ == 12 || jn_ == 32 || jn_ == 36) { so = xb1 + xrel[(jn_ >= 32 ? 2 : 0) + ((jn_ >> 2) & 1)]; PIN(so); \
                asm volatile("s_add_u32 m0, %0, %1" :: "s"(lds2), "i"(((jn_ >= 32 ? 2 : 0) + ((jn_ >> 2) & 1)) * XS) : "memory", "scc"); } \
              \
            if (p_ == 2) { t2 = i + 2; PIN(t2); } \
            if (p_ == 5) { cc = t2 >= e0 ? c1 : c0; PIN(cc); } \
            if (p_ == 6) { cc = t2 >= e1 ? c2 : cc; PIN(cc); } \
            if (p_ == 14) { kp = (kp + kstep) & 15; PIN(kp); } \
            if (p_ == 18) { kw = kp | (kq * 16); PIN(kw); } \
            if (p_ == 21) { wb2 = ((unsigned)kw << 22) + (unsigned)cc; PIN(wb2); } \
            if (p_ == 22) { xb2 = (unsigned)kw << 16; PIN(xb2); } \
            if (p_ == 26) { lds2 ^= 4 * XS; PIN(lds2); } \
            if (p_ == 29) { wb2 = t2 < nwin ? wb2 : OOB; xb2 = t2 < nwin ? xb2 : OOB; PIN(wb2); PIN(xb2); }     \
            if (p_ == 109 || p_ == 237) { an ^= 4 * XS; asm volatile("" : "+v"(an)); } \
            if ((MODE) == 1 && q_ >= 0) DUMPC(q_); else if ((MODE) == 2 && q_ >= 0) DUMP(q_); \
            SCHED(); } while (0)
#define ITER(MODE, B, bc, bn, rc, m, ac) do { \
            constexpr bool FIRST = (MODE) == 1; \
            constexpr int t0 = (m & 1) ? 3 : 0, dt = (m & 1) ? -1 : 1, hh = m >> 3, qq = (m >> 1) & 3; \
            if (m == 14) { WAIT_V((MODE) == 1 ? dwait(112) : vwait((B) ? 15 : 39, (B) * 128 + 112, MODE)); BAR(); } \
            if (FIRST) MFMA0(m, t0, a[m & 3][0], bc[0][t0]); else MFMA(m, t0, a[m & 3][0], bc[0][t0]); \
            if (m < 14) LDA(a[(m + 2) & 3][0], ac, (m + 2) * 1024); else LDA(a[(m + 2) & 3][0], an, (m - 14) * 1024); \
            GAP(MODE, B, m, 0); \
            MFMA(m, t0, a[m & 3][1], bc[1][t0]); \
            if (m == 0) WAIT_V(vwait((B) ? 47 : 23, (B) * 128 + 1, MODE)); \
            if (m == 8) WAIT_V(vwait((B) ? 7 : 31, (B) * 128 + 65, MODE)); \
            CVT(bn, rc, hh, (m & 1) * 2, qq); \
            GAP(MODE, B, m, 1); \
            if (FIRST) MFMA0(m, t0 + dt, a[m & 3][1], bc[1][t0 + dt]); else MFMA(m, t0 + dt, a[m & 3][1], bc[1][t0 + dt]); \
            if ((MODE) == 1 && m == 14) PUBLISH(); \
            GAP(MODE, B, m, 2); \
            MFMA(m, t0 + dt, a[m & 3][0], bc[0][t0 + dt]); \
            if (m < 14) LDA(a[(m + 2) & 3][1], ac, XS + (m + 2) * 1024); else LDA(a[(m + 2) & 3][1], an, XS + (m - 14) * 1024); \
            GAP(MODE, B, m, 3); \
            if (FIRST) MFMA0(m, t0 + 2 * dt, a[m & 3][0], bc[0][t0 + 2 * dt]); else MFMA(m, t0 + 2 * dt, a[m & 3][0], bc[0][t0 + 2 * dt]); \
            CVT(bn, rc, hh, (m & 1) * 2 + 1, qq); \
            GAP(MODE, B, m, 4); \
            if ((B) && m == 0) asm volatile("global_load_dwordx2 %0, %1, %2 sc1" : "=v"(xv) : "v"(lane8), "s"(xf) : "memory"); \
            MFMA(m, t0 + 2 * dt, a[m & 3][1], bc[1][t0 + 2 * dt]); \
            GAP(MODE, B, m, 5); \
            if (FIRST) MFMA0(m, t0 + 3 * dt, a[m & 3][1], bc[1][t0 + 3 * dt]); else MFMA(m, t0 + 3 * dt, a[m & 3][1], bc[1][t0 + 3 * dt]); \
            GAP(MODE, B, m, 6); \
            MFMA(m, t0 + 3 * dt, a[m & 3][0], bc[0][t0 + 3 * dt]); \
            GAP(MODE, B, m, 7); \
        } while (0)
#define STEP(MODE, B, bc, bn, rc, ac, an_) do { unsigned& an = an_; \
        ITER(MODE, B, bc, bn, rc, 0, ac); ITER(MODE, B, bc, bn, rc, 1, ac); ITER(MODE, B, bc, bn, rc, 2, ac); ITER(MODE, B, bc, bn, rc, 3, ac); \
        ITER(MODE, B, bc, bn, rc, 4, ac); ITER(MODE, B, bc, bn, rc, 5, ac); ITER(MODE, B, bc, bn, rc, 6, ac); ITER(MODE, B, bc, bn, rc, 7, ac); \
        ITER(MODE, B, bc, bn, rc, 8, ac); ITER(MODE, B, bc, bn, rc, 9, ac); ITER(MODE, B, bc, bn, rc, 10, ac); ITER(MODE, B, bc, bn, rc, 11, ac); \
        ITER(MODE, B, bc, bn, rc, 12, ac); ITER(MODE, B, bc, bn, rc, 13, ac); ITER(MODE, B, bc, bn, rc, 14, ac); ITER(MODE, B, bc, bn, rc, 15, ac); \
    } while (0)
        int i = 0;
        unsigned pend = OOB, ob = OOB;
        unsigned so = 0, wb2 = 0, xb2 = 0, lds2 = ldsx, aA = lane16, aB = lane16 + 6 * XS, callv = call, zerov = 0;
        unsigned tprev = (unsigned)__builtin_amdgcn_s_memrealtime();
        KNOWN_LGKM0();
        int t2 = 0, cc = 0, kw = 0, kp = WIN(1) & 15;
        const int kstep = g ? -1 : 1;
        asm volatile("" : "+v"(callv), "+v"(zerov));
#define STEP_A(MODE) STEP(MODE, 0, b0, b1, r1, aA, aB)
#define STEP_B(MODE) do { STEP(MODE, 1, b1, b0, r0, aB, aA); \
            wb1 = wb2; xb1 = xb2; ++i; \
            if (i <= PACEN) { const int pc_ = PACE0 - i * PACED; const unsigned due = tprev + (pc_ > PACEF ? pc_ : PACEF); \
                while ((int)((tprev = (unsigned)__builtin_amdgcn_s_memrealtime()) - due) < 0) __builtin_amdgcn_s_sleep(1); \
                KNOWN_LGKM0(); } \
            asm volatile("s_waitcnt vmcnt(%1)" : "+v"(xv) : "n"(lafter(132, MODE)) : "memory"); \
            if (i < 15) {     \
                const int pw = g ? 14 - i : i + 1; \
                if (((__ballot(xv[0] >= call && xv[1] >= call) >> (pw * 4 + wid)) & 1) == 0) { \
                    if (!wave_wait<false, 4>(xf + pw * 32 + wid * 8 + (lane & 1) * 4, call, tmo, call)) bad = true; \
                    KNOWN_LGKM0(); } \
                SCHED(); \
            } } while (0)
        for (int jb = 0; i < nwin; ++jb) {
            const int iend = jb == 0 ? e0 : jb == 1 ? e1 : nwin;
            STEP_A(1);
            while (i + 1 < iend) { STEP_B(0); STEP_A(0); }
            const int o = column_of(jb, ni, role), pslot = jb == 0 ? role * 4 + kq : kq;
            ob = part_slot(o, mt) + pslot * PSTRIDE + wid * 65536u;
            STEP_B(2);
            pend = bad ? OOB : (unsigned)(((o * 4 + mt) * 4 + wid) * 128 + pslot * 4);
        }
        asm volatile("s_nop 15"); SCHED();
#pragma unroll
        for (int q = dfirst(); q < 64; ++q) DUMP(q);
        WAIT_V(0);
        PUBLISH();
        if (tid == 0) __hip_atomic_fetch_add((GAS unsigned*)(ws + WS_TICKET), 1u, __ATOMIC_RELAXED, __HIP_MEMORY_SCOPE_AGENT);
        if (blockIdx.x == 0 && wid == 0) {
            if (!wave_wait<true, 4>(ws + WS_DONE + ((rank + 1 + lane % 7) & (NR - 1)) * 128, call * (HWV + TWV), tmo, call) && lane == 0)
                ((float*)A.out[rank])[(size_t)rank * 1024 * 1024] = __builtin_nanf("");
        }
        if (role && ni >= 3) {
            const int o = column_of(2, ni, 1);
            const __amdgpu_buffer_rsrc_t ro = __builtin_amdgcn_make_buffer_rsrc(A.out[rank], 0, 8192u * 1024 * 4, 0x00020000);
            if (!wave_wait<false, 4>(ws + WS_PF + ((o * 4 + mt) * 4 + wid) * 128 + (lane & 3) * 4, call, tmo, call)) bad = true;
            const unsigned pb = part_slot(o, mt) + wid * 65536u + kq * 16384u;
            f4 v[4][4], sum[4], outv[4];
#pragma unroll
            for (int c = 0; c < 4; ++c) { PLOAD(v[c], pb + c * PSTRIDE); outv[c] = (f4){0.f, 0.f, 0.f, 0.f}; }
            unsigned oprev = OOB;
#pragma nounroll
            for (int b = 0; b < 4; ++b) {
                const unsigned nx = b < 3 ? pb + (b + 1) * 4096u : OOB;
#pragma unroll
                for (int c = 1; c < 4; ++c) {
#pragma unroll
                    for (int u = 0; u < 4; ++u) { if (c == 1) sum[u] = v[0][u] + v[1][u]; else sum[u] += v[c][u]; }
#pragma unroll
                    for (int u = 0; u < 4; ++u) asm volatile("" : "+v"(sum[u]));
                    SCHED();
                    if (c == 1) PLOAD(v[0], nx);
                    PLOAD(v[c], nx + c * PSTRIDE);
                    if (c == 1) OSTORE(0);
                    OSTORE(c);
                    SCHED();
                }
                ACTIVATE();
                oprev = (unsigned)(rank * 1024 + mt * 256 + kq * 64 + b * 16) * 4096u + ((o - 28) * 4 + wid) * 256u;
                SCHED();
            }
#pragma unroll
            for (int j = 0; j < 4; ++j) OSTORE(j);
        }
    } else {
        WAIT_V(0);
        if (lane == 0) st_agt(xf + XFW(XPR(ni)), call);
        if (blockIdx.x == 224 && wid == 0 && lane < NR) st_sys((unsigned char*)A.wsp[lane & (NR - 1)] + WS_START + rank * 128, call);
        f4 u0[8][2], u1[8][2];
        XLD(u0, XPR(8)); XLD(u1, XPR(9));
#pragma nounroll
        for (int p = 8; p < 14; p += 2) {
            XST(u0, XPR(p)); SCHED(); XLD(u0, XPR(p + 2)); SCHED();
            XST(u1, XPR(p + 1)); SCHED(); XLD(u1, XPR(p + 3)); SCHED();
            WAIT_V(16);
            if (lane < 2) st_agt(xf + XFW(XPR(p) & ~1) + lane * 4, call);
        }
        XST(u0, XPR(14)); XST(u1, XPR(15));
        WAIT_V(0);
        if (lane < 2) st_agt(xf + XFW(XPR(14) & ~1) + lane * 4, call);
        BAR();
        if (tid == 0) __hip_atomic_fetch_add((GAS unsigned*)(ws + WS_TICKET), 1u, __ATOMIC_RELAXED, __HIP_MEMORY_SCOPE_AGENT);
        KNOWN_VM0();
    }
    if (ni == 7 || (role && ni < 3)) {
        const bool hlp = ni == 7;
        const int h = hlp ? (xcd * 4 + mt) * 4 + wid : wid < 2 ? ((ni * 4 + kq) * 4 + mt) * 2 + wid : 999, pi = h % 7, hs = hlp ? HWV : TWV;
        const int hj = h / 7 + (hlp ? 0 : HB), dst = (rank + 1 + pi) & (NR - 1);
        if (h < 7 * hs) {
            if (!wave_wait<true, 8>(ws + WS_START + dst * 128, call, tmo, call)) bad = true;
            const __amdgpu_buffer_rsrc_t ro = __builtin_amdgcn_make_buffer_rsrc(A.out[dst], 0, 8192u * 1024 * 4, 0x00020000);
#define IT_Q(t) (pi + 7 * ((hj + hs * (t)) >> 6))
#define IT_MT(t) (((hj + hs * (t)) >> 4) & 3)
#define IT_FLAG(t, hf) (ws + WS_PF + (((IT_Q(t) >> 2) * 4 + IT_MT(t)) * 4 + (IT_Q(t) & 3)) * 128 + ((hf) * 4 + (lane & 3)) * 4)
#define IT_HF(t) (IT_Q(t) < 28 ? 1 : 0)
#define IT_PB(t, hf) (part_slot(IT_Q(t) >> 2, IT_MT(t)) + (hf) * (4 * PSTRIDE) + (IT_Q(t) & 3) * 65536u + ((hj + hs * (t)) & 15) * 4096u)
            f4 v[8][4], sum[4], outv[4];
            asm volatile("" ::: ACLOB);
            if (hlp) {
#pragma unroll
                for (int t = 14; t >= 0; t -= 2) {
                    const bool va = hj + HWV * t < 256, vb = t > 0;
                    if (va && !wave_wait<false, 4, true>(IT_FLAG(t, 0), call, tmo, call)) bad = true;
                    const unsigned pa = va ? IT_PB(t, 0) : OOB;
#pragma unroll
                    for (int i = 0; i < 4; ++i) PLOAD(v[i], pa + i * PSTRIDE);
                    if (vb) {
                        if (!wave_wait<false, 4, true>(IT_FLAG(t - 1, 0), call, tmo, call)) bad = true;
#pragma unroll
                        for (int i = 0; i < 4; ++i) PLOAD(v[4 + i], IT_PB(t - 1, 0) + i * PSTRIDE);
                    }
                    SCHED();
#pragma unroll
                    for (int b = 0; b < (vb ? 2 : 1); ++b)
#pragma unroll
                        for (int u = 0; u < 4; ++u) {
                            const f4 x = (v[b * 4][u] + v[b * 4 + 1][u]) + (v[b * 4 + 2][u] + v[b * 4 + 3][u]);
#pragma unroll
                            for (int j = 0; j < 4; ++j) asm volatile("v_accvgpr_write_b32 a[%c1], %0" :: "v"(x[j]), "i"((t - b) * 16 + u * 4 + j));
                        }
                    SCHED();
                }
            }
            const int nit = ((hlp ? HB : 1024) - hj + hs - 1) / hs, tpre = hlp ? 0 : 64;
            bool h0 = false, h1 = false;
            unsigned fv = 0;
#define PRE_CASE(T) case T: _Pragma("unroll") for (int u = 0; u < 16; ++u) { float f; asm volatile("v_accvgpr_read_b32 %0, a[%c1]" : "=v"(f) : "i"((T) * 16 + u)); sum[u >> 2][u & 3] = f; } break;
#define ITEM(B, hv) do { \
                if (!hv) { \
                    if (!wave_wait<false, 4, true>(IT_FLAG(t, IT_HF(t)), call, tmo, call)) bad = true; \
                    fv = ld_agt(IT_FLAG(t + 2, IT_HF(t + 2))); \
                    const unsigned pb = IT_PB(t, IT_HF(t)); \
                    _Pragma("unroll") for (int i = 0; i < 4; ++i) PLOAD(v[(B) * 4 + i], pb + i * PSTRIDE); \
                    KNOWN_VM0(); \
                } \
                SCHED(); \
                switch (t + tpre) { \
                    PRE_CASE(B) PRE_CASE(2 + B) PRE_CASE(4 + B) PRE_CASE(6 + B) PRE_CASE(8 + B) PRE_CASE(10 + B) PRE_CASE(12 + B) PRE_CASE(14 + B) \
                    default: _Pragma("unroll") for (int u = 0; u < 4; ++u) sum[u] = (f4){0.f, 0.f, 0.f, 0.f}; \
                } \
                hv = t + 2 >= nit || __all(fv >= call); \
                __builtin_amdgcn_fence(__ATOMIC_ACQUIRE, "wavefront"); \
                const unsigned nb = (hv && t + 2 < nit) ? IT_PB(t + 2, IT_HF(t + 2)) : OOB;     \
                fv = ld_agt(IT_FLAG(t + 3, IT_HF(t + 3))); \
                _Pragma("unroll") for (int c = 0; c < 4; ++c) { \
                    _Pragma("unroll") for (int u = 0; u < 4; ++u) sum[u] += v[(B) * 4 + c][u]; \
                    _Pragma("unroll") for (int u = 0; u < 4; ++u) asm volatile("" : "+v"(sum[u])); \
                    SCHED(); \
                    PLOAD(v[(B) * 4 + c], nb + c * PSTRIDE); \
                    SCHED(); \
                } \
                ACTIVATE(); \
                const unsigned oprev = t < nit ? (unsigned)(rank * 1024 + ((hj + hs * t) & 63) * 16) * 4096u + ((((hj + hs * t) >> 6) + pi) & 15) * 256u : OOB; \
                SCHED(); \
                BAR();     \
                _Pragma("unroll") for (int j = 0; j < 4; ++j) OSTORE(j); \
                SCHED(); \
                ++t; \
            } while (0)
#pragma unroll
            for (int u = 240; u < 256; ++u) asm volatile("v_accvgpr_write_b32 a[%c0], 0" :: "i"(u));
            for (int t = 0; t < nit;) {
                ITEM(0, h0);
                ITEM(1, h1);
            }
            WAIT_V(0);
            if (lane == 0) __hip_atomic_fetch_add((GAS unsigned*)((unsigned char*)A.wsp[dst] + WS_DONE + rank * 128), 1u, __ATOMIC_RELAXED, __HIP_MEMORY_SCOPE_SYSTEM);
        }
    }
}

extern "C" void kernel_launch(int rank, int nranks, ncclComm_t comm, hipStream_t stream,
                              void* const* d_in, const int* in_sizes, int n_in,
                              void* d_out, int out_size, const int* in_dtypes, int out_dtype,
                              void* const* d_out_all, void* const* d_ws_all, size_t ws_bytes) {
    if (nranks != NR || (unsigned)rank >= (unsigned)NR || ws_bytes < WS_NEED || n_in < 2 || in_sizes[0] != M * K || in_sizes[1] != K * N || out_size != 8192 * 1024) {
        fprintf(stderr, "unexpected problem shape\n");
        return;
    }
    static bool attr = false;
    if (!attr) { HIP_CHECK(hipFuncSetAttribute((const void*)gemm_a2a, hipFuncAttributeMaxDynamicSharedMemorySize, LDS_BYTES)); attr = true; }
    Args a; a.x = (const float*)d_in[0]; a.w = (const float*)d_in[1]; a.ws = d_ws_all[rank]; a.rank = rank; a.pad = 0;
    for (int p = 0; p < NR; ++p) { a.out[p] = d_out_all[p]; a.wsp[p] = d_ws_all[p]; }
    hipLaunchKernelGGL(gemm_a2a, dim3(256), dim3(256), LDS_BYTES, stream, a);
    HIP_CHECK(hipGetLastError());
}
```
